```python
import numpy as np

kernel_name = "gpu_gemm_rs_m4096_k4096_n16384_bf16_reluqe4m3"


ABSMAX_RTOL = 0.09

COMPUTE_DTYPE = "bfloat16"
EPILOGUE = "relu_quant_e4m3"
IN_QUANT = None
OUT_QUANT = "e4m3"
IN_DTYPES = {"x": "bfloat16", "w_mat": "bfloat16"}
WORLD_SIZE = 8

def _to_bf16(x):
    u = np.ascontiguousarray(x, dtype=np.float32).view(np.uint32)
    return ((u + 0x7FFF + ((u >> 16) & 1)) & 0xFFFF0000).view(np.float32)

def setup_inputs():

    x = _to_bf16(np.random.randn(4096, 4096).astype(np.float32))
    w_mat = _to_bf16((np.random.randn(4096, 16384) / np.sqrt(4096)).astype(np.float32))
    return {"x": x, "w_mat": w_mat}

def _snap_fp8(x, mant_bits, max_abs, min_normal):
    a = np.abs(x.astype(np.float64))
    _m, _e = np.frexp(a)
    q = np.where(a >= min_normal, np.ldexp(1.0, _e - 1 - mant_bits),
                 min_normal * 2.0 ** -mant_bits)
    snapped = np.minimum(np.round(a / q) * q, max_abs)
    return (np.sign(x) * snapped).astype(np.float32)
def _quant_dequant(y, amax):
    scale = amax / 448.0
    q = _snap_fp8(y / scale, 3, 448.0, 2.0 ** -6)
    return (q * scale).astype(np.float32)
def _epilogue(y):
    y = np.maximum(y, 0.0)
    return _quant_dequant(y, np.abs(y).max())

def reference(x, w_mat):
    y = (x @ w_mat).astype(np.float32)
    return _epilogue(y)
```

```cpp
#include <hip/hip_runtime.h>
#include <rccl/rccl.h>
#include <cstdio>
#include <cstdint>

#define HIP_CHECK(expr) do { hipError_t _e = (expr); if (_e != hipSuccess) { \
    fprintf(stderr, "HIP error %s at %s:%d\n", hipGetErrorString(_e), __FILE__, __LINE__); \
    return; } } while (0)

namespace {

#define LAS __attribute__((address_space(3)))
typedef short bf16x8 __attribute__((ext_vector_type(8)));
typedef short s16x4 __attribute__((ext_vector_type(4)));
typedef float f32x4 __attribute__((ext_vector_type(4)));
typedef unsigned u32x4 __attribute__((ext_vector_type(4)));
typedef unsigned u32x2 __attribute__((ext_vector_type(2)));
typedef unsigned long long u64;

constexpr int NR = 8;
constexpr int GM = 4096, GN = 16384;
constexpr int KS = 512, NS = 2048, MS = 512;
constexpr unsigned PIECE = 32768;
constexpr unsigned HALFB = 16384;
constexpr unsigned OFF_X = 0;
constexpr unsigned OFF_WIN = 32u << 20;
constexpr unsigned OFF_WEX = 48u << 20;
constexpr unsigned OFF_CTL = 64u << 20;
constexpr unsigned CTL_SEQ = 0;
constexpr unsigned CTL_TMO = 32768;
constexpr unsigned CTL_EXPF = 36864;
constexpr unsigned CTL_XCNT = 45056;
constexpr unsigned CTL_WCNT = 49152;
constexpr unsigned CTL_AMAX = 53248;
constexpr unsigned CTL_DONE = 61440;
constexpr unsigned CTL_DBG = 65536;
constexpr size_t WS_NEED = (size_t)OFF_CTL + (1u << 20);
constexpr unsigned OUT_BYTES = (unsigned)MS * GN;

constexpr int N_EXP = 72, N_PULL = 56, N_GEMM = 128, N_WG = N_EXP + N_PULL + N_GEMM;
constexpr int LDS_STAGE = 131072, LDS_BYTES = LDS_STAGE + 256;
constexpr u64 DEADLINE = 50000000ull;

struct Params {
    const char* x;
    const char* w;
    char* ws[NR];
    char* out[NR];
    int rank;
};

__device__ __forceinline__ __amdgpu_buffer_rsrc_t mk_rsrc(const void* p, unsigned bytes) {
    return __builtin_amdgcn_make_buffer_rsrc(const_cast<void*>(p), 0, bytes, 0x00020000);
}
constexpr int AUX_SYS = 17;
constexpr int AUX_AGT = 16;

__device__ __forceinline__ unsigned ld_sys(const void* p) { return __hip_atomic_load((const unsigned*)p, __ATOMIC_RELAXED, __HIP_MEMORY_SCOPE_SYSTEM); }
__device__ __forceinline__ unsigned ld_agt(const void* p) { return __hip_atomic_load((const unsigned*)p, __ATOMIC_RELAXED, __HIP_MEMORY_SCOPE_AGENT); }
__device__ __forceinline__ void st_sys(void* p, unsigned v) { __hip_atomic_store((unsigned*)p, v, __ATOMIC_RELAXED, __HIP_MEMORY_SCOPE_SYSTEM); }
__device__ __forceinline__ void add_agt(void* p, unsigned v) { __hip_atomic_fetch_add((unsigned*)p, v, __ATOMIC_RELAXED, __HIP_MEMORY_SCOPE_AGENT); }
#define WAIT_V(n) asm volatile("s_waitcnt vmcnt(" #n ")" ::: "memory")
#define WAIT_L(n) asm volatile("s_waitcnt lgkmcnt(" #n ")" ::: "memory")
#define BAR __builtin_amdgcn_s_barrier()
#define SCHED __builtin_amdgcn_sched_barrier(0)

__device__ __forceinline__ bool give_up(char* ctl, unsigned call, u64 t0, unsigned& spins) {
    if ((++spins & 31u) != 0u) return false;
    const unsigned tm = __builtin_amdgcn_readfirstlane(ld_agt(ctl + CTL_TMO));
    if (tm == call) return true;
    if (__builtin_amdgcn_s_memrealtime() - t0 > DEADLINE) {
        __hip_atomic_store((unsigned*)(ctl + CTL_TMO), call, __ATOMIC_RELAXED, __HIP_MEMORY_SCOPE_AGENT);
        return true;
    }
    return false;
}

#define STAMP(k) do { } while (0)

__device__ __forceinline__ int rot_of(int j) { return (j * 16 + 3) / 7; }

__global__ void __launch_bounds__(512, 2) fused_gemm_quant(Params p) {
    extern __shared__ __attribute__((aligned(16))) unsigned char smem[];
    LAS unsigned char* lds = (LAS unsigned char*)smem;
    const int tid = threadIdx.x, lane = tid & 63;
    const int wid = __builtin_amdgcn_readfirstlane(tid >> 6);
    const int rank = p.rank, bid = blockIdx.x;
    char* const ws = p.ws[rank];
    char* const ctl = ws + OFF_CTL;
    STAMP(0);
    unsigned call_v = 0;
    if (lane == 0 && (wid == 0)) call_v = __hip_atomic_fetch_add((unsigned*)(ctl + CTL_SEQ + bid * 128), 1u, __ATOMIC_RELAXED, __HIP_MEMORY_SCOPE_AGENT) + 1u;
    LAS unsigned* lctl = (LAS unsigned*)(lds + LDS_STAGE);

    if (bid < N_EXP) {
        const __amdgpu_buffer_rsrc_t rx = mk_rsrc(p.x, (unsigned)GM * KS * 2), rw = mk_rsrc(p.w, (unsigned)KS * GN * 2);
        const __amdgpu_buffer_rsrc_t rd = mk_rsrc(ws, OFF_CTL);
        constexpr int PITCH = 528;
        for (int q = bid; q < 640; q += N_EXP) {
            const int c = q / 80, idx = q % 80;
            const bool isx = idx >= 56 && idx < 72;
            int dst = rank, pj = 0;
            if (isx) {
                pj = idx - 56;
                u32x4 v[4];
                const unsigned src = (unsigned)((pj * 256 + wid * 32 + (lane & 15)) * (KS * 2) + c * 128 + (lane >> 4) * 16);
#pragma unroll
                for (int u = 0; u < 4; ++u) v[u] = __builtin_amdgcn_raw_buffer_load_b128(rx, src + (u >> 1) * 16 * (KS * 2) + (u & 1) * 64, 0, 0);
                const unsigned d0 = OFF_X + (unsigned)((rank * 8 + c) * 16 + pj) * PIECE + wid * 4096 + lane * 16;
#pragma unroll
                for (int u = 0; u < 4; ++u) __builtin_amdgcn_raw_buffer_store_b128(v[u], rd, d0 + u * 1024, 0, AUX_SYS);
            } else {
                if (idx < 56) { dst = (rank + 1 + idx % 7) & 7; pj = idx / 7; } else { pj = idx - 72; }
                u32x4 v[4];
                const unsigned src = (unsigned)(c * 64 + (tid >> 5)) * (GN * 2) + (unsigned)(dst * NS + pj * 256) * 2 + (tid & 31) * 16;
#pragma unroll
                for (int u = 0; u < 4; ++u) v[u] = __builtin_amdgcn_raw_buffer_load_b128(rw, src + u * 16 * (GN * 2), 0, 0);
#pragma unroll
                for (int u = 0; u < 4; ++u) *(LAS u32x4*)(lds + ((tid >> 5) + 16 * u) * PITCH + (tid & 31) * 16) = v[u];
                __syncthreads();
                const int bj = wid >> 2, wc = wid & 3;
                const int g = lane >> 4, qq = (lane >> 2) & 3, pp = lane & 3;
                const unsigned d0 = (dst == rank ? OFF_WIN : OFF_WEX) + (unsigned)((dst * 8 + c) * 8 + pj) * PIECE + wid * 4096 + lane * 16;
#pragma unroll
                for (int u = 0; u < 4; ++u) {
                    const int n = u >> 1, kf = u & 1;
                    const int a = (kf * 32 + g * 8 + qq) * PITCH + (wc * 64 + pp * 16 + (bj * 2 + n) * 4) * 2;
                    const s16x4 lo = __builtin_amdgcn_ds_read_tr16_b64_v4i16((LAS s16x4*)(lds + a));
                    const s16x4 hi = __builtin_amdgcn_ds_read_tr16_b64_v4i16((LAS s16x4*)(lds + a + 4 * PITCH));
                    const u32x2 l2 = __builtin_bit_cast(u32x2, lo), h2 = __builtin_bit_cast(u32x2, hi);
                    const u32x4 o = {l2.x, l2.y, h2.x, h2.y};
                    __builtin_amdgcn_raw_buffer_store_b128(o, rd, d0 + u * 1024, 0, AUX_SYS);
                }
            }
            WAIT_V(0);
            __syncthreads();
            if (wid == 0) {
                const unsigned call = __builtin_amdgcn_readfirstlane(call_v);
                if (isx) {
                    if (lane < NR) {
                        if (lane == rank) add_agt(ctl + CTL_XCNT + (pj * 64 + rank * 8 + c) * 4, 32u);
                        else st_sys(p.ws[lane] + OFF_CTL + CTL_EXPF + (rank * 256 + c * 24 + 8 + pj) * 4, call);
                    }
                } else if (lane == 0) {
                    if (dst == rank) add_agt(ctl + CTL_WCNT + (pj * 64 + rank * 8 + c) * 4, 32u);
                    else st_sys(p.ws[dst] + OFF_CTL + CTL_EXPF + (rank * 256 + c * 24 + pj) * 4, call);
                }
            }
            if (q == bid) STAMP(1);
        }
        STAMP(2);
        if (bid == 0 && wid == 0) {
            const unsigned call = __builtin_amdgcn_readfirstlane(call_v);
            const u64 t0 = __builtin_amdgcn_s_memrealtime(); unsigned spins = 0; bool bad = false;
            for (;;) {
                const u64 f = __hip_atomic_load((const u64*)(ctl + CTL_DONE + lane * 8), __ATOMIC_RELAXED, __HIP_MEMORY_SCOPE_SYSTEM);
                if (__all((unsigned)f >= call && (unsigned)(f >> 32) >= call)) break;
                if (give_up(ctl, call, t0, spins)) { bad = true; break; }
                __builtin_amdgcn_s_sleep(4);
            }
            if (bad && lane == 0) p.out[rank][(size_t)rank * NS] = (char)0x7f;
            STAMP(3);
        }
        return;
    }

    if (tid == 0) { lctl[0] = call_v; lctl[2] = 0u; lctl[3] = 0u; }
    __syncthreads();
    const unsigned call = __builtin_amdgcn_readfirstlane(lctl[0]);

    if (bid < N_EXP + N_PULL) {
        const int q = bid - N_EXP, j = q % 7, g = (q / 7) * 8 + wid;
        const int src = (rank + 1 + j) & 7, rot = rot_of(j), hi = g >> 5;
        const __amdgpu_buffer_rsrc_t rs = mk_rsrc(p.ws[src], OFF_CTL), rd = mk_rsrc(ws, OFF_CTL);
        const unsigned voff = (unsigned)((g & 31) * 1024 + lane * 16);
        const char* ef = ctl + CTL_EXPF + src * 1024 + lane * 4;
        u64 m0 = 0, m1 = 0, m2 = 0;
        unsigned prev_cnt = 0;
        for (int m = 0; m < 96; ++m) {
            const int pp = 2 * m + hi, c = pp / 24, r = pp % 24, sc = src * 8 + c;
            int id; unsigned so, dof, cnt;
            if (r < 8) { id = c * 24 + r; so = OFF_WEX + (unsigned)((rank * 8 + c) * 8 + r) * PIECE; dof = OFF_WIN + (unsigned)(sc * 8 + r) * PIECE; cnt = CTL_WCNT + (r * 64 + sc) * 4; }
            else { const int i = (rot + r - 8) & 15; id = c * 24 + 8 + i; so = OFF_X + (unsigned)(sc * 16 + i) * PIECE; dof = so; cnt = CTL_XCNT + (i * 64 + sc) * 4; }
            const u64 mm = id < 64 ? m0 : (id < 128 ? m1 : m2);
            if (!((mm >> (id & 63)) & 1)) {
                const u64 t0 = __builtin_amdgcn_s_memrealtime(); unsigned spins = 0;
                for (;;) {
                    const unsigned f0 = ld_sys(ef), f1 = ld_sys(ef + 256), f2 = ld_sys(ef + 512);
                    m0 = __ballot(f0 >= call); m1 = __ballot(f1 >= call); m2 = __ballot(f2 >= call);
                    const u64 mn = id < 64 ? m0 : (id < 128 ? m1 : m2);
                    if ((mn >> (id & 63)) & 1) break;
                    if (give_up(ctl, call, t0, spins)) { m0 = m1 = m2 = ~0ull; break; }
                    __builtin_amdgcn_s_sleep(1);
                }
                if (m == 0) STAMP(1);
            }
            const u32x4 v = __builtin_amdgcn_raw_buffer_load_b128(rs, voff, so, AUX_SYS);
            WAIT_V(0);
            if (m > 0 && lane == 0) add_agt(ctl + prev_cnt, 1u);
            __builtin_amdgcn_raw_buffer_store_b128(v, rd, voff, dof, AUX_AGT);
            prev_cnt = cnt;
        }
        WAIT_V(0);
        if (lane == 0) add_agt(ctl + prev_cnt, 1u);
        STAMP(2);
        return;
    }

    const int gi = bid - (N_EXP + N_PULL), xg = gi & 7, ix = gi >> 3;
    const int ti = (xg >> 1) * 4 + (ix & 3), tj = (xg & 1) * 4 + (ix >> 2);
    const int wr = wid >> 2, wc = wid & 3;
    int scv;
    {
        int j0 = 0;
#pragma unroll
        for (int j = 1; j < 7; ++j) if (rot_of(j) <= ti) j0 = j;
        const int t2 = lane - 8, c = t2 / 7, k = t2 - c * 7;
        int j = j0 - k; if (j < 0) j += 7;
        scv = lane < 8 ? rank * 8 + lane : ((rank + 1 + j) & 7) * 8 + c;
    }
    const unsigned target = call * 32u;
    u64 rmask = 0;
    auto ensure_ready = [&](int t) {
        if (!((rmask >> t) & 1)) {
            if (wid == 0) {
                const u64 t0 = __builtin_amdgcn_s_memrealtime(); unsigned spins = 0;
                for (;;) {
                    const unsigned a = ld_agt(ctl + CTL_XCNT + (ti * 64 + scv) * 4), b = ld_agt(ctl + CTL_WCNT + (tj * 64 + scv) * 4);
                    rmask = __ballot(a >= target && b >= target);
                    if ((rmask >> t) & 1) break;
                    if (give_up(ctl, call, t0, spins)) { rmask = ~0ull; break; }
                    __builtin_amdgcn_s_sleep(1);
                }
                const u32x2 mv = {(unsigned)rmask, (unsigned)(rmask >> 32)};
                asm volatile("ds_write_b64 %0, %1" :: "v"(LDS_STAGE + 8), "v"(mv) : "memory");
            } else {
                for (;;) {
                    u32x2 mv;
                    asm volatile("ds_read_b64 %0, %1\n\ts_waitcnt lgkmcnt(0)" : "=v"(mv) : "v"(LDS_STAGE + 8) : "memory");
                    rmask = ((u64)(unsigned)__builtin_amdgcn_readfirstlane(mv.y) << 32) | (unsigned)__builtin_amdgcn_readfirstlane(mv.x);
                    if ((rmask >> t) & 1) break;
                    __builtin_amdgcn_s_sleep(1);
                }
            }
        }
    };
    const char* const xbase = ws + OFF_X + (size_t)ti * PIECE;
    const char* const wbase = ws + OFF_WIN + (size_t)tj * PIECE;
    auto a_of = [&](int t) { return xbase + (size_t)((unsigned)__builtin_amdgcn_readlane(scv, t) * (16u * PIECE)); };
    auto b_of = [&](int t) { return wbase + (size_t)((unsigned)__builtin_amdgcn_readlane(scv, t) * (8u * PIECE)); };
    const unsigned voff = (unsigned)tid * 16u, ldsw = (unsigned)wid * 1024u;
    const int aoff = wr * 8192 + lane * 16, boff = wc * 4096 + lane * 16;
#define SA(b, h) (((b) * 2 + (h)) * HALFB)
#define SB(b, h) ((4 + (b) * 2 + (h)) * HALFB)
#define STAGE(bufoff, gbase) do { _Pragma("unroll") for (int _i = 0; _i < 2; ++_i) \
        __builtin_amdgcn_global_load_lds((const unsigned*)((gbase) + voff + _i * 8192), (LAS unsigned*)(lds + (bufoff) + ldsw + _i * 8192), 16, 0, AUX_AGT); } while (0)
#define LDA(dst, b, h) do { _Pragma("unroll") for (int m = 0; m < 4; ++m) _Pragma("unroll") for (int k = 0; k < 2; ++k) dst[m][k] = *(const LAS bf16x8*)(lds + SA(b, h) + aoff + m * 2048 + k * 1024); } while (0)
#define LDB(dst, b, h) do { _Pragma("unroll") for (int n = 0; n < 2; ++n) _Pragma("unroll") for (int k = 0; k < 2; ++k) dst[n][k] = *(const LAS bf16x8*)(lds + SB(b, h) + boff + n * 2048 + k * 1024); } while (0)
#define MMA(ai, bj, At, Bt) do { __builtin_amdgcn_s_setprio(1); _Pragma("unroll") for (int m = 0; m < 4; ++m) _Pragma("unroll") for (int n = 0; n < 2; ++n) _Pragma("unroll") for (int k = 0; k < 2; ++k) \
        acc[ai][bj][m][n] = __builtin_amdgcn_mfma_f32_16x16x32_bf16(Bt[n][k], At[m][k], acc[ai][bj][m][n], 0, 0, 0); __builtin_amdgcn_s_setprio(0); } while (0)
    f32x4 acc[2][2][4][2];
#pragma unroll
    for (int a = 0; a < 2; ++a)
#pragma unroll
        for (int b = 0; b < 2; ++b)
#pragma unroll
            for (int m = 0; m < 4; ++m)
#pragma unroll
                for (int n = 0; n < 2; ++n) acc[a][b][m][n] = (f32x4){0.f, 0.f, 0.f, 0.f};
    bf16x8 At[4][2], B0[2][2], B1[2][2];
    constexpr int NT = 64;
    {
        ensure_ready(0);
        const char *a0 = a_of(0), *b0 = b_of(0);
        STAGE(SB(0, 0), b0); STAGE(SA(0, 0), a0); STAGE(SB(0, 1), b0 + HALFB); STAGE(SA(0, 1), a0 + HALFB);
        ensure_ready(1);
        if (wr == 1) BAR;
        WAIT_V(4); BAR;
        const char *a1 = a_of(1), *b1 = b_of(1);
        STAGE(SB(1, 0), b1); STAGE(SA(1, 0), a1); STAGE(SB(1, 1), b1 + HALFB);
        WAIT_V(6); BAR;
    }
    STAMP(1);
    for (int t = 0; t < NT - 2; t += 2) {
        const char* a1 = a_of(t + 1);
        LDB(B0, 0, 0); SCHED; LDA(At, 0, 0); STAGE(SA(1, 1), a1 + HALFB);
        WAIT_L(8); BAR; WAIT_L(0); MMA(0, 0, At, B0); BAR; SCHED;
        ensure_ready(t + 2);
        const char *a2 = a_of(t + 2), *b2 = b_of(t + 2);
        LDB(B1, 0, 1); STAGE(SB(0, 0), b2);
        BAR; WAIT_L(0); MMA(0, 1, At, B1); BAR;
        LDA(At, 0, 1); STAGE(SA(0, 0), a2);
        BAR; WAIT_L(0); MMA(1, 0, At, B0); BAR; SCHED;
        STAGE(SB(0, 1), b2 + HALFB);
        WAIT_V(6); BAR; MMA(1, 1, At, B1); BAR;
        LDB(B0, 1, 0); SCHED; LDA(At, 1, 0); STAGE(SA(0, 1), a2 + HALFB);
        WAIT_L(8); BAR; WAIT_L(0); MMA(0, 0, At, B0); BAR; SCHED;
        ensure_ready(t + 3);
        const char *a3 = a_of(t + 3), *b3 = b_of(t + 3);
        LDB(B1, 1, 1); STAGE(SB(1, 0), b3);
        BAR; WAIT_L(0); MMA(0, 1, At, B1); BAR;
        LDA(At, 1, 1); STAGE(SA(1, 0), a3);
        BAR; WAIT_L(0); MMA(1, 0, At, B0); BAR; SCHED;
        STAGE(SB(1, 1), b3 + HALFB);
        WAIT_V(6); BAR; MMA(1, 1, At, B1); BAR;
    }
    STAMP(2);
    {
        const char* a1 = a_of(NT - 1);
        LDB(B0, 0, 0); LDA(At, 0, 0); STAGE(SA(1, 1), a1 + HALFB);
        BAR; WAIT_L(0); MMA(0, 0, At, B0); BAR;
        LDB(B1, 0, 1); BAR; WAIT_L(0); MMA(0, 1, At, B1); BAR;
        LDA(At, 0, 1); WAIT_V(0); BAR; WAIT_L(0); MMA(1, 0, At, B0); MMA(1, 1, At, B1); BAR;
    }
    {
        LDB(B0, 1, 0); LDA(At, 1, 0); BAR; WAIT_L(0); MMA(0, 0, At, B0); BAR;
        LDB(B1, 1, 1); BAR; WAIT_L(0); MMA(0, 1, At, B1); BAR;
        LDA(At, 1, 1); BAR; WAIT_L(0); MMA(1, 0, At, B0); MMA(1, 1, At, B1); BAR;
    }
    if (wr == 0) BAR;
    STAMP(3);

    float mx = 0.f;
#pragma unroll
    for (int a = 0; a < 2; ++a)
#pragma unroll
        for (int b = 0; b < 2; ++b)
#pragma unroll
            for (int m = 0; m < 4; ++m)
#pragma unroll
                for (int n = 0; n < 2; ++n)
#pragma unroll
                    for (int e = 0; e < 4; ++e) mx = fmaxf(mx, acc[a][b][m][n][e]);
#pragma unroll
    for (int o = 32; o >= 1; o >>= 1) mx = fmaxf(mx, __shfl_xor(mx, o));
    if (lane == 0) lctl[8 + wid] = __float_as_uint(mx);
    __syncthreads();
    if (wid == 0) {
        unsigned mb = lctl[8 + (lane & 7)];
#pragma unroll
        for (int o = 4; o >= 1; o >>= 1) mb = max(mb, (unsigned)__shfl_xor((int)mb, o));
        if (lane < NR)
            __hip_atomic_store((u64*)(p.ws[lane] + OFF_CTL + CTL_AMAX + (rank * 128 + gi) * 8), ((u64)call << 32) | mb, __ATOMIC_RELAXED, __HIP_MEMORY_SCOPE_SYSTEM);
    }
    unsigned gmx;
    {
        const __amdgpu_buffer_rsrc_t ra = mk_rsrc(ctl + CTL_AMAX, 8192);
        const u64 t0 = __builtin_amdgcn_s_memrealtime(); unsigned spins = 0;
        for (;;) {
            const u32x4 g = __builtin_amdgcn_raw_buffer_load_b128(ra, tid * 16, 0, AUX_SYS);
            gmx = max(g.x, g.z);
            if (__all(g.y >= call && g.w >= call)) break;
            if (give_up(ctl, call, t0, spins)) break;
            __builtin_amdgcn_s_sleep(1);
        }
    }
#pragma unroll
    for (int o = 32; o >= 1; o >>= 1) gmx = max(gmx, (unsigned)__shfl_xor((int)gmx, o));
    if (lane == 0) lctl[16 + wid] = gmx;
    const unsigned tmo = ld_agt(ctl + CTL_TMO);
    __syncthreads();
    STAMP(4);
    {
        unsigned g = lctl[16 + (lane & 7)];
#pragma unroll
        for (int o = 4; o >= 1; o >>= 1) g = max(g, (unsigned)__shfl_xor((int)g, o));
        gmx = g;
    }
    const float scale = __uint_as_float(gmx) / 448.0f, inv = 1.0f / scale;
    const bool bad = tmo == call;
    const int owner = ti >> 1;
    const __amdgpu_buffer_rsrc_t ro = mk_rsrc(p.out[owner], OUT_BYTES);
    const unsigned obase = (unsigned)((ti & 1) * 256 + wr * 64 + (lane & 15)) * GN + (unsigned)(rank * NS + tj * 256 + wc * 64 + (lane >> 4) * 16);
#pragma unroll
    for (int a = 0; a < 2; ++a)
#pragma unroll
        for (int m = 0; m < 4; ++m) {
            u32x4 o;
#pragma unroll
            for (int b = 0; b < 2; ++b)
#pragma unroll
                for (int n = 0; n < 2; ++n) {
                    const f32x4 v = acc[a][b][m][n];
                    int r = __builtin_amdgcn_cvt_pk_fp8_f32(fmaxf(v[0], 0.f) * inv, fmaxf(v[1], 0.f) * inv, 0, false);
                    r = __builtin_amdgcn_cvt_pk_fp8_f32(fmaxf(v[2], 0.f) * inv, fmaxf(v[3], 0.f) * inv, r, true);
                    o[b * 2 + n] = bad ? 0x7f7f7f7fu : (unsigned)r;
                }
            __builtin_amdgcn_raw_buffer_store_b128(o, ro, obase + (unsigned)(a * 128 + m * 16) * GN, 0, AUX_SYS);
        }
    WAIT_V(0);
    __syncthreads();
    if (tid == 0) st_sys(p.ws[owner] + OFF_CTL + CTL_DONE + (rank * 16 + (ti & 1) * 8 + tj) * 4, call);
    STAMP(5);
}

}

extern "C" void kernel_launch(int rank, int nranks, ncclComm_t comm,
                              hipStream_t stream,
                              void* const* d_in, const int* in_sizes, int n_in,
                              void* d_out, int out_size,
                              const int* in_dtypes, int out_dtype,
                              void* const* d_out_all,
                              void* const* d_ws_all, size_t ws_bytes) {
    (void)comm; (void)in_dtypes; (void)out_dtype; (void)d_out;
    if (nranks != NR || (unsigned)rank >= (unsigned)NR || n_in < 2 || ws_bytes < WS_NEED || in_sizes[0] != GM * KS ||
        in_sizes[1] != KS * GN || (unsigned)out_size != OUT_BYTES) {
        fprintf(stderr, "kernel_launch: unexpected problem shape\n");
        return;
    }
    HIP_CHECK(hipFuncSetAttribute((const void*)fused_gemm_quant, hipFuncAttributeMaxDynamicSharedMemorySize, LDS_BYTES));
    Params p;
    p.x = (const char*)d_in[0];
    p.w = (const char*)d_in[1];
    for (int r = 0; r < NR; ++r) { p.ws[r] = (char*)d_ws_all[r]; p.out[r] = (char*)d_out_all[r]; }
    p.rank = rank;
    hipLaunchKernelGGL(fused_gemm_quant, dim3(N_WG), dim3(512), LDS_BYTES, stream, p);
    HIP_CHECK(hipGetLastError());
}
```

```cpp
#include <hip/hip_runtime.h>
#include <rccl/rccl.h>
#include <cstdio>
#include <cstdint>

#define HIP_CHECK(expr) do { hipError_t _e = (expr); if (_e != hipSuccess) { \
    fprintf(stderr, "HIP error %s at %s:%d\n", hipGetErrorString(_e), __FILE__, __LINE__); \
    return; } } while (0)

#define POLL_SLEEP 16
#define PULL_U 2

namespace {

#define LAS __attribute__((address_space(3)))
typedef short bf16x8 __attribute__((ext_vector_type(8)));
typedef short s16x4 __attribute__((ext_vector_type(4)));
typedef float f32x4 __attribute__((ext_vector_type(4)));
typedef unsigned u32x4 __attribute__((ext_vector_type(4)));
typedef unsigned u32x2 __attribute__((ext_vector_type(2)));
typedef unsigned long long u64;

constexpr int NR = 8;
constexpr int GM = 4096, GN = 16384;
constexpr int KS = 512, NS = 2048, MS = 512;
constexpr unsigned PIECE = 32768;
constexpr unsigned HALFB = 16384;
constexpr unsigned OFF_X = 0;
constexpr unsigned OFF_WIN = 32u << 20;
constexpr unsigned OFF_WEX = 48u << 20;
constexpr unsigned OFF_CTL = 64u << 20;
constexpr unsigned CTL_SEQ = 0;
constexpr unsigned CTL_TMO = 32768;
constexpr unsigned CTL_EXPF = 36864;
constexpr unsigned CTL_XCNT = 45056;
constexpr unsigned CTL_WCNT = 49152;
constexpr unsigned CTL_AMAX = 53248;
constexpr unsigned CTL_DONE = 61440;
constexpr unsigned CTL_SINK = 62464;
constexpr unsigned CTL_TICK = 63488;
constexpr unsigned CTL_DBG = 65536;
constexpr size_t WS_NEED = (size_t)OFF_CTL + (1u << 20);
constexpr unsigned OUT_BYTES = (unsigned)MS * GN;

constexpr int N_EXP = 72, N_PULL = 56, N_GEMM = 128, N_WG = N_EXP + N_PULL + N_GEMM;
constexpr int LDS_STAGE = 131072, LDS_BYTES = LDS_STAGE + 256;
constexpr u64 DEADLINE = 50000000ull;

struct Params {
    const char* x;
    const char* w;
    char* ws[NR];
    char* out[NR];
    int rank;
};

__device__ __forceinline__ __amdgpu_buffer_rsrc_t mk_rsrc(const void* p, unsigned bytes) {
    return __builtin_amdgcn_make_buffer_rsrc(const_cast<void*>(p), 0, bytes, 0x00020000);
}
constexpr int AUX_SYS = 17;
constexpr int AUX_AGT = 16;

__device__ __forceinline__ unsigned ld_sys(const void* p) { return __hip_atomic_load((const unsigned*)p, __ATOMIC_RELAXED, __HIP_MEMORY_SCOPE_SYSTEM); }
__device__ __forceinline__ unsigned ld_agt(const void* p) { return __hip_atomic_load((const unsigned*)p, __ATOMIC_RELAXED, __HIP_MEMORY_SCOPE_AGENT); }
__device__ __forceinline__ void st_sys(void* p, unsigned v) { __hip_atomic_store((unsigned*)p, v, __ATOMIC_RELAXED, __HIP_MEMORY_SCOPE_SYSTEM); }
__device__ __forceinline__ void add_agt(void* p, unsigned v) { __hip_atomic_fetch_add((unsigned*)p, v, __ATOMIC_RELAXED, __HIP_MEMORY_SCOPE_AGENT); }
#define WAIT_V(n) asm volatile("s_waitcnt vmcnt(" #n ")" ::: "memory")
#define WAIT_L(n) asm volatile("s_waitcnt lgkmcnt(" #n ")" ::: "memory")
#define BAR __builtin_amdgcn_s_barrier()
#define SCHED __builtin_amdgcn_sched_barrier(0)

__device__ __forceinline__ bool give_up(char* ctl, unsigned call, u64 t0, unsigned& spins) {
    if ((++spins & 31u) != 0u) return false;
    const unsigned tm = __builtin_amdgcn_readfirstlane(ld_agt(ctl + CTL_TMO));
    if (tm == call) return true;
    if (__builtin_amdgcn_s_memrealtime() - t0 > DEADLINE) {
        __hip_atomic_store((unsigned*)(ctl + CTL_TMO), call, __ATOMIC_RELAXED, __HIP_MEMORY_SCOPE_AGENT);
        return true;
    }
    return false;
}

#define STAMP(k) do { } while (0)

__device__ __forceinline__ int rot_of(int j) { return (j * 16 + 3) / 7; }

__global__ void __launch_bounds__(512, 2) fused_gemm_quant(Params p) {
    extern __shared__ __attribute__((aligned(16))) unsigned char smem[];
    LAS unsigned char* lds = (LAS unsigned char*)smem;
    const int tid = threadIdx.x, lane = tid & 63;
    const int wid = __builtin_amdgcn_readfirstlane(tid >> 6);
    const int rank = p.rank, bid = blockIdx.x;
    char* const ws = p.ws[rank];
    char* const ctl = ws + OFF_CTL;
    STAMP(0);
    unsigned call_v = 0;
    if (lane == 0 && (wid == 0)) call_v = __hip_atomic_fetch_add((unsigned*)(ctl + CTL_SEQ + bid * 128), 1u, __ATOMIC_RELAXED, __HIP_MEMORY_SCOPE_AGENT) + 1u;
    LAS unsigned* lctl = (LAS unsigned*)(lds + LDS_STAGE);

    if (bid < N_EXP) {
        const __amdgpu_buffer_rsrc_t rx = mk_rsrc(p.x, (unsigned)GM * KS * 2), rw = mk_rsrc(p.w, (unsigned)KS * GN * 2);
        const __amdgpu_buffer_rsrc_t rd = mk_rsrc(ws, OFF_CTL);
        constexpr int PITCH = 528;
        for (int q = bid; q < 640; q += N_EXP) {
            const int c = q / 80, idx = q % 80;
            const bool isx = idx >= 56 && idx < 72;
            int dst = rank, pj = 0;
            if (isx) {
                pj = idx - 56;
                u32x4 v[4];
                const unsigned src = (unsigned)((pj * 256 + wid * 32 + (lane & 15)) * (KS * 2) + c * 128 + (lane >> 4) * 16);
#pragma unroll
                for (int u = 0; u < 4; ++u) v[u] = __builtin_amdgcn_raw_buffer_load_b128(rx, src + (u >> 1) * 16 * (KS * 2) + (u & 1) * 64, 0, 0);
                const unsigned d0 = OFF_X + (unsigned)((rank * 8 + c) * 16 + pj) * PIECE + wid * 4096 + lane * 16;
#pragma unroll
                for (int u = 0; u < 4; ++u) __builtin_amdgcn_raw_buffer_store_b128(v[u], rd, d0 + u * 1024, 0, AUX_SYS);
            } else {
                if (idx < 56) { dst = (rank + 1 + idx % 7) & 7; pj = idx / 7; } else { pj = idx - 72; }
                u32x4 v[4];
                const unsigned src = (unsigned)(c * 64 + (tid >> 5)) * (GN * 2) + (unsigned)(dst * NS + pj * 256) * 2 + (tid & 31) * 16;
#pragma unroll
                for (int u = 0; u < 4; ++u) v[u] = __builtin_amdgcn_raw_buffer_load_b128(rw, src + u * 16 * (GN * 2), 0, 0);
#pragma unroll
                for (int u = 0; u < 4; ++u) *(LAS u32x4*)(lds + ((tid >> 5) + 16 * u) * PITCH + (tid & 31) * 16) = v[u];
                __syncthreads();
                const int bj = wid >> 2, wc = wid & 3;
                const int g = lane >> 4, qq = (lane >> 2) & 3, pp = lane & 3;
                const unsigned d0 = (dst == rank ? OFF_WIN : OFF_WEX) + (unsigned)((dst * 8 + c) * 8 + pj) * PIECE + wid * 4096 + lane * 16;
#pragma unroll
                for (int u = 0; u < 4; ++u) {
                    const int n = u >> 1, kf = u & 1;
                    const int a = (kf * 32 + g * 8 + qq) * PITCH + (wc * 64 + pp * 16 + (bj * 2 + n) * 4) * 2;
                    const s16x4 lo = __builtin_amdgcn_ds_read_tr16_b64_v4i16((LAS s16x4*)(lds + a));
                    const s16x4 hi = __builtin_amdgcn_ds_read_tr16_b64_v4i16((LAS s16x4*)(lds + a + 4 * PITCH));
                    const u32x2 l2 = __builtin_bit_cast(u32x2, lo), h2 = __builtin_bit_cast(u32x2, hi);
                    const u32x4 o = {l2.x, l2.y, h2.x, h2.y};
                    __builtin_amdgcn_raw_buffer_store_b128(o, rd, d0 + u * 1024, 0, AUX_SYS);
                }
            }
            WAIT_V(0);
            __syncthreads();
            if (wid == 0) {
                const unsigned call = __builtin_amdgcn_readfirstlane(call_v);
                if (isx) {
                    if (lane < NR) {
                        if (lane == rank) add_agt(ctl + CTL_XCNT + (pj * 64 + rank * 8 + c) * 4, 32u);
                        else st_sys(p.ws[lane] + OFF_CTL + CTL_EXPF + (rank * 256 + c * 24 + 8 + pj) * 4, call);
                    }
                } else if (lane == 0) {
                    if (dst == rank) add_agt(ctl + CTL_WCNT + (pj * 64 + rank * 8 + c) * 4, 32u);
                    else st_sys(p.ws[dst] + OFF_CTL + CTL_EXPF + (rank * 256 + c * 24 + pj) * 4, call);
                }
            }
            if (q == bid) STAMP(1);
        }
        STAMP(2);
        if (bid == 0 && wid == 0) {
            const unsigned call = __builtin_amdgcn_readfirstlane(call_v);
            const u64 t0 = __builtin_amdgcn_s_memrealtime(); unsigned spins = 0; bool bad = false;
            for (;;) {
                const u64 f = __hip_atomic_load((const u64*)(ctl + CTL_DONE + lane * 8), __ATOMIC_RELAXED, __HIP_MEMORY_SCOPE_SYSTEM);
                if (__all((unsigned)f >= call && (unsigned)(f >> 32) >= call)) break;
                if (give_up(ctl, call, t0, spins)) { bad = true; break; }
                __builtin_amdgcn_s_sleep(4);
            }
            if (bad && lane == 0) p.out[rank][(size_t)rank * NS] = (char)0x7f;
            STAMP(3);
        }
        return;
    }

    if (tid == 0) { lctl[0] = call_v; lctl[2] = 0u; lctl[3] = 0u; }
    __syncthreads();
    const unsigned call = __builtin_amdgcn_readfirstlane(lctl[0]);

    if (bid < N_EXP + N_PULL) {
        const int q = bid - N_EXP, j = q % 7;
        const int src = (rank + 1 + j) & 7, rot = rot_of(j);
        const __amdgpu_buffer_rsrc_t rs = mk_rsrc(p.ws[src], OFF_CTL), rd = mk_rsrc(ws, OFF_CTL);
        const char* ef = ctl + CTL_EXPF + src * 1024 + lane * 4;
        u64 m0 = 0, m1 = 0, m2 = 0;
        const __amdgpu_buffer_rsrc_t rc = mk_rsrc(ctl, 65536);
        const unsigned aoff = lane == 0 ? 0u : 0x80000000u;
        constexpr unsigned UPP = 32 / PULL_U, NU = 192 * UPP;
        const unsigned tbase = (call - 1u) * (NU + 8u * (N_PULL / 7)), toff = CTL_TICK + j * 128;
        unsigned cnt1 = CTL_SINK, cnt2 = CTL_SINK;
        unsigned tk = (unsigned)__builtin_amdgcn_readfirstlane(__builtin_amdgcn_raw_ptr_buffer_atomic_add_i32(1, rc, aoff, toff, 0)) - tbase;
        while (tk < NU) {
            const int pp = tk / UPP, c = pp / 24, r = pp % 24, sc = src * 8 + c;
            const unsigned voff = (tk % UPP) * (1024u * PULL_U) + lane * 16;
            int id; unsigned so, dof, cnt;
            if (r < 8) { id = c * 24 + r; so = OFF_WEX + (unsigned)((rank * 8 + c) * 8 + r) * PIECE; dof = OFF_WIN + (unsigned)(sc * 8 + r) * PIECE; cnt = CTL_WCNT + (r * 64 + sc) * 4; }
            else { const int i = (rot + r - 8) & 15; id = c * 24 + 8 + i; so = OFF_X + (unsigned)(sc * 16 + i) * PIECE; dof = so; cnt = CTL_XCNT + (i * 64 + sc) * 4; }
            const u64 mm = id < 64 ? m0 : (id < 128 ? m1 : m2);
            if (!((mm >> (id & 63)) & 1)) {
                const u64 t0 = __builtin_amdgcn_s_memrealtime(); unsigned spins = 0;
                for (;;) {
                    const unsigned f0 = ld_sys(ef), f1 = ld_sys(ef + 256), f2 = ld_sys(ef + 512);
                    m0 = __ballot(f0 >= call); m1 = __ballot(f1 >= call); m2 = __ballot(f2 >= call);
                    const u64 mn = id < 64 ? m0 : (id < 128 ? m1 : m2);
                    if ((mn >> (id & 63)) & 1) break;
                    if (give_up(ctl, call, t0, spins)) { m0 = m1 = m2 = ~0ull; break; }
                    __builtin_amdgcn_s_sleep(2);
                }
                if (cnt1 == CTL_SINK) STAMP(1);
            }
            int tnext = 0;
#pragma unroll
            for (int u = 0; u < PULL_U; ++u) {
                const u32x4 v = __builtin_amdgcn_raw_buffer_load_b128(rs, voff + u * 1024, so, AUX_SYS);
                if (u == PULL_U - 1) tnext = __builtin_amdgcn_raw_ptr_buffer_atomic_add_i32(1, rc, aoff, toff, 0);
                if (PULL_U == 1 || u == 1) {
                    __builtin_amdgcn_raw_ptr_buffer_atomic_add_i32(PULL_U, rc, aoff, PULL_U == 1 ? cnt2 : cnt1, 0);
                    WAIT_V(1);
                } else {
                    WAIT_V(0);
                }
                __builtin_amdgcn_raw_buffer_store_b128(v, rd, voff + u * 1024, dof, AUX_AGT);
            }
            cnt2 = cnt1; cnt1 = cnt;
            tk = (unsigned)__builtin_amdgcn_readfirstlane(tnext) - tbase;
        }
        if (PULL_U == 1) __builtin_amdgcn_raw_ptr_buffer_atomic_add_i32(PULL_U, rc, aoff, cnt2, 0);
        WAIT_V(0);
        __builtin_amdgcn_raw_ptr_buffer_atomic_add_i32(PULL_U, rc, aoff, cnt1, 0);
        STAMP(2);
        return;
    }

    const int gi = bid - (N_EXP + N_PULL), xg = gi & 7, ix = gi >> 3;
    const int ti = (xg >> 1) * 4 + (ix & 3), tj = (xg & 1) * 4 + (ix >> 2);
    const int wr = wid >> 2, wc = wid & 3;
    int scv;
    {
        int j0 = 0;
#pragma unroll
        for (int j = 1; j < 7; ++j) if (rot_of(j) <= ti) j0 = j;
        const int t2 = lane - 8, c = t2 / 7, k = t2 - c * 7;
        int j = j0 - k; if (j < 0) j += 7;
        scv = lane < 8 ? rank * 8 + lane : ((rank + 1 + j) & 7) * 8 + c;
    }
    const unsigned target = call * 32u;
    u64 rmask = 0;
    auto ensure_ready = [&](int t) {
        if (!((rmask >> t) & 1)) {
            if (wid == 0) {
                const u64 t0 = __builtin_amdgcn_s_memrealtime(); unsigned spins = 0;
                for (;;) {
                    const unsigned a = ld_agt(ctl + CTL_XCNT + (ti * 64 + scv) * 4), b = ld_agt(ctl + CTL_WCNT + (tj * 64 + scv) * 4);
                    rmask = __ballot(a >= target && b >= target);
                    if ((rmask >> t) & 1) break;
                    if (give_up(ctl, call, t0, spins)) { rmask = ~0ull; break; }
                    __builtin_amdgcn_s_sleep(POLL_SLEEP);
                }
                const u32x2 mv = {(unsigned)rmask, (unsigned)(rmask >> 32)};
                asm volatile("ds_write_b64 %0, %1" :: "v"(LDS_STAGE + 8), "v"(mv) : "memory");
            } else {
                for (;;) {
                    u32x2 mv;
                    asm volatile("ds_read_b64 %0, %1\n\ts_waitcnt lgkmcnt(0)" : "=v"(mv) : "v"(LDS_STAGE + 8) : "memory");
                    rmask = ((u64)(unsigned)__builtin_amdgcn_readfirstlane(mv.y) << 32) | (unsigned)__builtin_amdgcn_readfirstlane(mv.x);
                    if ((rmask >> t) & 1) break;
                    __builtin_amdgcn_s_sleep(1);
                }
            }
        }
    };
    const char* const xbase = ws + OFF_X + (size_t)ti * PIECE;
    const char* const wbase = ws + OFF_WIN + (size_t)tj * PIECE;
    auto a_of = [&](int t) { return xbase + (size_t)((unsigned)__builtin_amdgcn_readlane(scv, t) * (16u * PIECE)); };
    auto b_of = [&](int t) { return wbase + (size_t)((unsigned)__builtin_amdgcn_readlane(scv, t) * (8u * PIECE)); };
    const unsigned voff = (unsigned)tid * 16u, ldsw = (unsigned)wid * 1024u;
    const int aoff = wr * 8192 + lane * 16, boff = wc * 4096 + lane * 16;
#define SA(b, h) (((b) * 2 + (h)) * HALFB)
#define SB(b, h) ((4 + (b) * 2 + (h)) * HALFB)
#define STAGE(bufoff, gbase) do { _Pragma("unroll") for (int _i = 0; _i < 2; ++_i) \
        __builtin_amdgcn_global_load_lds((const unsigned*)((gbase) + voff + _i * 8192), (LAS unsigned*)(lds + (bufoff) + ldsw + _i * 8192), 16, 0, AUX_AGT); } while (0)
#define LDA(dst, b, h) do { _Pragma("unroll") for (int m = 0; m < 4; ++m) _Pragma("unroll") for (int k = 0; k < 2; ++k) dst[m][k] = *(const LAS bf16x8*)(lds + SA(b, h) + aoff + m * 2048 + k * 1024); } while (0)
#define LDB(dst, b, h) do { _Pragma("unroll") for (int n = 0; n < 2; ++n) _Pragma("unroll") for (int k = 0; k < 2; ++k) dst[n][k] = *(const LAS bf16x8*)(lds + SB(b, h) + boff + n * 2048 + k * 1024); } while (0)
#define MMA(ai, bj, At, Bt) do { __builtin_amdgcn_s_setprio(1); _Pragma("unroll") for (int m = 0; m < 4; ++m) _Pragma("unroll") for (int n = 0; n < 2; ++n) _Pragma("unroll") for (int k = 0; k < 2; ++k) \
        acc[ai][bj][m][n] = __builtin_amdgcn_mfma_f32_16x16x32_bf16(Bt[n][k], At[m][k], acc[ai][bj][m][n], 0, 0, 0); __builtin_amdgcn_s_setprio(0); } while (0)
    f32x4 acc[2][2][4][2];
#pragma unroll
    for (int a = 0; a < 2; ++a)
#pragma unroll
        for (int b = 0; b < 2; ++b)
#pragma unroll
            for (int m = 0; m < 4; ++m)
#pragma unroll
                for (int n = 0; n < 2; ++n) acc[a][b][m][n] = (f32x4){0.f, 0.f, 0.f, 0.f};
    bf16x8 At[4][2], B0[2][2], B1[2][2];
    constexpr int NT = 64;
    {
        ensure_ready(0);
        const char *a0 = a_of(0), *b0 = b_of(0);
        STAGE(SB(0, 0), b0); STAGE(SA(0, 0), a0); STAGE(SB(0, 1), b0 + HALFB); STAGE(SA(0, 1), a0 + HALFB);
        ensure_ready(1);
        if (wr == 1) BAR;
        WAIT_V(4); BAR;
        const char *a1 = a_of(1), *b1 = b_of(1);
        STAGE(SB(1, 0), b1); STAGE(SA(1, 0), a1); STAGE(SB(1, 1), b1 + HALFB);
        WAIT_V(6); BAR;
    }
    STAMP(1);
    for (int t = 0; t < NT - 2; t += 2) {
        const char* a1 = a_of(t + 1);
        LDB(B0, 0, 0); SCHED; LDA(At, 0, 0); STAGE(SA(1, 1), a1 + HALFB);
        WAIT_L(8); BAR; WAIT_L(0); MMA(0, 0, At, B0); BAR; SCHED;
        ensure_ready(t + 2);
        const char *a2 = a_of(t + 2), *b2 = b_of(t + 2);
        LDB(B1, 0, 1); STAGE(SB(0, 0), b2);
        BAR; WAIT_L(0); MMA(0, 1, At, B1); BAR;
        LDA(At, 0, 1); STAGE(SA(0, 0), a2);
        BAR; WAIT_L(0); MMA(1, 0, At, B0); BAR; SCHED;
        STAGE(SB(0, 1), b2 + HALFB);
        WAIT_V(6); BAR; MMA(1, 1, At, B1); BAR;
        LDB(B0, 1, 0); SCHED; LDA(At, 1, 0); STAGE(SA(0, 1), a2 + HALFB);
        WAIT_L(8); BAR; WAIT_L(0); MMA(0, 0, At, B0); BAR; SCHED;
        ensure_ready(t + 3);
        const char *a3 = a_of(t + 3), *b3 = b_of(t + 3);
        LDB(B1, 1, 1); STAGE(SB(1, 0), b3);
        BAR; WAIT_L(0); MMA(0, 1, At, B1); BAR;
        LDA(At, 1, 1); STAGE(SA(1, 0), a3);
        BAR; WAIT_L(0); MMA(1, 0, At, B0); BAR; SCHED;
        STAGE(SB(1, 1), b3 + HALFB);
        WAIT_V(6); BAR; MMA(1, 1, At, B1); BAR;
    }
    STAMP(2);
    {
        const char* a1 = a_of(NT - 1);
        LDB(B0, 0, 0); LDA(At, 0, 0); STAGE(SA(1, 1), a1 + HALFB);
        BAR; WAIT_L(0); MMA(0, 0, At, B0); BAR;
        LDB(B1, 0, 1); BAR; WAIT_L(0); MMA(0, 1, At, B1); BAR;
        LDA(At, 0, 1); WAIT_V(0); BAR; WAIT_L(0); MMA(1, 0, At, B0); MMA(1, 1, At, B1); BAR;
    }
    {
        LDB(B0, 1, 0); LDA(At, 1, 0); BAR; WAIT_L(0); MMA(0, 0, At, B0); BAR;
        LDB(B1, 1, 1); BAR; WAIT_L(0); MMA(0, 1, At, B1); BAR;
        LDA(At, 1, 1); BAR; WAIT_L(0); MMA(1, 0, At, B0); MMA(1, 1, At, B1); BAR;
    }
    if (wr == 0) BAR;
    STAMP(3);

    float mx = 0.f;
#pragma unroll
    for (int a = 0; a < 2; ++a)
#pragma unroll
        for (int b = 0; b < 2; ++b)
#pragma unroll
            for (int m = 0; m < 4; ++m)
#pragma unroll
                for (int n = 0; n < 2; ++n)
#pragma unroll
                    for (int e = 0; e < 4; ++e) mx = fmaxf(mx, acc[a][b][m][n][e]);
#pragma unroll
    for (int o = 32; o >= 1; o >>= 1) mx = fmaxf(mx, __shfl_xor(mx, o));
    if (lane == 0) lctl[8 + wid] = __float_as_uint(mx);
    __syncthreads();
    if (wid == 0) {
        unsigned mb = lctl[8 + (lane & 7)];
#pragma unroll
        for (int o = 4; o >= 1; o >>= 1) mb = max(mb, (unsigned)__shfl_xor((int)mb, o));
        if (lane < NR)
            __hip_atomic_store((u64*)(p.ws[lane] + OFF_CTL + CTL_AMAX + (rank * 128 + gi) * 8), ((u64)call << 32) | mb, __ATOMIC_RELAXED, __HIP_MEMORY_SCOPE_SYSTEM);
    }
    unsigned gmx;
    {
        const __amdgpu_buffer_rsrc_t ra = mk_rsrc(ctl + CTL_AMAX, 8192);
        const u64 t0 = __builtin_amdgcn_s_memrealtime(); unsigned spins = 0;
        for (;;) {
            const u32x4 g = __builtin_amdgcn_raw_buffer_load_b128(ra, tid * 16, 0, AUX_SYS);
            gmx = max(g.x, g.z);
            if (__all(g.y >= call && g.w >= call)) break;
            if (give_up(ctl, call, t0, spins)) break;
            __builtin_amdgcn_s_sleep(1);
        }
    }
#pragma unroll
    for (int o = 32; o >= 1; o >>= 1) gmx = max(gmx, (unsigned)__shfl_xor((int)gmx, o));
    if (lane == 0) lctl[16 + wid] = gmx;
    const unsigned tmo = ld_agt(ctl + CTL_TMO);
    __syncthreads();
    STAMP(4);
    {
        unsigned g = lctl[16 + (lane & 7)];
#pragma unroll
        for (int o = 4; o >= 1; o >>= 1) g = max(g, (unsigned)__shfl_xor((int)g, o));
        gmx = g;
    }
    const float scale = __uint_as_float(gmx) / 448.0f, inv = 1.0f / scale;
    const bool bad = tmo == call;
    const int owner = ti >> 1;
    const __amdgpu_buffer_rsrc_t ro = mk_rsrc(p.out[owner], OUT_BYTES);
    const unsigned obase = (unsigned)((ti & 1) * 256 + wr * 64 + (lane & 15)) * GN + (unsigned)(rank * NS + tj * 256 + wc * 64 + (lane >> 4) * 16);
#pragma unroll
    for (int a = 0; a < 2; ++a)
#pragma unroll
        for (int m = 0; m < 4; ++m) {
            u32x4 o;
#pragma unroll
            for (int b = 0; b < 2; ++b)
#pragma unroll
                for (int n = 0; n < 2; ++n) {
                    const f32x4 v = acc[a][b][m][n];
                    int r = __builtin_amdgcn_cvt_pk_fp8_f32(fmaxf(v[0], 0.f) * inv, fmaxf(v[1], 0.f) * inv, 0, false);
                    r = __builtin_amdgcn_cvt_pk_fp8_f32(fmaxf(v[2], 0.f) * inv, fmaxf(v[3], 0.f) * inv, r, true);
                    o[b * 2 + n] = bad ? 0x7f7f7f7fu : (unsigned)r;
                }
            __builtin_amdgcn_raw_buffer_store_b128(o, ro, obase + (unsigned)(a * 128 + m * 16) * GN, 0, AUX_SYS);
        }
    WAIT_V(0);
    __syncthreads();
    if (tid == 0) st_sys(p.ws[owner] + OFF_CTL + CTL_DONE + (rank * 16 + (ti & 1) * 8 + tj) * 4, call);
    STAMP(5);
}

}

extern "C" void kernel_launch(int rank, int nranks, ncclComm_t comm,
                              hipStream_t stream,
                              void* const* d_in, const int* in_sizes, int n_in,
                              void* d_out, int out_size,
                              const int* in_dtypes, int out_dtype,
                              void* const* d_out_all,
                              void* const* d_ws_all, size_t ws_bytes) {
    (void)comm; (void)in_dtypes; (void)out_dtype; (void)d_out;
    if (nranks != NR || (unsigned)rank >= (unsigned)NR || n_in < 2 || ws_bytes < WS_NEED || in_sizes[0] != GM * KS ||
        in_sizes[1] != KS * GN || (unsigned)out_size != OUT_BYTES) {
        fprintf(stderr, "kernel_launch: unexpected problem shape\n");
        return;
    }
    HIP_CHECK(hipFuncSetAttribute((const void*)fused_gemm_quant, hipFuncAttributeMaxDynamicSharedMemorySize, LDS_BYTES));
    Params p;
    p.x = (const char*)d_in[0];
    p.w = (const char*)d_in[1];
    for (int r = 0; r < NR; ++r) { p.ws[r] = (char*)d_ws_all[r]; p.out[r] = (char*)d_out_all[r]; }
    p.rank = rank;
    hipLaunchKernelGGL(fused_gemm_quant, dim3(N_WG), dim3(512), LDS_BYTES, stream, p);
    HIP_CHECK(hipGetLastError());
}
```

```cpp
#include <hip/hip_runtime.h>
#include <rccl/rccl.h>
#include <cstdio>
#include <cstdint>

#define HIP_CHECK(expr) do { hipError_t _e = (expr); if (_e != hipSuccess) { \
    fprintf(stderr, "HIP error %s at %s:%d\n", hipGetErrorString(_e), __FILE__, __LINE__); \
    return; } } while (0)

#define POLL_SLEEP 16
#define AMAX_AUX AUX_SYS
#define AMAX_SLEEP 1
#define N_PUSH 2
#define TAIL_KB 0
#define ASYNC_FROM 99
#define N_PUSH1 2
#define PUSH_DELAY 230
#define OPOLL_SLEEP 8
#define OFLG_LD ld_sys
#define OFLG_AT 1
#define PULL_U 2

namespace {

#define LAS __attribute__((address_space(3)))
typedef short bf16x8 __attribute__((ext_vector_type(8)));
typedef short s16x4 __attribute__((ext_vector_type(4)));
typedef float f32x4 __attribute__((ext_vector_type(4)));
typedef unsigned u32x4 __attribute__((ext_vector_type(4)));
typedef unsigned u32x2 __attribute__((ext_vector_type(2)));
typedef unsigned long long u64;

constexpr int NR = 8;
constexpr int GM = 4096, GN = 16384;
constexpr int KS = 512, NS = 2048, MS = 512;
constexpr unsigned PIECE = 32768;
constexpr unsigned HALFB = 16384;
constexpr unsigned OFF_X = 0;
constexpr unsigned OFF_WIN = 32u << 20;
constexpr unsigned OFF_WEX = 48u << 20;
constexpr unsigned OFF_OEX = 64u << 20;
constexpr unsigned OFF_CTL = 72u << 20;
constexpr unsigned CTL_SEQ = 0;
constexpr unsigned CTL_TMO = 32768;
constexpr unsigned CTL_EXPF = 36864;
constexpr unsigned CTL_XCNT = 45056;
constexpr unsigned CTL_WCNT = 49152;
constexpr unsigned CTL_AMAX = 53248;
constexpr unsigned CTL_OFLG = 16384;
constexpr unsigned CTL_TICK = 63488;
constexpr unsigned CTL_DBG = 65536;
constexpr size_t WS_NEED = (size_t)OFF_CTL + (1u << 20);
constexpr unsigned OUT_BYTES = (unsigned)MS * GN;

constexpr int N_EXP = 72, N_PULL = 56, N_GEMM = 128, N_WG = N_EXP + N_PULL + N_GEMM;
constexpr int LDS_STAGE = 131072, LDS_BYTES = LDS_STAGE + 2048;
constexpr u64 DEADLINE = 50000000ull;

struct Params {
    const char* x;
    const char* w;
    char* ws[NR];
    char* out[NR];
    int rank;
};

__device__ __forceinline__ __amdgpu_buffer_rsrc_t mk_rsrc(const void* p, unsigned bytes) {
    return __builtin_amdgcn_make_buffer_rsrc(const_cast<void*>(p), 0, bytes, 0x00020000);
}
constexpr int AUX_SYS = 17;
constexpr int AUX_AGT = 16;

__device__ __forceinline__ unsigned ld_sys(const void* p) { return __hip_atomic_load((const unsigned*)p, __ATOMIC_RELAXED, __HIP_MEMORY_SCOPE_SYSTEM); }
__device__ __forceinline__ unsigned ld_agt(const void* p) { return __hip_atomic_load((const unsigned*)p, __ATOMIC_RELAXED, __HIP_MEMORY_SCOPE_AGENT); }
typedef __attribute__((address_space(1))) unsigned gu32;
__device__ __forceinline__ void st_sys(void* p, unsigned v) { __hip_atomic_store((gu32*)(size_t)p, v, __ATOMIC_RELAXED, __HIP_MEMORY_SCOPE_SYSTEM); }
__device__ __forceinline__ void add_agt(void* p, unsigned v) { __hip_atomic_fetch_add((unsigned*)p, v, __ATOMIC_RELAXED, __HIP_MEMORY_SCOPE_AGENT); }
#define WAIT_V(n) asm volatile("s_waitcnt vmcnt(" #n ")" ::: "memory")
#define WAIT_L(n) asm volatile("s_waitcnt lgkmcnt(" #n ")" ::: "memory")
#define BAR __builtin_amdgcn_s_barrier()
#define SCHED __builtin_amdgcn_sched_barrier(0)

__device__ __forceinline__ bool give_up(char* ctl, unsigned call, u64 t0, unsigned& spins) {
    if ((++spins & 31u) != 0u) return false;
    const unsigned tm = __builtin_amdgcn_readfirstlane(ld_agt(ctl + CTL_TMO));
    if (tm == call) return true;
    if (__builtin_amdgcn_s_memrealtime() - t0 > DEADLINE) {
        __hip_atomic_store((unsigned*)(ctl + CTL_TMO), call, __ATOMIC_RELAXED, __HIP_MEMORY_SCOPE_AGENT);
        return true;
    }
    return false;
}

__device__ __forceinline__ unsigned wave_max_u32(unsigned v) {
    v = max(v, (unsigned)__builtin_amdgcn_update_dpp(0, (int)v, 0x111, 0xf, 0xf, false));
    v = max(v, (unsigned)__builtin_amdgcn_update_dpp(0, (int)v, 0x112, 0xf, 0xf, false));
    v = max(v, (unsigned)__builtin_amdgcn_update_dpp(0, (int)v, 0x114, 0xf, 0xf, false));
    v = max(v, (unsigned)__builtin_amdgcn_update_dpp(0, (int)v, 0x118, 0xf, 0xf, false));
    v = max(v, (unsigned)__builtin_amdgcn_update_dpp(0, (int)v, 0x142, 0xa, 0xf, false));
    v = max(v, (unsigned)__builtin_amdgcn_update_dpp(0, (int)v, 0x143, 0xc, 0xf, false));
    return v;
}

#define STAMP(k) do { } while (0)

__device__ __forceinline__ int rot_of(int j) { return (j * 16 + 3) / 7; }

__global__ void __launch_bounds__(512, 2) fused_gemm_quant(Params p) {
    extern __shared__ __attribute__((aligned(16))) unsigned char smem[];
    LAS unsigned char* lds = (LAS unsigned char*)smem;
    const int tid = threadIdx.x, lane = tid & 63;
    const int wid = __builtin_amdgcn_readfirstlane(tid >> 6);
    const int rank = p.rank, bid = blockIdx.x;
    char* const ws = p.ws[rank];
    char* const ctl = ws + OFF_CTL;
    STAMP(0);
    const u64 t_start = __builtin_amdgcn_s_memrealtime();
    const __amdgpu_buffer_rsrc_t rc = mk_rsrc(ctl, 65536);
    const unsigned call_v = (unsigned)__builtin_amdgcn_raw_ptr_buffer_atomic_add_i32(1, rc, tid == 0 ? 0u : 0x80000000u, CTL_SEQ + bid * 64, 0) + 1u;
    LAS unsigned* lctl = (LAS unsigned*)(lds + LDS_STAGE);
    const unsigned lctl_addr = (unsigned)(size_t)lctl;

    if (bid < N_EXP) {
        const __amdgpu_buffer_rsrc_t rx = mk_rsrc(p.x, (unsigned)GM * KS * 2), rw = mk_rsrc(p.w, (unsigned)KS * GN * 2);
        const __amdgpu_buffer_rsrc_t rd = mk_rsrc(ws, OFF_CTL);
        constexpr int PITCH = 528;
        for (int q = bid; q < 640; q += N_EXP) {
            const int c = q / 80, idx = q % 80;
            const bool isx = idx >= 56 && idx < 72, push = q < 7 * N_PUSH;
            int dst = rank, pj = 0;
            if (isx) {
                pj = idx - 56;
                u32x4 v[4];
                const unsigned src = (unsigned)((pj * 256 + wid * 32 + (lane & 15)) * (KS * 2) + c * 128 + (lane >> 4) * 16);
#pragma unroll
                for (int u = 0; u < 4; ++u) v[u] = __builtin_amdgcn_raw_buffer_load_b128(rx, src + (u >> 1) * 16 * (KS * 2) + (u & 1) * 64, 0, 0);
                const unsigned d0 = OFF_X + (unsigned)((rank * 8 + c) * 16 + pj) * PIECE + wid * 4096 + lane * 16;
#pragma unroll
                for (int u = 0; u < 4; ++u) __builtin_amdgcn_raw_buffer_store_b128(v[u], rd, d0 + u * 1024, 0, AUX_SYS);
            } else {
                if (idx < 56) { dst = (rank + 1 + idx % 7) & 7; pj = idx / 7; } else { pj = idx - 72; }
                u32x4 v[4];
                const unsigned src = (unsigned)(c * 64 + (tid >> 5)) * (GN * 2) + (unsigned)(dst * NS + pj * 256) * 2 + (tid & 31) * 16;
#pragma unroll
                for (int u = 0; u < 4; ++u) v[u] = __builtin_amdgcn_raw_buffer_load_b128(rw, src + u * 16 * (GN * 2), 0, 0);
#pragma unroll
                for (int u = 0; u < 4; ++u) *(LAS u32x4*)(lds + ((tid >> 5) + 16 * u) * PITCH + (tid & 31) * 16) = v[u];
                __syncthreads();
                if (push && q >= 7 * N_PUSH1) while (__builtin_amdgcn_s_memrealtime() - t_start < PUSH_DELAY) __builtin_amdgcn_s_sleep(1);
                const int bj = wid >> 2, wc = wid & 3;
                const int g = lane >> 4, qq = (lane >> 2) & 3, pp = lane & 3;
                const __amdgpu_buffer_rsrc_t rw2 = mk_rsrc(push ? p.ws[dst] : ws, OFF_CTL);
                const unsigned d0 = (push ? OFF_WIN + (unsigned)(rank * 64 + pj) * PIECE : (dst == rank ? OFF_WIN : OFF_WEX) + (unsigned)((dst * 8 + c) * 8 + pj) * PIECE) + wid * 4096 + lane * 16;
#pragma unroll
                for (int u = 0; u < 4; ++u) {
                    const int n = u >> 1, kf = u & 1;
                    const int a = (kf * 32 + g * 8 + qq) * PITCH + (wc * 64 + pp * 16 + (bj * 2 + n) * 4) * 2;
                    const s16x4 lo = __builtin_amdgcn_ds_read_tr16_b64_v4i16((LAS s16x4*)(lds + a));
                    const s16x4 hi = __builtin_amdgcn_ds_read_tr16_b64_v4i16((LAS s16x4*)(lds + a + 4 * PITCH));
                    const u32x2 l2 = __builtin_bit_cast(u32x2, lo), h2 = __builtin_bit_cast(u32x2, hi);
                    const u32x4 o = {l2.x, l2.y, h2.x, h2.y};
                    __builtin_amdgcn_raw_buffer_store_b128(o, rw2, d0 + u * 1024, 0, AUX_SYS);
                }
            }
            WAIT_V(0);
            __syncthreads();
            if (wid == 0) {
                const unsigned call = __builtin_amdgcn_readfirstlane(call_v);
                if (isx) {
                    if (lane < NR) {
                        if (lane == rank) st_sys(ctl + CTL_XCNT + (pj * 64 + rank * 8 + c) * 4, call * 32u);
                        else st_sys(p.ws[lane] + OFF_CTL + CTL_EXPF + (rank * 256 + c * 24 + 8 + pj) * 4, call);
                    }
                } else if (lane == 0) {
                    if (dst == rank || push) st_sys(p.ws[dst] + OFF_CTL + CTL_WCNT + (pj * 64 + rank * 8 + c) * 4, call * 32u);
                    else st_sys(p.ws[dst] + OFF_CTL + CTL_EXPF + (rank * 256 + c * 24 + pj) * 4, call);
                }
            }
            if (q == bid) STAMP(1);
        }
        STAMP(2);
        return;
    }

    if (tid == 0) { lctl[0] = call_v; lctl[6] = 0u; lctl[8] = 0u; lctl[9] = 0u; }
    if (tid >= 64 && tid < 64 + NR) *(LAS u64*)(lctl + 16 + 2 * (tid - 64)) = (u64)(size_t)p.ws[tid - 64];
    __syncthreads();
    const unsigned call = __builtin_amdgcn_readfirstlane(lctl[0]);

    if (bid < N_EXP + N_PULL) {
        const int q = bid - N_EXP, j = q % 7;
        const int src = (rank + 1 + j) & 7, rot = rot_of(j);
        const __amdgpu_buffer_rsrc_t rs = mk_rsrc(p.ws[src], OFF_CTL), rd = mk_rsrc(ws, OFF_CTL);
        const char* ef = ctl + CTL_EXPF + src * 1024 + lane * 4;
        u64 m0 = 0, m1 = 0, m2 = 0;
        const unsigned aoff = lane == 0 ? 0u : 0x80000000u;
        constexpr unsigned TOTAL_KB = (192 - N_PUSH) * 32, NA = (TOTAL_KB - TAIL_KB) / PULL_U, NU = NA + TAIL_KB;
        constexpr unsigned NU2 = 512;
        const unsigned tbase = (call - 1u) * (NU + NU2 + 8u * (N_PULL / 7)), toff = CTL_TICK + j * 128;
        unsigned cnt1 = 0, cnt2 = 0, add1 = 0, add2 = 0;
        unsigned tk = (unsigned)__builtin_amdgcn_readfirstlane(__builtin_amdgcn_raw_ptr_buffer_atomic_add_i32(1, rc, aoff, toff, 0)) - tbase;
        while (tk < NU) {
            const unsigned kb = tk < NA ? tk * PULL_U : tk + NA * (PULL_U - 1), nk = tk < NA ? PULL_U : 1;
            const int pp = kb / 32 + N_PUSH, c = pp / 24, r = pp % 24, sc = src * 8 + c;
            int id; unsigned so, dof, cnt;
            if (r < 8) { id = c * 24 + r; so = OFF_WEX + (unsigned)((rank * 8 + c) * 8 + r) * PIECE; dof = OFF_WIN + (unsigned)(sc * 8 + r) * PIECE; cnt = CTL_WCNT + (r * 64 + sc) * 4; }
            else { const int i = (rot + r - 8) & 15; id = c * 24 + 8 + i; so = OFF_X + (unsigned)(sc * 16 + i) * PIECE; dof = so; cnt = CTL_XCNT + (i * 64 + sc) * 4; }
            so += (kb % 32) * 1024u; dof += (kb % 32) * 1024u;
            const u64 mm = id < 64 ? m0 : (id < 128 ? m1 : m2);
            if (!((mm >> (id & 63)) & 1)) {
                const u64 t0 = __builtin_amdgcn_s_memrealtime(); unsigned spins = 0;
                for (;;) {
                    const unsigned f0 = ld_sys(ef), f1 = ld_sys(ef + 256), f2 = ld_sys(ef + 512);
                    m0 = __ballot(f0 >= call); m1 = __ballot(f1 >= call); m2 = __ballot(f2 >= call);
                    const u64 mn = id < 64 ? m0 : (id < 128 ? m1 : m2);
                    if ((mn >> (id & 63)) & 1) break;
                    if (give_up(ctl, call, t0, spins)) { m0 = m1 = m2 = ~0ull; break; }
                    __builtin_amdgcn_s_sleep(2);
                }
                if (add1 + add2 == 0) STAMP(1);
            }
            int tnext = 0;
            for (unsigned u = 0; u < nk; ++u) {
                const u32x4 v = __builtin_amdgcn_raw_buffer_load_b128(rs, lane * 16, so + u * 1024, AUX_SYS);
                const bool last = u + 1 == nk;
                tnext = __builtin_amdgcn_raw_ptr_buffer_atomic_add_i32(1, rc, last ? aoff : 0x80000000u, toff, 0);
                __builtin_amdgcn_raw_ptr_buffer_atomic_add_i32((int)add2, rc, add2 ? aoff : 0x80000000u, cnt2, 0);
                WAIT_V(1);
                __builtin_amdgcn_raw_buffer_store_b128(v, rd, lane * 16, dof + u * 1024, AUX_AGT);
                cnt2 = cnt1; add2 = add1; cnt1 = cnt; add1 = last ? nk : 0;
            }
            tk = (unsigned)__builtin_amdgcn_readfirstlane(tnext) - tbase;
        }
        __builtin_amdgcn_raw_ptr_buffer_atomic_add_i32((int)add2, rc, add2 ? aoff : 0x80000000u, cnt2, 0);
        WAIT_V(0);
        __builtin_amdgcn_raw_ptr_buffer_atomic_add_i32((int)add1, rc, add1 ? aoff : 0x80000000u, cnt1, 0);
        STAMP(2);
        {
            const __amdgpu_buffer_rsrc_t ro = mk_rsrc(p.out[rank], OUT_BYTES);
            const char* of = ctl + CTL_OFLG + src * 2048;
            bool bad = false;
            unsigned r1 = 0, r2 = 0;
            while (tk < NU + NU2) {
                const unsigned u2 = tk - NU, tw = u2 & 127, b0 = (u2 >> 7) * 2, tl = tw >> 3;
                if (!(((b0 == 0 ? r1 : r2) >> tl) & 1)) {
                    const u64 t0 = __builtin_amdgcn_s_memrealtime(); unsigned spins = 0;
                    const unsigned need = 2u * call - (b0 == 0 ? 1u : 0u);
                    while ((unsigned)__builtin_amdgcn_readfirstlane(OFLG_LD(of + tl * 128)) < need) {
                        if (give_up(ctl, call, t0, spins)) { bad = true; break; }
                        __builtin_amdgcn_s_sleep(OPOLL_SLEEP);
                    }
                    if (u2 < 64) STAMP(3);
                    if (bad) r1 = r2 = ~0u;
                }
                const unsigned so = OFF_OEX + (unsigned)rank * (1u << 20) + tw * 8192u + b0 * 1024u;
                const unsigned row = (tw >> 6) * 256 + (b0 >> 2) * 128 + ((tw >> 2) & 1) * 64 + (b0 & 3) * 16, col = src * NS + ((tw >> 3) & 7) * 256 + (tw & 3) * 64;
                const unsigned vo = (lane & 15) * GN + (lane >> 4) * 16;
                u32x4 v = __builtin_amdgcn_raw_buffer_load_b128(rs, lane * 16, so, AUX_SYS);
                if (bad) v = (u32x4){0x7f7f7f7fu, 0x7f7f7f7fu, 0x7f7f7f7fu, 0x7f7f7f7fu};
                __builtin_amdgcn_raw_buffer_store_b128(v, ro, vo, row * GN + col, AUX_SYS);
                v = __builtin_amdgcn_raw_buffer_load_b128(rs, lane * 16, so + 1024, AUX_SYS);
                const int tnext = __builtin_amdgcn_raw_ptr_buffer_atomic_add_i32(1, rc, aoff, toff, 0);
                const unsigned f = __builtin_amdgcn_raw_buffer_load_b32(rc, (r2 != ~0u && lane < 16) ? lane * 128u : 0x80000000u, CTL_OFLG + src * 2048, AUX_SYS);
                if (bad) v = (u32x4){0x7f7f7f7fu, 0x7f7f7f7fu, 0x7f7f7f7fu, 0x7f7f7f7fu};
                r1 |= (unsigned)__ballot(f >= 2u * call - 1u) | 0xffff0000u; r2 |= (unsigned)__ballot(f >= 2u * call) | 0xffff0000u;
                __builtin_amdgcn_raw_buffer_store_b128(v, ro, vo, (row + 16) * GN + col, AUX_SYS);
                tk = (unsigned)__builtin_amdgcn_readfirstlane(tnext) - tbase;
            }
            WAIT_V(0);
            STAMP(4);
        }
        return;
    }

    const int gi = bid - (N_EXP + N_PULL), xg = gi & 7, ix = gi >> 3;
    const int ti = (xg >> 1) * 4 + (ix & 3), tj = (xg & 1) * 4 + (ix >> 2);
    const int wr = wid >> 2, wc = wid & 3;
    int scv;
    {
        int j0 = 0;
#pragma unroll
        for (int j = 1; j < 7; ++j) if (rot_of(j) <= ti) j0 = j;
        const int t2 = lane - 8, c = t2 / 7, k = t2 - c * 7;
        int j = j0 - k; if (j < 0) j += 7;
        scv = lane < 8 ? rank * 8 + lane : ((rank + 1 + j) & 7) * 8 + c;
    }
    const unsigned target = call * 32u;
    u64 avail = 0, taken = 0;
    auto next_step = [&](int n) -> unsigned {
        if (avail == 0) {
            if (wid == 0) {
                const u64 t0 = __builtin_amdgcn_s_memrealtime(); unsigned spins = 0;
                for (;;) {
                    const unsigned a = ld_agt(ctl + CTL_XCNT + (ti * 64 + scv) * 4), b = ld_agt(ctl + CTL_WCNT + (tj * 64 + scv) * 4);
                    avail = __ballot(a >= target && b >= target) & ~taken;
                    if (avail != 0) break;
                    if (give_up(ctl, call, t0, spins)) { avail = ~taken; break; }
                    if (n < 56) __builtin_amdgcn_s_sleep(POLL_SLEEP); else __builtin_amdgcn_s_sleep(1);
                }
                const u32x4 mv = {(unsigned)avail, (unsigned)(avail >> 32), (unsigned)n + 1u, 0u};
                asm volatile("ds_write_b128 %0, %1" :: "v"(lctl_addr + 16), "v"(mv) : "memory");
            } else {
                for (;;) {
                    u32x4 mv;
                    asm volatile("ds_read_b128 %0, %1\n\ts_waitcnt lgkmcnt(0)" : "=v"(mv) : "v"(lctl_addr + 16) : "memory");
                    if ((unsigned)__builtin_amdgcn_readfirstlane(mv.z) == (unsigned)n + 1u) {
                        avail = ((u64)(unsigned)__builtin_amdgcn_readfirstlane(mv.y) << 32) | (unsigned)__builtin_amdgcn_readfirstlane(mv.x);
                        break;
                    }
                    __builtin_amdgcn_s_sleep(1);
                }
            }
        }
        const int bit = __builtin_ctzll(avail);
        avail &= avail - 1; taken |= 1ull << bit;
        return (unsigned)__builtin_amdgcn_readlane(scv, bit);
    };
    unsigned pa = 0, pb = 0;
    const char* const pxa = ctl + CTL_XCNT + (ti * 64 + scv) * 4;
    const char* const pwa = ctl + CTL_WCNT + (tj * 64 + scv) * 4;
#define POLL_ISSUE do { if (t >= ASYNC_FROM && wid == 0) asm volatile("global_load_dword %0, %2, off sc1\n\tglobal_load_dword %1, %3, off sc1" : "=&v"(pa), "=&v"(pb) : "v"(pxa), "v"(pwa) : "memory"); } while (0)
#define POLL_TAKE(n) do { if (t >= ASYNC_FROM && wid == 0 && avail == 0) { asm volatile("" : "+v"(pa), "+v"(pb)); \
        avail = __ballot(pa >= target && pb >= target) & ~taken; \
        if (avail != 0) { const u32x4 mv = {(unsigned)avail, (unsigned)(avail >> 32), (unsigned)(n) + 1u, 0u}; \
            asm volatile("ds_write_b128 %0, %1" :: "v"(lctl_addr + 16), "v"(mv) : "memory"); } } } while (0)
    const char* const xbase = ws + OFF_X + (size_t)ti * PIECE;
    const char* const wbase = ws + OFF_WIN + (size_t)tj * PIECE;
#define A_OF(sc) (xbase + (size_t)((sc) * (16u * PIECE)))
#define B_OF(sc) (wbase + (size_t)((sc) * (8u * PIECE)))
    const unsigned voff = (unsigned)tid * 16u, ldsw = (unsigned)wid * 1024u;
    const int aoff = wr * 8192 + lane * 16, boff = wc * 4096 + lane * 16;
#define SA(b, h) (((b) * 2 + (h)) * HALFB)
#define SB(b, h) ((4 + (b) * 2 + (h)) * HALFB)
#define STAGE(bufoff, gbase) do { _Pragma("unroll") for (int _i = 0; _i < 2; ++_i) \
        __builtin_amdgcn_global_load_lds((const unsigned*)((gbase) + voff + _i * 8192), (LAS unsigned*)(lds + (bufoff) + ldsw + _i * 8192), 16, 0, AUX_AGT); } while (0)
#define LDA(dst, b, h) do { _Pragma("unroll") for (int m = 0; m < 4; ++m) _Pragma("unroll") for (int k = 0; k < 2; ++k) dst[m][k] = *(const LAS bf16x8*)(lds + SA(b, h) + aoff + m * 2048 + k * 1024); } while (0)
#define LDB(dst, b, h) do { _Pragma("unroll") for (int n = 0; n < 2; ++n) _Pragma("unroll") for (int k = 0; k < 2; ++k) dst[n][k] = *(const LAS bf16x8*)(lds + SB(b, h) + boff + n * 2048 + k * 1024); } while (0)
#define MMA(ai, bj, At, Bt) do { __builtin_amdgcn_s_setprio(1); _Pragma("unroll") for (int m = 0; m < 4; ++m) _Pragma("unroll") for (int n = 0; n < 2; ++n) _Pragma("unroll") for (int k = 0; k < 2; ++k) \
        acc[ai][bj][m][n] = __builtin_amdgcn_mfma_f32_16x16x32_bf16(Bt[n][k], At[m][k], acc[ai][bj][m][n], 0, 0, 0); __builtin_amdgcn_s_setprio(0); } while (0)
    f32x4 acc[2][2][4][2];
#pragma unroll
    for (int a = 0; a < 2; ++a)
#pragma unroll
        for (int b = 0; b < 2; ++b)
#pragma unroll
            for (int m = 0; m < 4; ++m)
#pragma unroll
                for (int n = 0; n < 2; ++n) acc[a][b][m][n] = (f32x4){0.f, 0.f, 0.f, 0.f};
    bf16x8 At[4][2], B0[2][2], B1[2][2];
    constexpr int NT = 64;
    const char* a1;
    {
        const unsigned s0 = next_step(0);
        const char *a0 = A_OF(s0), *b0 = B_OF(s0);
        STAGE(SB(0, 0), b0); STAGE(SA(0, 0), a0); STAGE(SB(0, 1), b0 + HALFB); STAGE(SA(0, 1), a0 + HALFB);
        if (wr == 1) BAR;
        WAIT_V(4); BAR;
        const unsigned s1 = next_step(1);
        a1 = A_OF(s1);
        const char* b1 = B_OF(s1);
        STAGE(SB(1, 0), b1); STAGE(SA(1, 0), a1); STAGE(SB(1, 1), b1 + HALFB);
        WAIT_V(6); BAR;
    }
    STAMP(1);
    for (int t = 0; t < NT - 2; t += 2) {
        LDB(B0, 0, 0); SCHED; LDA(At, 0, 0); POLL_ISSUE; STAGE(SA(1, 1), a1 + HALFB);
        WAIT_L(8); BAR; WAIT_L(0); MMA(0, 0, At, B0); BAR; SCHED;
        const unsigned s2 = next_step(t + 2);
        const char *a2 = A_OF(s2), *b2 = B_OF(s2);
        LDB(B1, 0, 1); STAGE(SB(0, 0), b2);
        BAR; WAIT_L(0); MMA(0, 1, At, B1); BAR;
        LDA(At, 0, 1); STAGE(SA(0, 0), a2);
        BAR; WAIT_L(0); MMA(1, 0, At, B0); BAR; SCHED;
        STAGE(SB(0, 1), b2 + HALFB);
        WAIT_V(6); POLL_TAKE(t + 3); BAR; MMA(1, 1, At, B1); BAR;
        LDB(B0, 1, 0); SCHED; LDA(At, 1, 0); POLL_ISSUE; STAGE(SA(0, 1), a2 + HALFB);
        WAIT_L(8); BAR; WAIT_L(0); MMA(0, 0, At, B0); BAR; SCHED;
        const unsigned s3 = next_step(t + 3);
        a1 = A_OF(s3);
        const char* b3 = B_OF(s3);
        LDB(B1, 1, 1); STAGE(SB(1, 0), b3);
        BAR; WAIT_L(0); MMA(0, 1, At, B1); BAR;
        LDA(At, 1, 1); STAGE(SA(1, 0), a1);
        BAR; WAIT_L(0); MMA(1, 0, At, B0); BAR; SCHED;
        STAGE(SB(1, 1), b3 + HALFB);
        WAIT_V(6); POLL_TAKE(t + 4); BAR; MMA(1, 1, At, B1); BAR;
    }
    STAMP(2);
    {
        LDB(B0, 0, 0); LDA(At, 0, 0); STAGE(SA(1, 1), a1 + HALFB);
        BAR; WAIT_L(0); MMA(0, 0, At, B0); BAR;
        LDB(B1, 0, 1); BAR; WAIT_L(0); MMA(0, 1, At, B1); BAR;
        LDA(At, 0, 1); WAIT_V(0); BAR; WAIT_L(0); MMA(1, 0, At, B0); MMA(1, 1, At, B1); BAR;
    }
    {
        LDB(B0, 1, 0); LDA(At, 1, 0); BAR; WAIT_L(0); MMA(0, 0, At, B0); BAR;
        LDB(B1, 1, 1); BAR; WAIT_L(0); MMA(0, 1, At, B1); BAR;
        LDA(At, 1, 1); BAR; WAIT_L(0); MMA(1, 0, At, B0); MMA(1, 1, At, B1); BAR;
    }
    if (wr == 0) BAR;
    STAMP(3);

    float mx = 0.f;
#pragma unroll
    for (int a = 0; a < 2; ++a)
#pragma unroll
        for (int b = 0; b < 2; ++b)
#pragma unroll
            for (int m = 0; m < 4; ++m)
#pragma unroll
                for (int n = 0; n < 2; ++n)
#pragma unroll
                    for (int e = 0; e < 4; ++e) mx = fmaxf(mx, acc[a][b][m][n][e]);
    {
        const unsigned wm = wave_max_u32(__float_as_uint(mx));
        if (lane == 63) __hip_atomic_fetch_max(lctl + 8, wm, __ATOMIC_RELAXED, __HIP_MEMORY_SCOPE_WORKGROUP);
    }
    __syncthreads();
    if (tid < NR) {
        char* pw = (char*)(size_t)*(LAS u64*)(lctl + 16 + 2 * tid);
        typedef __attribute__((address_space(1))) u64 gu64;
        __hip_atomic_store((gu64*)(pw + OFF_CTL + CTL_AMAX + (rank * 128 + gi) * 8), ((u64)call << 32) | lctl[8], __ATOMIC_RELAXED, __HIP_MEMORY_SCOPE_SYSTEM);
    }
    STAMP(6);
#pragma unroll
    for (int a = 0; a < 2; ++a)
#pragma unroll
        for (int b = 0; b < 2; ++b)
#pragma unroll
            for (int m = 0; m < 4; ++m)
#pragma unroll
                for (int n = 0; n < 2; ++n) {
#pragma unroll
                    for (int e = 0; e < 4; ++e) acc[a][b][m][n][e] = fmaxf(acc[a][b][m][n][e], 0.f);
                    asm volatile("" : "+v"(acc[a][b][m][n]));
                }
    const int owner = ti >> 1;
    char* optr = p.out[owner];
    char* octl = p.ws[owner] + OFF_CTL;
    asm volatile("" : "+s"(optr), "+s"(octl));
    const __amdgpu_buffer_rsrc_t ro = mk_rsrc(optr, OUT_BYTES), rx = mk_rsrc(ws + OFF_OEX, 8u << 20);
    const unsigned obase = (unsigned)((ti & 1) * 256 + wr * 64 + (lane & 15)) * GN + (unsigned)(rank * NS + tj * 256 + wc * 64 + (lane >> 4) * 16);
    const unsigned tw = (unsigned)(((ti & 1) * 8 + tj) * 8 + wid), xbase2 = (unsigned)owner * (1u << 20) + tw * 8192u;
    const bool local = owner == rank;
    unsigned gmx, tmo;
    {
        const __amdgpu_buffer_rsrc_t ra = mk_rsrc(ctl + CTL_AMAX, 8192);
        const u64 t0 = __builtin_amdgcn_s_memrealtime(); unsigned spins = 0;
        for (;;) {
            const u32x4 g = __builtin_amdgcn_raw_buffer_load_b128(ra, tid * 16, 0, AMAX_AUX);
            tmo = ld_agt(ctl + CTL_TMO);
            gmx = max(g.x, g.z);
            if (__all(g.y >= call && g.w >= call)) break;
            if (give_up(ctl, call, t0, spins)) { tmo = call; break; }
            __builtin_amdgcn_s_sleep(AMAX_SLEEP);
        }
    }
    if (tmo == call) gmx = 0xffffffffu;
    gmx = wave_max_u32(gmx);
    if (lane == 63) __hip_atomic_fetch_max(lctl + 9, gmx, __ATOMIC_RELAXED, __HIP_MEMORY_SCOPE_WORKGROUP);
    __syncthreads();
    STAMP(4);
    gmx = lctl[9];
    const float scale = __uint_as_float(gmx) / 448.0f, inv = 1.0f / scale;
    typedef float f32x2 __attribute__((ext_vector_type(2)));
    const f32x2 inv2 = {inv, inv};
#pragma unroll
    for (int a = 0; a < 2; ++a)
#pragma unroll
        for (int m = 0; m < 4; ++m) {
            u32x4 o;
#pragma unroll
            for (int b = 0; b < 2; ++b)
#pragma unroll
                for (int n = 0; n < 2; ++n) {
                    const f32x4 v = acc[a][b][m][n];
                    const f32x2 lo = (f32x2){v[0], v[1]} * inv2, hi = (f32x2){v[2], v[3]} * inv2;
                    int r = __builtin_amdgcn_cvt_pk_fp8_f32(lo.x, lo.y, 0, false);
                    r = __builtin_amdgcn_cvt_pk_fp8_f32(hi.x, hi.y, r, true);
                    o[b * 2 + n] = (unsigned)r;
                }
            if (!local) __builtin_amdgcn_raw_buffer_store_b128(o, rx, lane * 16, xbase2 + (a * 4 + m) * 1024, AUX_SYS);
            else __builtin_amdgcn_raw_buffer_store_b128(o, ro, obase + (unsigned)(a * 128 + m * 16) * GN, 0, AUX_SYS);
            if (a * 4 + m == OFLG_AT && !local) {
                asm volatile("s_waitcnt vmcnt(%0)" :: "i"(OFLG_AT - 1) : "memory");
                BAR;
                if (tid == 0) st_sys(octl + CTL_OFLG + (rank * 16 + (tw >> 3)) * 128, 2u * call - 1u);
            }
        }
    if (!local) {
        STAMP(7);
        WAIT_V(0);
        BAR;
        if (tid == 0) st_sys(octl + CTL_OFLG + (rank * 16 + (tw >> 3)) * 128, 2u * call);
    }
    STAMP(5);
}

}

extern "C" void kernel_launch(int rank, int nranks, ncclComm_t comm,
                              hipStream_t stream,
                              void* const* d_in, const int* in_sizes, int n_in,
                              void* d_out, int out_size,
                              const int* in_dtypes, int out_dtype,
                              void* const* d_out_all,
                              void* const* d_ws_all, size_t ws_bytes) {
    (void)comm; (void)in_dtypes; (void)out_dtype; (void)d_out;
    if (nranks != NR || (unsigned)rank >= (unsigned)NR || n_in < 2 || ws_bytes < WS_NEED || in_sizes[0] != GM * KS ||
        in_sizes[1] != KS * GN || (unsigned)out_size != OUT_BYTES) {
        fprintf(stderr, "kernel_launch: unexpected problem shape\n");
        return;
    }
    HIP_CHECK(hipFuncSetAttribute((const void*)fused_gemm_quant, hipFuncAttributeMaxDynamicSharedMemorySize, LDS_BYTES));
    Params p;
    p.x = (const char*)d_in[0];
    p.w = (const char*)d_in[1];
    for (int r = 0; r < NR; ++r) { p.ws[r] = (char*)d_ws_all[r]; p.out[r] = (char*)d_out_all[r]; }
    p.rank = rank;
    hipLaunchKernelGGL(fused_gemm_quant, dim3(N_WG), dim3(512), LDS_BYTES, stream, p);
    HIP_CHECK(hipGetLastError());
}
```

```cpp
#include <hip/hip_runtime.h>
#include <rccl/rccl.h>
#include <cstdio>
#include <cstdint>

#define HIP_CHECK(expr) do { hipError_t _e = (expr); if (_e != hipSuccess) { \
    fprintf(stderr, "HIP error %s at %s:%d\n", hipGetErrorString(_e), __FILE__, __LINE__); \
    return; } } while (0)

#define N_PUSH 2
#define MERGE_B 512u
#define BM_AT 15
#define EARLY_FROM 57
#define LATE_SLEEP 1
#define POLL_SLEEP 16
#define OPOLL_SLEEP 8

namespace {

#define LAS __attribute__((address_space(3)))
typedef short bf16x8 __attribute__((ext_vector_type(8)));
typedef short s16x4 __attribute__((ext_vector_type(4)));
typedef float f32x4 __attribute__((ext_vector_type(4)));
typedef float f32x2 __attribute__((ext_vector_type(2)));
typedef int i32x4 __attribute__((ext_vector_type(4)));
typedef unsigned u32x4 __attribute__((ext_vector_type(4)));
typedef unsigned u32x3 __attribute__((ext_vector_type(3)));
typedef unsigned u32x2 __attribute__((ext_vector_type(2)));
typedef unsigned long long u64;
typedef __attribute__((address_space(1))) unsigned gu32;
typedef __attribute__((address_space(1))) u64 gu64;

constexpr int NR = 8;
constexpr int GM = 4096, GN = 16384;
constexpr int KS = 512, NS = 2048, MS = 512;
constexpr unsigned PIECE = 32768;
constexpr unsigned HALFB = 16384;
constexpr unsigned OSLOT = 16384;
constexpr unsigned OFF_X = 0;
constexpr unsigned OFF_WIN = 32u << 20;
constexpr unsigned OFF_WEX = 48u << 20;
constexpr unsigned OFF_OEX = 64u << 20;
constexpr unsigned OFF_CTL = 80u << 20;
constexpr unsigned CTL_SEQ = 0;
constexpr unsigned CTL_OFLG = 16384;
constexpr unsigned CTL_TMO = 32768;
constexpr unsigned CTL_EXPF = 36864;
constexpr unsigned CTL_XCNT = 45056;
constexpr unsigned CTL_WCNT = 49152;
constexpr unsigned CTL_AMAX = 53248;
constexpr unsigned CTL_TICK = 63488;
constexpr unsigned CTL_DBG = 65536;
constexpr size_t WS_NEED = (size_t)OFF_CTL + (1u << 20);
constexpr unsigned OUT_BYTES = (unsigned)MS * GN;
constexpr unsigned OOB = 0x80000000u;

constexpr int N_PULL = 56, N_GEMM = 128, N_EXP = 72, N_WG = N_EXP + N_PULL + N_GEMM;
constexpr int LAREA = 9216;
constexpr int LDS_CTL = 16 * LAREA, LDS_BYTES = LDS_CTL + 3072;
static_assert(LDS_CTL >= 131072, "the GEMM staging buffers live below the control words");
constexpr u64 DEADLINE = 50000000ull;

struct Params {
    const char* x;
    const char* w;
    char* ws[NR];
    char* out[NR];
    int rank;
};

__device__ __forceinline__ __amdgpu_buffer_rsrc_t mk_rsrc(const void* p, unsigned bytes) {
    return __builtin_amdgcn_make_buffer_rsrc(const_cast<void*>(p), 0, bytes, 0x00020000);
}
constexpr int AUX_SYS = 17;
constexpr int AUX_AGT = 16;

template <int AUX> __device__ __forceinline__ void st128(u32x4 v, __amdgpu_buffer_rsrc_t r, unsigned voff, unsigned soff) { __builtin_amdgcn_raw_buffer_store_b128(v, r, voff + soff, 0, AUX); }
template <int AUX> __device__ __forceinline__ void st96(u32x3 v, __amdgpu_buffer_rsrc_t r, unsigned voff, unsigned soff) { __builtin_amdgcn_raw_buffer_store_b96(v, r, voff + soff, 0, AUX); }
__device__ __forceinline__ unsigned ld_sys(const void* p) { return __hip_atomic_load((const gu32*)(size_t)p, __ATOMIC_RELAXED, __HIP_MEMORY_SCOPE_SYSTEM); }
__device__ __forceinline__ unsigned ld_agt(const void* p) { return __hip_atomic_load((const gu32*)(size_t)p, __ATOMIC_RELAXED, __HIP_MEMORY_SCOPE_AGENT); }
__device__ __forceinline__ void st_sys(void* p, unsigned v) { __hip_atomic_store((gu32*)(size_t)p, v, __ATOMIC_RELAXED, __HIP_MEMORY_SCOPE_SYSTEM); }
#define WAIT_V(n) asm volatile("s_waitcnt vmcnt(" #n ")" ::: "memory")
#define WAIT_L(n) asm volatile("s_waitcnt lgkmcnt(" #n ")" ::: "memory")
#define BAR __builtin_amdgcn_s_barrier()
#define SCHED __builtin_amdgcn_sched_barrier(0)

__device__ __forceinline__ bool give_up(char* ctl, unsigned call, u64 t0, unsigned& spins) {
    if ((++spins & 31u) != 0u) return false;
    const unsigned tm = __builtin_amdgcn_readfirstlane(ld_agt(ctl + CTL_TMO));
    if (tm == call) return true;
    if (__builtin_amdgcn_s_memrealtime() - t0 > DEADLINE) {
        __hip_atomic_store((gu32*)(size_t)(ctl + CTL_TMO), call, __ATOMIC_RELAXED, __HIP_MEMORY_SCOPE_AGENT);
        return true;
    }
    return false;
}
#define DPP(v, ctrl, rows) ((unsigned)__builtin_amdgcn_update_dpp(0, (int)(v), ctrl, rows, 0xf, false))
__device__ __forceinline__ unsigned wave_max_u32(unsigned v) {
    v = max(v, DPP(v, 0x111, 0xf)); v = max(v, DPP(v, 0x112, 0xf)); v = max(v, DPP(v, 0x114, 0xf)); v = max(v, DPP(v, 0x118, 0xf));
    v = max(v, DPP(v, 0x142, 0xa)); v = max(v, DPP(v, 0x143, 0xc));
    return v;
}
__device__ __forceinline__ unsigned wave_scan_u32(unsigned v) {
    v += DPP(v, 0x111, 0xf); v += DPP(v, 0x112, 0xf); v += DPP(v, 0x114, 0xf); v += DPP(v, 0x118, 0xf);
    v += DPP(v, 0x142, 0xa); v += DPP(v, 0x143, 0xc);
    return v;
}
__device__ __forceinline__ unsigned stream_base(const u32x4 bm, unsigned area, unsigned& a) {
    const unsigned len = __builtin_popcount(bm.x) + __builtin_popcount(bm.y) + __builtin_popcount(bm.z) + __builtin_popcount(bm.w);
    const unsigned reg = (len + 3u) & ~3u, inc = wave_scan_u32(reg);
    a = area + inc - reg;
    return (unsigned)__builtin_amdgcn_readlane((int)inc, 63);
}

#define STAMP(k) do { } while (0)

__device__ __forceinline__ int rot_of(int j) { return (j * 16 + 3) / 7; }

__global__ void __launch_bounds__(512, 2) fused_gemm_quant(Params p) {
    extern __shared__ __attribute__((aligned(16))) unsigned char smem[];
    LAS unsigned char* lds = (LAS unsigned char*)smem;
    const int tid = threadIdx.x, lane = tid & 63;
    const int wid = __builtin_amdgcn_readfirstlane(tid >> 6);
    const int rank = p.rank, bid = blockIdx.x;
    char* const ws = p.ws[rank];
    char* const ctl = ws + OFF_CTL;
    STAMP(0);
    const __amdgpu_buffer_rsrc_t rc = mk_rsrc(ctl, 65536);
    const unsigned call_v = (unsigned)__builtin_amdgcn_raw_ptr_buffer_atomic_add_i32(1, rc, tid == 0 ? 0u : OOB, CTL_SEQ + bid * 64, 0) + 1u;
    LAS unsigned* lctl = (LAS unsigned*)(lds + LDS_CTL);
    const unsigned lds0 = (unsigned)(size_t)lds, lctl_addr = lds0 + LDS_CTL;

    if (bid < N_EXP) {
        const __amdgpu_buffer_rsrc_t rx = mk_rsrc(p.x, (unsigned)GM * KS * 2), rw = mk_rsrc(p.w, (unsigned)KS * GN * 2);
        const __amdgpu_buffer_rsrc_t rd = mk_rsrc(ws, OFF_CTL);
        constexpr int PITCH = 528;
        for (int q = bid; q < 640; q += N_EXP) {
            const int c = q / 80, idx = q % 80;
            const bool isx = idx >= 56 && idx < 72, push = q < 7 * N_PUSH;
            int dst = rank, pj = 0;
            if (isx) {
                pj = idx - 56;
                u32x4 v[4];
                const unsigned src = (unsigned)((pj * 256 + wid * 32 + (lane & 15)) * (KS * 2) + c * 128 + (lane >> 4) * 16);
#pragma unroll
                for (int u = 0; u < 4; ++u) v[u] = __builtin_amdgcn_raw_buffer_load_b128(rx, src + (u >> 1) * 16 * (KS * 2) + (u & 1) * 64, 0, 0);
                const unsigned d0 = OFF_X + (unsigned)((rank * 8 + c) * 16 + pj) * PIECE + wid * 4096 + lane * 16;
#pragma unroll
                for (int u = 0; u < 4; ++u) st128<AUX_SYS>(v[u], rd, d0 + u * 1024, 0);
            } else {
                if (idx < 56) { dst = (rank + 1 + idx % 7) & 7; pj = idx / 7; } else { pj = idx - 72; }
                u32x4 v[4];
                const unsigned src = (unsigned)(c * 64 + (tid >> 5)) * (GN * 2) + (unsigned)(dst * NS + pj * 256) * 2 + (tid & 31) * 16;
#pragma unroll
                for (int u = 0; u < 4; ++u) v[u] = __builtin_amdgcn_raw_buffer_load_b128(rw, src + u * 16 * (GN * 2), 0, 0);
#pragma unroll
                for (int u = 0; u < 4; ++u) *(LAS u32x4*)(lds + ((tid >> 5) + 16 * u) * PITCH + (tid & 31) * 16) = v[u];
                __syncthreads();
                const int bj = wid >> 2, wc = wid & 3;
                const int g = lane >> 4, qq = (lane >> 2) & 3, pp = lane & 3;
                const __amdgpu_buffer_rsrc_t rw2 = mk_rsrc(push ? p.ws[dst] : ws, OFF_CTL);
                const unsigned d0 = (push ? OFF_WIN + (unsigned)(rank * 64 + pj) * PIECE : (dst == rank ? OFF_WIN : OFF_WEX) + (unsigned)((dst * 8 + c) * 8 + pj) * PIECE) + wid * 4096 + lane * 16;
#pragma unroll
                for (int u = 0; u < 4; ++u) {
                    const int n = u >> 1, kf = u & 1;
                    const int a = (kf * 32 + g * 8 + qq) * PITCH + (wc * 64 + pp * 16 + (bj * 2 + n) * 4) * 2;
                    const s16x4 lo = __builtin_amdgcn_ds_read_tr16_b64_v4i16((LAS s16x4*)(lds + a));
                    const s16x4 hi = __builtin_amdgcn_ds_read_tr16_b64_v4i16((LAS s16x4*)(lds + a + 4 * PITCH));
                    const u32x2 l2 = __builtin_bit_cast(u32x2, lo), h2 = __builtin_bit_cast(u32x2, hi);
                    const u32x4 o = {l2.x, l2.y, h2.x, h2.y};
                    st128<AUX_SYS>(o, rw2, d0 + u * 1024, 0);
                }
            }
            WAIT_V(0);
            __syncthreads();
            if (wid == 0) {
                const unsigned call = __builtin_amdgcn_readfirstlane(call_v);
                if (isx) {
                    if (lane < NR) {
                        if (lane == rank) st_sys(ctl + CTL_XCNT + (pj * 64 + rank * 8 + c) * 4, call * 32u);
                        else st_sys(p.ws[lane] + OFF_CTL + CTL_EXPF + (rank * 256 + c * 24 + 8 + pj) * 4, call);
                    }
                } else if (lane == 0) {
                    if (dst == rank || push) st_sys(p.ws[dst] + OFF_CTL + CTL_WCNT + (pj * 64 + rank * 8 + c) * 4, call * 32u);
                    else st_sys(p.ws[dst] + OFF_CTL + CTL_EXPF + (rank * 256 + c * 24 + pj) * 4, call);
                }
            }
            if (q == bid) STAMP(1);
        }
        STAMP(2);
        return;
    }

    if (tid == 0) { lctl[0] = call_v; lctl[6] = 0u; lctl[8] = 0u; lctl[9] = 0u; }
    if (tid >= 64 && tid < 64 + NR) *(LAS u64*)(lctl + 16 + 2 * (tid - 64)) = (u64)(size_t)p.ws[tid - 64];
    if (tid >= 128 && tid < 192) {
        const unsigned m = (tid - 128) >> 2, off = tid & 3;
        u64 cs = 0x0c0c0c0c0c0c0c0cull; unsigned es = 0x0c0c0c0cu, k = off;
        for (unsigned e = 0; e < off; ++e) cs = (cs & ~(0xffull << (8 * e))) | ((u64)(4 + e) << (8 * e));
        for (unsigned e = 0; e < 4; ++e)
            if ((m >> e) & 1u) { cs = (cs & ~(0xffull << (8 * k))) | ((u64)e << (8 * k)); es = (es & ~(0xffu << (8 * e))) | (k << (8 * e)); ++k; }
        *(LAS u32x2*)(lctl + 256 + 2 * (tid - 128)) = (u32x2){(unsigned)cs, k >= 4 ? (unsigned)(cs >> 32) : (unsigned)cs};
        lctl[64 + tid - 128] = es;
    }
    __syncthreads();
    const unsigned call = __builtin_amdgcn_readfirstlane(lctl[0]);

    if (bid < N_EXP + N_PULL) {
        const int q = bid - N_EXP, j = q % 7;
        const int src = (rank + 1 + j) & 7, rot = rot_of(j);
        const __amdgpu_buffer_rsrc_t rs = mk_rsrc(p.ws[src], OFF_CTL), rd = mk_rsrc(ws, OFF_CTL);
        const char* ef = ctl + CTL_EXPF + src * 1024 + lane * 4;
        u64 m0 = 0, m1 = 0, m2 = 0;
        const unsigned aoff = lane == 0 ? 0u : OOB;
        constexpr unsigned NU = (192 - N_PUSH) * 16;
        const unsigned tbase = (call - 1u) * (NU + 64u), toff = CTL_TICK + j * 128;
        unsigned cnt1 = 0, off1 = OOB;
        unsigned tk = (unsigned)__builtin_amdgcn_readfirstlane(__builtin_amdgcn_raw_ptr_buffer_atomic_add_i32(1, rc, aoff, toff, 0)) - tbase;
        while (tk < NU) {
            const int pp = tk / 16 + N_PUSH, c = pp / 24, r = pp % 24, sc = src * 8 + c;
            int id; unsigned so, dof, cnt;
            if (r < 8) { id = c * 24 + r; so = OFF_WEX + (unsigned)((rank * 8 + c) * 8 + r) * PIECE; dof = OFF_WIN + (unsigned)(sc * 8 + r) * PIECE; cnt = CTL_WCNT + (r * 64 + sc) * 4; }
            else { const int i = (rot + r - 8) & 15; id = c * 24 + 8 + i; so = OFF_X + (unsigned)(sc * 16 + i) * PIECE; dof = so; cnt = CTL_XCNT + (i * 64 + sc) * 4; }
            so += (tk % 16) * 2048u; dof += (tk % 16) * 2048u;
            const u64 mm = id < 64 ? m0 : (id < 128 ? m1 : m2);
            if (!((mm >> (id & 63)) & 1)) {
                const u64 t0 = __builtin_amdgcn_s_memrealtime(); unsigned spins = 0;
                for (;;) {
                    const unsigned f0 = ld_sys(ef), f1 = ld_sys(ef + 256), f2 = ld_sys(ef + 512);
                    m0 = __ballot(f0 >= call); m1 = __ballot(f1 >= call); m2 = __ballot(f2 >= call);
                    const u64 mn = id < 64 ? m0 : (id < 128 ? m1 : m2);
                    if ((mn >> (id & 63)) & 1) break;
                    if (give_up(ctl, call, t0, spins)) { m0 = m1 = m2 = ~0ull; break; }
                    __builtin_amdgcn_s_sleep(2);
                }
                if (off1 == OOB) STAMP(1);
            }
            u32x4 v = __builtin_amdgcn_raw_buffer_load_b128(rs, lane * 16, so, AUX_SYS);
            WAIT_V(0);
            st128<AUX_AGT>(v, rd, lane * 16, dof);
            v = __builtin_amdgcn_raw_buffer_load_b128(rs, lane * 16, so + 1024, AUX_SYS);
            const int tnext = __builtin_amdgcn_raw_ptr_buffer_atomic_add_i32(1, rc, aoff, toff, 0);
            __builtin_amdgcn_raw_ptr_buffer_atomic_add_i32(2, rc, off1, cnt1, 0);
            WAIT_V(1);
            st128<AUX_AGT>(v, rd, lane * 16, dof + 1024);
            cnt1 = cnt; off1 = aoff;
            tk = (unsigned)__builtin_amdgcn_readfirstlane(tnext) - tbase;
        }
        WAIT_V(0);
        __builtin_amdgcn_raw_ptr_buffer_atomic_add_i32(2, rc, off1, cnt1, 0);
        STAMP(2);

        const int g = (q / 7) * 8 + wid, tA = g & 15, tB = tA ^ 8, w2 = g >> 4;
        const __amdgpu_buffer_rsrc_t ro = mk_rsrc(p.out[rank], OUT_BYTES);
        const char* ofA = ctl + CTL_OFLG + (src * 16 + tA) * 128;
        const char* ofB = ctl + CTL_OFLG + (src * 16 + tB) * 128;
        const unsigned slA = OFF_OEX + (unsigned)((rank * 16 + tA) * 8 + w2) * OSLOT, slB = OFF_OEX + (unsigned)((rank * 16 + tB) * 8 + w2 + 4) * OSLOT;
        const unsigned arA = lds0 + wid * 2 * LAREA, arB = arA + LAREA;
        const unsigned obase = (unsigned)(lane & 15) * GN + (unsigned)(src * NS + (tA & 7) * 256 + w2 * 64 + (lane >> 4) * 16);
        const unsigned rowA = (tA >> 3) * 256, rowB = (tB >> 3) * 256 + 64;
        unsigned adA[32], seA[32], adB[32], seB[32], pA, pB, lenA, lenB;
        u32x4 bmA, bmB;
#define PLAN(X, i0, i1) do { _Pragma("unroll") for (int i = i0; i < i1; ++i) { const unsigned nib = (bm##X[i >> 3] >> (4 * (i & 7))) & 15u; \
                ad##X[i] = p##X & ~3u; se##X[i] = lctl[64 + nib * 4 + (p##X & 3u)]; p##X += __builtin_popcount(nib); } \
            _Pragma("unroll") for (int i = i0; i < i1; ++i) asm volatile("" : "+v"(ad##X[i]), "+v"(se##X[i])); } while (0)
        bool ok = true;
        unsigned fl = 0;
        auto wait2 = [&](unsigned need, unsigned who) {
            const u64 t0 = __builtin_amdgcn_s_memrealtime(); unsigned spins = 0;
            for (;;) {
                fl = ld_sys(lane & 1 ? ofB : ofA);
                if ((unsigned)__ballot(fl >= need) & who) break;
                if (give_up(ctl, call, t0, spins)) { ok = false; fl = ~0u; break; }
                __builtin_amdgcn_s_sleep(OPOLL_SLEEP);
            }
        };
        constexpr unsigned whoA = 1, whoB = 2;
        wait2(2u * call - 1u, 3);
        const bool a_first = (unsigned)__builtin_amdgcn_readlane((int)fl, 0) >= 2u * call - 1u;
        bool both = false;
        STAMP(3);
#define WSTAMP(k) do { } while (0)
#define LOAD_EVEN(X, k) __builtin_amdgcn_raw_buffer_load_b128(rs, (k) * 1024u + lane * 16u < len##X ? lane * 16u : OOB, sl##X + 1024u + (k) * 1024u, AUX_SYS)
#define LOAD_ODD(X, k) __builtin_amdgcn_raw_buffer_load_b96(rs, (k) * 1024u + lane * 16u < len##X ? lane * 12u : OOB, sl##X + 1024u + (k) * 1024u, AUX_SYS)
#define PUT_EVEN(X, k, v) do { d7 = ((v.x >> 7) & 0x01010101u) | ((v.y >> 6) & 0x02020202u) | ((v.z >> 5) & 0x04040404u) | ((v.w >> 4) & 0x08080808u); \
            *(LAS u32x4*)(size_t)(ar##X + (k) * 1024u + lane * 16u) = v & 0x7f7f7f7fu; } while (0)
#define PUT_ODD(X, k, w) do { d7 |= ((w.x >> 3) & 0x10101010u) | ((w.y >> 2) & 0x20202020u) | ((w.z >> 1) & 0x40404040u); \
            *(LAS u32x4*)(size_t)(ar##X + (k) * 1024u + lane * 16u) = (u32x4){w.x & 0x7f7f7f7fu, w.y & 0x7f7f7f7fu, w.z & 0x7f7f7f7fu, d7}; } while (0)
#define REST(X, other) do { bool have = false; u32x4 v; \
            for (unsigned k = 2; k * 1024u < len##X; k += 2) { \
                if (!have) v = LOAD_EVEN(X, k); \
                PUT_EVEN(X, k, v); \
                if ((k + 1) * 1024u < len##X) { \
                    const u32x3 w = LOAD_ODD(X, k + 1); \
                    have = len##X <= (k + 2) * 1024u + MERGE_B; \
                    if (have) v = LOAD_EVEN(X, k + 2); \
                    const unsigned fo = __builtin_amdgcn_raw_buffer_load_b32(rc, both ? OOB : aoff, CTL_OFLG + (src * 16 + (other)) * 128, AUX_SYS); \
                    PUT_ODD(X, k + 1, w); \
                    both = both || (unsigned)__builtin_amdgcn_readfirstlane(fo) >= 2u * call; \
                } } } while (0)
#define EXPAND(X) do { unsigned hi[32], lo = *(const LAS unsigned*)(size_t)ad##X[0]; \
            _Pragma("unroll") for (int i = 0; i < 32; ++i) hi[i] = *(const LAS unsigned*)(size_t)(ad##X[i] + 4u); \
            if (!ok) { _Pragma("unroll") for (int i = 0; i < 32; ++i) se##X[i] = ~0u; }        \
            _Pragma("unroll") for (int blk = 0; blk < 8; ++blk) { u32x4 o; \
                _Pragma("unroll") for (int e = 0; e < 4; ++e) { const int i = blk * 4 + e; \
                    if (i) lo = ad##X[i] != ad##X[i - 1] ? hi[i - 1] : lo; \
                    o[e] = __builtin_amdgcn_perm(hi[i], lo, se##X[i]); } \
                st128<AUX_SYS>(o, ro, obase, (row##X + (blk >> 2) * 128 + (blk & 3) * 16) * GN); } } while (0)
#define RUN(X, Y) do { unsigned d7; \
            bm##X = __builtin_amdgcn_raw_buffer_load_b128(rs, lane * 16, sl##X, AUX_SYS); \
            const bool early = (unsigned)__ballot(fl >= 2u * call - 1u) & who##Y; \
            bm##Y = __builtin_amdgcn_raw_buffer_load_b128(rs, early ? lane * 16u : OOB, sl##Y, AUX_SYS); \
            len##X = stream_base(bm##X, ar##X, p##X); \
            PLAN(X, 0, 32); \
            if (!early) { wait2(2u * call - 1u, who##Y); bm##Y = __builtin_amdgcn_raw_buffer_load_b128(rs, lane * 16, sl##Y, AUX_SYS); } \
            len##Y = stream_base(bm##Y, ar##Y, p##Y); \
            wait2(2u * call, who##X); \
            both = (unsigned)__ballot(fl >= 2u * call) & who##Y; \
            STAMP(5); WSTAMP(0); \
            { const u32x4 v = LOAD_EVEN(X, 0u); PLAN(Y, 0, 16); PUT_EVEN(X, 0u, v); } \
            { const u32x3 w = LOAD_ODD(X, 1u); PLAN(Y, 16, 32); PUT_ODD(X, 1u, w); } \
            REST(X, t##Y); \
            WSTAMP(1); \
            if (!both) { wait2(2u * call, who##Y); both = true; } \
            const u32x4 v0 = LOAD_EVEN(Y, 0u); const u32x3 w1 = LOAD_ODD(Y, 1u);        \
            EXPAND(X); \
            WSTAMP(2); \
            PUT_EVEN(Y, 0u, v0); PUT_ODD(Y, 1u, w1); \
            REST(Y, t##X); \
            WSTAMP(3); \
            EXPAND(Y); } while (0)
        if (a_first) RUN(A, B); else RUN(B, A);
        WSTAMP(4);
        WAIT_V(0);
        WSTAMP(5);
        STAMP(4);
        return;
    }

    const int gi = bid - (N_EXP + N_PULL), xg = gi & 7, ix = gi >> 3;
    const int ti = (xg >> 1) * 4 + (ix & 3), tj = (xg & 1) * 4 + (ix >> 2);
    const int wr = wid >> 2, wc = wid & 3;
    int scv;
    {
        int j0 = 0;
#pragma unroll
        for (int j = 1; j < 7; ++j) if (rot_of(j) <= ti) j0 = j;
        const int t2 = lane - 8, c = t2 / 7, k = t2 - c * 7;
        int j = j0 - k; if (j < 0) j += 7;
        scv = lane < 8 ? rank * 8 + lane : ((rank + 1 + j) & 7) * 8 + c;
    }
    const unsigned target = call * 32u;
    u64 avail = 0, taken = 0;
    unsigned pa = 0, pb = 0;
    const char* const pxa = ctl + CTL_XCNT + (ti * 64 + scv) * 4;
    const char* const pwa = ctl + CTL_WCNT + (tj * 64 + scv) * 4;
#define POLL_WAIT_V6(n) do { if ((n) >= EARLY_FROM && (n) < 64 && wid == 0 && avail == 0) { \
        asm volatile("global_load_dword %0, %2, off sc1\n\tglobal_load_dword %1, %3, off sc1\n\ts_waitcnt vmcnt(8)" : "=&v"(pa), "=&v"(pb) : "v"(pxa), "v"(pwa) : "memory"); \
    } else WAIT_V(6); } while (0)
    auto next_step = [&](int n) -> unsigned {
        if (avail == 0) {
            if (wid == 0) {
                if (n >= EARLY_FROM) {
                    asm volatile("s_waitcnt vmcnt(2)" : "+v"(pa), "+v"(pb) :: "memory");
                    avail = __ballot(pa >= target && pb >= target) & ~taken;
                }
                const u64 t0 = __builtin_amdgcn_s_memrealtime(); unsigned spins = 0;
                while (avail == 0) {
                    const unsigned a = ld_agt(pxa), b = ld_agt(pwa);
                    avail = __ballot(a >= target && b >= target) & ~taken;
                    if (avail != 0) break;
                    if (give_up(ctl, call, t0, spins)) { avail = ~taken; break; }
                    if (n < 56) __builtin_amdgcn_s_sleep(POLL_SLEEP); else __builtin_amdgcn_s_sleep(LATE_SLEEP);
                }
                const u32x4 mv = {(unsigned)avail, (unsigned)(avail >> 32), (unsigned)n + 1u, 0u};
                asm volatile("ds_write_b128 %0, %1" :: "v"(lctl_addr + 16), "v"(mv) : "memory");
            } else {
                for (;;) {
                    u32x4 mv;
                    asm volatile("ds_read_b128 %0, %1\n\ts_waitcnt lgkmcnt(0)" : "=v"(mv) : "v"(lctl_addr + 16) : "memory");
                    if ((unsigned)__builtin_amdgcn_readfirstlane(mv.z) == (unsigned)n + 1u) {
                        avail = ((u64)(unsigned)__builtin_amdgcn_readfirstlane(mv.y) << 32) | (unsigned)__builtin_amdgcn_readfirstlane(mv.x);
                        break;
                    }
                    __builtin_amdgcn_s_sleep(1);
                }
            }
        }
        const int bit = __builtin_ctzll(avail);
        avail &= avail - 1; taken |= 1ull << bit;
        return (unsigned)__builtin_amdgcn_readlane(scv, bit);
    };
    const char* const xbase = ws + OFF_X + (size_t)ti * PIECE;
    const char* const wbase = ws + OFF_WIN + (size_t)tj * PIECE;
#define A_OF(sc) (xbase + (size_t)((sc) * (16u * PIECE)))
#define B_OF(sc) (wbase + (size_t)((sc) * (8u * PIECE)))
    const unsigned voff = (unsigned)tid * 16u, ldsw = (unsigned)wid * 1024u;
    const int aoff = wr * 8192 + lane * 16, boff = wc * 4096 + lane * 16;
#define SA(b, h) (((b) * 2 + (h)) * HALFB)
#define SB(b, h) ((4 + (b) * 2 + (h)) * HALFB)
#define STAGE(bufoff, gbase) do { _Pragma("unroll") for (int _i = 0; _i < 2; ++_i) \
        __builtin_amdgcn_global_load_lds((const unsigned*)((gbase) + voff + _i * 8192), (LAS unsigned*)(lds + (bufoff) + ldsw + _i * 8192), 16, 0, AUX_AGT); } while (0)
#define LDA(dst, b, h) do { _Pragma("unroll") for (int m = 0; m < 4; ++m) _Pragma("unroll") for (int k = 0; k < 2; ++k) dst[m][k] = *(const LAS bf16x8*)(lds + SA(b, h) + aoff + m * 2048 + k * 1024); } while (0)
#define LDB(dst, b, h) do { _Pragma("unroll") for (int n = 0; n < 2; ++n) _Pragma("unroll") for (int k = 0; k < 2; ++k) dst[n][k] = *(const LAS bf16x8*)(lds + SB(b, h) + boff + n * 2048 + k * 1024); } while (0)
#define MMA(ai, bj, At, Bt) do { __builtin_amdgcn_s_setprio(1); _Pragma("unroll") for (int m = 0; m < 4; ++m) _Pragma("unroll") for (int n = 0; n < 2; ++n) _Pragma("unroll") for (int k = 0; k < 2; ++k) \
        acc[ai][bj][m][n] = __builtin_amdgcn_mfma_f32_16x16x32_bf16(Bt[n][k], At[m][k], acc[ai][bj][m][n], 0, 0, 0); __builtin_amdgcn_s_setprio(0); } while (0)
    f32x4 acc[2][2][4][2];
#pragma unroll
    for (int a = 0; a < 2; ++a)
#pragma unroll
        for (int b = 0; b < 2; ++b)
#pragma unroll
            for (int m = 0; m < 4; ++m)
#pragma unroll
                for (int n = 0; n < 2; ++n) acc[a][b][m][n] = (f32x4){0.f, 0.f, 0.f, 0.f};
    bf16x8 At[4][2], B0[2][2], B1[2][2];
    constexpr int NT = 64;
    const char* a1;
    {
        const unsigned s0 = next_step(0);
        const char *a0 = A_OF(s0), *b0 = B_OF(s0);
        STAGE(SB(0, 0), b0); STAGE(SA(0, 0), a0); STAGE(SB(0, 1), b0 + HALFB); STAGE(SA(0, 1), a0 + HALFB);
        if (wr == 1) BAR;
        WAIT_V(4); BAR;
        const unsigned s1 = next_step(1);
        a1 = A_OF(s1);
        const char* b1 = B_OF(s1);
        STAGE(SB(1, 0), b1); STAGE(SA(1, 0), a1); STAGE(SB(1, 1), b1 + HALFB);
        WAIT_V(6); BAR;
    }
    STAMP(1);
    for (int t = 0; t < NT - 2; t += 2) {
        LDB(B0, 0, 0); SCHED; LDA(At, 0, 0); STAGE(SA(1, 1), a1 + HALFB);
        WAIT_L(8); BAR; WAIT_L(0); MMA(0, 0, At, B0); BAR; SCHED;
        const unsigned s2 = next_step(t + 2);
        const char *a2 = A_OF(s2), *b2 = B_OF(s2);
        LDB(B1, 0, 1); STAGE(SB(0, 0), b2);
        BAR; WAIT_L(0); MMA(0, 1, At, B1); BAR;
        LDA(At, 0, 1); STAGE(SA(0, 0), a2);
        BAR; WAIT_L(0); MMA(1, 0, At, B0); BAR; SCHED;
        STAGE(SB(0, 1), b2 + HALFB);
        POLL_WAIT_V6(t + 3); BAR; MMA(1, 1, At, B1); BAR;
        LDB(B0, 1, 0); SCHED; LDA(At, 1, 0); STAGE(SA(0, 1), a2 + HALFB);
        WAIT_L(8); BAR; WAIT_L(0); MMA(0, 0, At, B0); BAR; SCHED;
        const unsigned s3 = next_step(t + 3);
        a1 = A_OF(s3);
        const char* b3 = B_OF(s3);
        LDB(B1, 1, 1); STAGE(SB(1, 0), b3);
        BAR; WAIT_L(0); MMA(0, 1, At, B1); BAR;
        LDA(At, 1, 1); STAGE(SA(1, 0), a1);
        BAR; WAIT_L(0); MMA(1, 0, At, B0); BAR; SCHED;
        STAGE(SB(1, 1), b3 + HALFB);
        POLL_WAIT_V6(t + 4); BAR; MMA(1, 1, At, B1); BAR;
    }
    STAMP(2);
    {
        LDB(B0, 0, 0); LDA(At, 0, 0); STAGE(SA(1, 1), a1 + HALFB);
        BAR; WAIT_L(0); MMA(0, 0, At, B0); BAR;
        LDB(B1, 0, 1); BAR; WAIT_L(0); MMA(0, 1, At, B1); BAR;
        LDA(At, 0, 1); WAIT_V(0); BAR; WAIT_L(0); MMA(1, 0, At, B0); MMA(1, 1, At, B1); BAR;
    }
    {
        LDB(B0, 1, 0); LDA(At, 1, 0); BAR; WAIT_L(0); MMA(0, 0, At, B0); BAR;
        LDB(B1, 1, 1); BAR; WAIT_L(0); MMA(0, 1, At, B1); BAR;
        LDA(At, 1, 1); BAR; WAIT_L(0); MMA(1, 0, At, B0); MMA(1, 1, At, B1); BAR;
    }
    if (wr == 0) BAR;
    STAMP(3);

    int mx = 0;
#pragma unroll
    for (int i = 0; i < 32; ++i) {
        const i32x4 v = __builtin_bit_cast(i32x4, acc[i >> 4][(i >> 1) & 1][(i >> 2) & 3][i & 1]);
        mx = max(max(mx, v[0]), v[1]); mx = max(max(mx, v[2]), v[3]);
    }
    {
        const unsigned wm = wave_max_u32((unsigned)mx);
        if (lane == 63) __hip_atomic_fetch_max(lctl + 8, wm, __ATOMIC_RELAXED, __HIP_MEMORY_SCOPE_WORKGROUP);
    }
    __syncthreads();
    if (tid < NR) {
        char* pw = (char*)(size_t)*(LAS u64*)(lctl + 16 + 2 * tid);
        __hip_atomic_store((gu64*)(size_t)(pw + OFF_CTL + CTL_AMAX + (rank * 128 + gi) * 8), ((u64)call << 32) | lctl[8], __ATOMIC_RELAXED, __HIP_MEMORY_SCOPE_SYSTEM);
    }
    STAMP(6);
    const int owner = ti >> 1;
    const bool local = owner == rank;
    char* optr = p.out[owner];
    char* oflg = p.ws[owner] + OFF_CTL + CTL_OFLG + (rank * 16 + (ti & 1) * 8 + tj) * 128;
    asm volatile("" : "+s"(optr), "+s"(oflg));
    const __amdgpu_buffer_rsrc_t ro = mk_rsrc(optr, OUT_BYTES), rx = mk_rsrc(ws + OFF_OEX, 16u << 20);
    const unsigned slot = (unsigned)((owner * 16 + (ti & 1) * 8 + tj) * 8 + wid) * OSLOT, area = lds0 + wid * 16384;
#define ACC(i) acc[(i) >> 4][((i) >> 1) & 1][((i) >> 2) & 3][(i) & 1]
    unsigned addr[33], slo[32], snx[32], len = 0, tmo = 0;
    u32x4 g0 = {0u, 0u, 0u, 0u};
    if (local) {
#pragma unroll
        for (int i = 0; i < 32; ++i) {
            i32x4 iv = __builtin_bit_cast(i32x4, ACC(i));
#pragma unroll
            for (int e = 0; e < 4; ++e) iv[e] = max(iv[e], 0);
            asm volatile("" : "+v"(iv));
            ACC(i) = __builtin_bit_cast(f32x4, iv);
        }
    } else {
        u32x4 bm;
#pragma unroll
        for (int k = 0; k < 4; ++k) {
            unsigned w = 0;
#pragma unroll
            for (int i = 31; i >= 0; --i) w = __builtin_amdgcn_alignbit(w, __float_as_uint(ACC(k * 8 + (i >> 2))[i & 3]), 31);
            bm[k] = ~w;
        }
        st128<AUX_SYS>(bm, rx, lane * 16, slot);
        unsigned a;
        const unsigned spare = area + 8192u + lane * 4u;
        len = stream_base(bm, area, a);
#pragma unroll
        for (int i = 0; i < 32; ++i) {
            const unsigned nib = (bm[i >> 3] >> (4 * (i & 7))) & 15u;
            const u32x2 s2 = *(const LAS u32x2*)(size_t)(((nib << 5) | (lctl_addr + 1024u)) | ((a << 3) & 0x18u));
            addr[i] = nib ? a & ~3u : spare; slo[i] = s2.x; snx[i] = s2.y; a += __builtin_popcount(nib);
            if (i == BM_AT) {
                WAIT_V(0);
                BAR;
                if (tid == 0) st_sys(oflg, 2u * call - 1u);
                g0 = __builtin_amdgcn_raw_buffer_load_b128(rc, tid * 16, CTL_AMAX, AUX_SYS);
                tmo = ld_agt(ctl + CTL_TMO);
            }
        }
        addr[32] = (a & 3u) ? a & ~3u : spare;
#pragma unroll
        for (int i = 0; i < 32; ++i) asm volatile("" : "+v"(addr[i]), "+v"(slo[i]), "+v"(snx[i]));
    }
    STAMP(7);
    unsigned gmx = max(g0.x, g0.z);
    if (!__all(g0.y >= call && g0.w >= call)) {
        const u64 t0 = __builtin_amdgcn_s_memrealtime(); unsigned spins = 0;
        for (;;) {
            const u32x4 g = __builtin_amdgcn_raw_buffer_load_b128(rc, tid * 16, CTL_AMAX, AUX_SYS);
            tmo = ld_agt(ctl + CTL_TMO);
            gmx = max(g.x, g.z);
            if (__all(g.y >= call && g.w >= call)) break;
            if (give_up(ctl, call, t0, spins)) { tmo = call; break; }
            __builtin_amdgcn_s_sleep(1);
        }
    }
    if (tmo == call) gmx = 0xffffffffu;
    gmx = wave_max_u32(gmx);
    if (lane == 63) __hip_atomic_fetch_max(lctl + 9, gmx, __ATOMIC_RELAXED, __HIP_MEMORY_SCOPE_WORKGROUP);
    __syncthreads();
    STAMP(4);
    const float scale = __uint_as_float(lctl[9]) / 448.0f, inv = 1.0f / scale;
    const f32x2 inv2 = {inv, inv};
    auto quant4 = [&](int i) -> unsigned {
        const f32x4 v = ACC(i);
        const f32x2 lo = (f32x2){v[0], v[1]} * inv2, hi = (f32x2){v[2], v[3]} * inv2;
        const int r = __builtin_amdgcn_cvt_pk_fp8_f32(lo.x, lo.y, __float_as_int(lo.x), false);
        return (unsigned)__builtin_amdgcn_cvt_pk_fp8_f32(hi.x, hi.y, r, true);
    };
    if (local) {
        const unsigned obase = (unsigned)((ti & 1) * 256 + wr * 64 + (lane & 15)) * GN + (unsigned)(rank * NS + tj * 256 + wc * 64 + (lane >> 4) * 16);
#pragma unroll
        for (int blk = 0; blk < 8; ++blk)
            st128<AUX_SYS>((u32x4){quant4(blk * 4), quant4(blk * 4 + 1), quant4(blk * 4 + 2), quant4(blk * 4 + 3)}, ro, obase,
                                                   (unsigned)((blk >> 2) * 128 + (blk & 3) * 16) * GN);
    } else {
        unsigned carry = 0;
#pragma unroll
        for (int i = 0; i < 32; ++i) {
            const unsigned r = quant4(i);
            *(volatile LAS unsigned*)(size_t)addr[i] = __builtin_amdgcn_perm(carry, r, slo[i]);
            carry = __builtin_amdgcn_perm(carry, r, snx[i]);
        }
        *(volatile LAS unsigned*)(size_t)addr[32] = carry;
        for (unsigned k = 0; k * 1024u < len; k += 2) {
            const u32x4 lo = *(volatile LAS u32x4*)(size_t)(area + k * 1024u + lane * 16u), hi = *(volatile LAS u32x4*)(size_t)(area + (k + 1) * 1024u + lane * 16u);
            unsigned d[7] = {lo.x, lo.y, lo.z, lo.w, hi.x, hi.y, hi.z};
#pragma unroll
            for (int b = 0; b < 7; ++b) d[b] = (d[b] & 0x7f7f7f7fu) | ((hi.w << (7 - b)) & 0x80808080u);
            st128<AUX_SYS>((u32x4){d[0], d[1], d[2], d[3]}, rx, k * 1024u + lane * 16u < len ? lane * 16u : OOB, slot + 1024u + k * 1024u);
            st96<AUX_SYS>((u32x3){d[4], d[5], d[6]}, rx, (k + 1) * 1024u + lane * 16u < len ? lane * 12u : OOB, slot + 2048u + k * 1024u);
        }
        STAMP(8);
        WAIT_V(0);
        BAR;
        if (tid == 0) st_sys(oflg, 2u * call);
    }
    STAMP(5);
}

}

extern "C" void kernel_launch(int rank, int nranks, ncclComm_t comm,
                              hipStream_t stream,
                              void* const* d_in, const int* in_sizes, int n_in,
                              void* d_out, int out_size,
                              const int* in_dtypes, int out_dtype,
                              void* const* d_out_all,
                              void* const* d_ws_all, size_t ws_bytes) {
    (void)comm; (void)in_dtypes; (void)out_dtype; (void)d_out;
    if (nranks != NR || (unsigned)rank >= (unsigned)NR || n_in < 2 || ws_bytes < WS_NEED || in_sizes[0] != GM * KS ||
        in_sizes[1] != KS * GN || (unsigned)out_size != OUT_BYTES) {
        fprintf(stderr, "kernel_launch: unexpected problem shape\n");
        return;
    }
    HIP_CHECK(hipFuncSetAttribute((const void*)fused_gemm_quant, hipFuncAttributeMaxDynamicSharedMemorySize, LDS_BYTES));
    Params p;
    p.x = (const char*)d_in[0];
    p.w = (const char*)d_in[1];
    for (int r = 0; r < NR; ++r) { p.ws[r] = (char*)d_ws_all[r]; p.out[r] = (char*)d_out_all[r]; }
    p.rank = rank;
    hipLaunchKernelGGL(fused_gemm_quant, dim3(N_WG), dim3(512), LDS_BYTES, stream, p);
    HIP_CHECK(hipGetLastError());
}
```

```cpp
#include <hip/hip_runtime.h>
#include <rccl/rccl.h>
#include <cstdio>
#include <cstdint>

#define HIP_CHECK(expr) do { hipError_t _e = (expr); if (_e != hipSuccess) { \
    fprintf(stderr, "HIP error %s at %s:%d\n", hipGetErrorString(_e), __FILE__, __LINE__); \
    return; } } while (0)

namespace {

#define LAS __attribute__((address_space(3)))
typedef short bf16x8 __attribute__((ext_vector_type(8)));
typedef short s16x4 __attribute__((ext_vector_type(4)));
typedef float f32x4 __attribute__((ext_vector_type(4)));
typedef float f32x2 __attribute__((ext_vector_type(2)));
typedef int i32x4 __attribute__((ext_vector_type(4)));
typedef unsigned u32x4 __attribute__((ext_vector_type(4)));
typedef unsigned u32x3 __attribute__((ext_vector_type(3)));
typedef unsigned u32x2 __attribute__((ext_vector_type(2)));
typedef unsigned long long u64;
typedef __attribute__((address_space(1))) unsigned gu32;
typedef __attribute__((address_space(1))) u64 gu64;

constexpr int NR = 8;
constexpr int GM = 4096, GN = 16384;
constexpr int KS = 512, NS = 2048, MS = 512;
constexpr unsigned PIECE = 32768;
constexpr unsigned HALFB = 16384;
constexpr unsigned OSLOT = 16384;
constexpr unsigned OFF_X = 0;
constexpr unsigned OFF_WIN = 32u << 20;
constexpr unsigned OFF_WEX = 48u << 20;
constexpr unsigned OFF_OEX = 64u << 20;
constexpr unsigned OFF_CTL = 80u << 20;
constexpr unsigned CTL_SEQ = 0;
constexpr unsigned CTL_OFLG = 16384;
constexpr unsigned CTL_TMO = 32768;
constexpr unsigned CTL_EXPF = 36864;
constexpr unsigned CTL_XCNT = 45056;
constexpr unsigned CTL_WCNT = 49152;
constexpr unsigned CTL_AMAX = 53248;
constexpr unsigned CTL_TICK = 63488;
constexpr unsigned CTL_DBG = 65536;
constexpr size_t WS_NEED = (size_t)OFF_CTL + (1u << 20);
constexpr unsigned OUT_BYTES = (unsigned)MS * GN;
constexpr unsigned OOB = 0x80000000u;

constexpr int N_PULL = 56, N_GEMM = 128, N_EXP = 72, N_WG = N_EXP + N_PULL + N_GEMM;
constexpr int LAREA = 9216;
constexpr int LDS_CTL = 16 * LAREA, LDS_BYTES = LDS_CTL + 3072;
static_assert(LDS_CTL >= 131072, "the GEMM staging buffers live below the control words");
constexpr u64 DEADLINE = 50000000ull;
constexpr int N_PUSH = 2;
constexpr int EARLY_FROM = 57;
constexpr unsigned MERGE_B = 512;

struct Params {
    const char* x;
    const char* w;
    char* ws[NR];
    char* out[NR];
    int rank;
};

__device__ __forceinline__ __amdgpu_buffer_rsrc_t mk_rsrc(const void* p, unsigned bytes) {
    return __builtin_amdgcn_make_buffer_rsrc(const_cast<void*>(p), 0, bytes, 0x00020000);
}
constexpr int AUX_SYS = 17;
constexpr int AUX_AGT = 16;

template <int AUX> __device__ __forceinline__ void st128(u32x4 v, __amdgpu_buffer_rsrc_t r, unsigned voff, unsigned soff) { __builtin_amdgcn_raw_buffer_store_b128(v, r, voff + soff, 0, AUX); }
template <int AUX> __device__ __forceinline__ void st96(u32x3 v, __amdgpu_buffer_rsrc_t r, unsigned voff, unsigned soff) { __builtin_amdgcn_raw_buffer_store_b96(v, r, voff + soff, 0, AUX); }
__device__ __forceinline__ unsigned ld_sys(const void* p) { return __hip_atomic_load((const gu32*)(size_t)p, __ATOMIC_RELAXED, __HIP_MEMORY_SCOPE_SYSTEM); }
__device__ __forceinline__ unsigned ld_agt(const void* p) { return __hip_atomic_load((const gu32*)(size_t)p, __ATOMIC_RELAXED, __HIP_MEMORY_SCOPE_AGENT); }
__device__ __forceinline__ void st_sys(void* p, unsigned v) { __hip_atomic_store((gu32*)(size_t)p, v, __ATOMIC_RELAXED, __HIP_MEMORY_SCOPE_SYSTEM); }
#define WAIT_V(n) asm volatile("s_waitcnt vmcnt(" #n ")" ::: "memory")
#define WAIT_L(n) asm volatile("s_waitcnt lgkmcnt(" #n ")" ::: "memory")
#define BAR __builtin_amdgcn_s_barrier()
#define BARF do { asm volatile("" ::: "memory"); __builtin_amdgcn_s_barrier(); asm volatile("" ::: "memory"); } while (0)
#define SCHED __builtin_amdgcn_sched_barrier(0)

__device__ __forceinline__ bool give_up(char* ctl, unsigned call, u64 t0, unsigned& spins) {
    if ((++spins & 31u) != 0u) return false;
    const unsigned tm = __builtin_amdgcn_readfirstlane(ld_agt(ctl + CTL_TMO));
    if (tm == call) return true;
    if (__builtin_amdgcn_s_memrealtime() - t0 > DEADLINE) {
        __hip_atomic_store((gu32*)(size_t)(ctl + CTL_TMO), call, __ATOMIC_RELAXED, __HIP_MEMORY_SCOPE_AGENT);
        return true;
    }
    return false;
}
#define DPP(v, ctrl, rows) ((unsigned)__builtin_amdgcn_update_dpp(0, (int)(v), ctrl, rows, 0xf, false))
__device__ __forceinline__ unsigned wave_max_u32(unsigned v) {
    v = max(v, DPP(v, 0x111, 0xf)); v = max(v, DPP(v, 0x112, 0xf)); v = max(v, DPP(v, 0x114, 0xf)); v = max(v, DPP(v, 0x118, 0xf));
    v = max(v, DPP(v, 0x142, 0xa)); v = max(v, DPP(v, 0x143, 0xc));
    return v;
}
__device__ __forceinline__ unsigned wave_scan_u32(unsigned v) {
    v += DPP(v, 0x111, 0xf); v += DPP(v, 0x112, 0xf); v += DPP(v, 0x114, 0xf); v += DPP(v, 0x118, 0xf);
    v += DPP(v, 0x142, 0xa); v += DPP(v, 0x143, 0xc);
    return v;
}
__device__ __forceinline__ unsigned stream_base(const u32x4 bm, unsigned area, unsigned& a) {
    const unsigned len = __builtin_popcount(bm.x) + __builtin_popcount(bm.y) + __builtin_popcount(bm.z) + __builtin_popcount(bm.w);
    const unsigned reg = (len + 3u) & ~3u, inc = wave_scan_u32(reg);
    a = area + inc - reg;
    return (unsigned)__builtin_amdgcn_readlane((int)inc, 63);
}

#define STAMP(k) do { } while (0)

__device__ __forceinline__ int rot_of(int j) { return (j * 16 + 3) / 7; }

__global__ void __launch_bounds__(512, 2) fused_gemm_quant(Params p) {
    extern __shared__ __attribute__((aligned(16))) unsigned char smem[];
    LAS unsigned char* lds = (LAS unsigned char*)smem;
    const int tid = threadIdx.x, lane = tid & 63;
    const int wid = __builtin_amdgcn_readfirstlane(tid >> 6);
    const int rank = p.rank, bid = blockIdx.x;
    char* const ws = p.ws[rank];
    char* const ctl = ws + OFF_CTL;
    STAMP(0);
    const __amdgpu_buffer_rsrc_t rc = mk_rsrc(ctl, 65536);
    const unsigned call_v = (unsigned)__builtin_amdgcn_raw_ptr_buffer_atomic_add_i32(1, rc, tid == 0 ? 0u : OOB, CTL_SEQ + bid * 64, 0) + 1u;
    LAS unsigned* lctl = (LAS unsigned*)(lds + LDS_CTL);
    const unsigned lds0 = (unsigned)(size_t)lds, lctl_addr = lds0 + LDS_CTL;

    if (bid < N_EXP) {
        const __amdgpu_buffer_rsrc_t rx = mk_rsrc(p.x, (unsigned)GM * KS * 2), rw = mk_rsrc(p.w, (unsigned)KS * GN * 2);
        const __amdgpu_buffer_rsrc_t rd = mk_rsrc(ws, OFF_CTL);
        constexpr int PITCH = 528;
        for (int q = bid; q < 640; q += N_EXP) {
            const int c = q / 80, idx = q % 80;
            const bool isx = idx >= 56 && idx < 72, push = q < 7 * N_PUSH;
            int dst = rank, pj = 0;
            if (isx) {
                pj = idx - 56;
                u32x4 v[4];
                const unsigned src = (unsigned)((pj * 256 + wid * 32 + (lane & 15)) * (KS * 2) + c * 128 + (lane >> 4) * 16);
#pragma unroll
                for (int u = 0; u < 4; ++u) v[u] = __builtin_amdgcn_raw_buffer_load_b128(rx, src + (u >> 1) * 16 * (KS * 2) + (u & 1) * 64, 0, 0);
                const unsigned d0 = OFF_X + (unsigned)((rank * 8 + c) * 16 + pj) * PIECE + wid * 4096 + lane * 16;
#pragma unroll
                for (int u = 0; u < 4; ++u) st128<AUX_SYS>(v[u], rd, d0 + u * 1024, 0);
            } else {
                if (idx < 56) { dst = (rank + 1 + idx % 7) & 7; pj = idx / 7; } else { pj = idx - 72; }
                u32x4 v[4];
                const unsigned src = (unsigned)(c * 64 + (tid >> 5)) * (GN * 2) + (unsigned)(dst * NS + pj * 256) * 2 + (tid & 31) * 16;
#pragma unroll
                for (int u = 0; u < 4; ++u) v[u] = __builtin_amdgcn_raw_buffer_load_b128(rw, src + u * 16 * (GN * 2), 0, 0);
#pragma unroll
                for (int u = 0; u < 4; ++u) *(LAS u32x4*)(lds + ((tid >> 5) + 16 * u) * PITCH + (tid & 31) * 16) = v[u];
                __syncthreads();
                const int bj = wid >> 2, wc = wid & 3;
                const int g = lane >> 4, qq = (lane >> 2) & 3, pp = lane & 3;
                const __amdgpu_buffer_rsrc_t rw2 = mk_rsrc(push ? p.ws[dst] : ws, OFF_CTL);
                const unsigned d0 = (push ? OFF_WIN + (unsigned)(rank * 64 + pj) * PIECE : (dst == rank ? OFF_WIN : OFF_WEX) + (unsigned)((dst * 8 + c) * 8 + pj) * PIECE) + wid * 4096 + lane * 16;
#pragma unroll
                for (int u = 0; u < 4; ++u) {
                    const int n = u >> 1, kf = u & 1;
                    const int a = (kf * 32 + g * 8 + qq) * PITCH + (wc * 64 + pp * 16 + (bj * 2 + n) * 4) * 2;
                    const s16x4 lo = __builtin_amdgcn_ds_read_tr16_b64_v4i16((LAS s16x4*)(lds + a));
                    const s16x4 hi = __builtin_amdgcn_ds_read_tr16_b64_v4i16((LAS s16x4*)(lds + a + 4 * PITCH));
                    const u32x2 l2 = __builtin_bit_cast(u32x2, lo), h2 = __builtin_bit_cast(u32x2, hi);
                    const u32x4 o = {l2.x, l2.y, h2.x, h2.y};
                    st128<AUX_SYS>(o, rw2, d0 + u * 1024, 0);
                }
            }
            WAIT_V(0);
            __syncthreads();
            if (wid == 0) {
                const unsigned call = __builtin_amdgcn_readfirstlane(call_v);
                if (isx) {
                    if (lane < NR) {
                        if (lane == rank) st_sys(ctl + CTL_XCNT + (pj * 64 + rank * 8 + c) * 4, call * 32u);
                        else st_sys(p.ws[lane] + OFF_CTL + CTL_EXPF + (rank * 256 + c * 24 + 8 + pj) * 4, call);
                    }
                } else if (lane == 0) {
                    if (dst == rank || push) st_sys(p.ws[dst] + OFF_CTL + CTL_WCNT + (pj * 64 + rank * 8 + c) * 4, call * 32u);
                    else st_sys(p.ws[dst] + OFF_CTL + CTL_EXPF + (rank * 256 + c * 24 + pj) * 4, call);
                }
            }
            if (q == bid) STAMP(1);
        }
        STAMP(2);
        return;
    }

    if (tid == 0) { lctl[0] = call_v; lctl[6] = 0u; lctl[8] = 0u; lctl[9] = 0u; }
    if (tid >= 64 && tid < 64 + NR) *(LAS u64*)(lctl + 16 + 2 * (tid - 64)) = (u64)(size_t)p.ws[tid - 64];
    if (tid >= 128 && tid < 192) {
        const unsigned m = (tid - 128) >> 2, off = tid & 3;
        u64 cs = 0x0c0c0c0c0c0c0c0cull; unsigned es = 0x0c0c0c0cu, k = off;
        for (unsigned e = 0; e < off; ++e) cs = (cs & ~(0xffull << (8 * e))) | ((u64)(4 + e) << (8 * e));
        for (unsigned e = 0; e < 4; ++e)
            if ((m >> e) & 1u) { cs = (cs & ~(0xffull << (8 * k))) | ((u64)e << (8 * k)); es = (es & ~(0xffu << (8 * e))) | (k << (8 * e)); ++k; }
        *(LAS u32x2*)(lctl + 256 + 2 * (tid - 128)) = (u32x2){(unsigned)cs, k >= 4 ? (unsigned)(cs >> 32) : (unsigned)cs};
        lctl[64 + tid - 128] = es;
    }
    __syncthreads();
    const unsigned call = __builtin_amdgcn_readfirstlane(lctl[0]);

    if (bid < N_EXP + N_PULL) {
        const int q = bid - N_EXP, j = q % 7;
        const int src = (rank + 1 + j) & 7, rot = rot_of(j);
        const __amdgpu_buffer_rsrc_t rs = mk_rsrc(p.ws[src], OFF_CTL), rd = mk_rsrc(ws, OFF_CTL);
        const char* ef = ctl + CTL_EXPF + src * 1024 + lane * 4;
        u64 m0 = 0, m1 = 0, m2 = 0;
        const unsigned aoff = lane == 0 ? 0u : OOB;
        constexpr unsigned NU = (192 - N_PUSH) * 16;
        const unsigned tbase = (call - 1u) * NU - 64u, toff = CTL_TICK + j * 128;
        unsigned tk = (q / 7) * 8 + wid;
        unsigned cnt1 = 0, off1 = OOB;
        while (tk < NU) {
            const int pp = tk / 16 + N_PUSH, c = pp / 24, r = pp % 24, sc = src * 8 + c;
            int id; unsigned so, dof, cnt;
            if (r < 8) { id = c * 24 + r; so = OFF_WEX + (unsigned)((rank * 8 + c) * 8 + r) * PIECE; dof = OFF_WIN + (unsigned)(sc * 8 + r) * PIECE; cnt = CTL_WCNT + (r * 64 + sc) * 4; }
            else { const int i = (rot + r - 8) & 15; id = c * 24 + 8 + i; so = OFF_X + (unsigned)(sc * 16 + i) * PIECE; dof = so; cnt = CTL_XCNT + (i * 64 + sc) * 4; }
            so += (tk % 16) * 2048u; dof += (tk % 16) * 2048u;
            const u64 mm = id < 64 ? m0 : (id < 128 ? m1 : m2);
            if (!((mm >> (id & 63)) & 1)) {
                const u64 t0 = __builtin_amdgcn_s_memrealtime(); unsigned spins = 0;
                for (;;) {
                    const unsigned f0 = ld_sys(ef), f1 = ld_sys(ef + 256), f2 = ld_sys(ef + 512);
                    m0 = __ballot(f0 >= call); m1 = __ballot(f1 >= call); m2 = __ballot(f2 >= call);
                    const u64 mn = id < 64 ? m0 : (id < 128 ? m1 : m2);
                    if ((mn >> (id & 63)) & 1) break;
                    if (give_up(ctl, call, t0, spins)) { m0 = m1 = m2 = ~0ull; break; }
                    __builtin_amdgcn_s_sleep(2);
                }
                if (off1 == OOB) STAMP(1);
            }
            u32x4 v = __builtin_amdgcn_raw_buffer_load_b128(rs, lane * 16, so, AUX_SYS);
            WAIT_V(1);
            __builtin_amdgcn_raw_ptr_buffer_atomic_add_i32(2, rc, off1, cnt1, 0);
            WAIT_V(1);
            st128<AUX_AGT>(v, rd, lane * 16, dof);
            v = __builtin_amdgcn_raw_buffer_load_b128(rs, lane * 16, so + 1024, AUX_SYS);
            const int tnext = __builtin_amdgcn_raw_ptr_buffer_atomic_add_i32(1, rc, aoff, toff, 0);
            WAIT_V(0);
            st128<AUX_AGT>(v, rd, lane * 16, dof + 1024);
            cnt1 = cnt; off1 = aoff;
            tk = (unsigned)__builtin_amdgcn_readfirstlane(tnext) - tbase;
        }
        WAIT_V(0);
        __builtin_amdgcn_raw_ptr_buffer_atomic_add_i32(2, rc, off1, cnt1, 0);
        STAMP(2);

        const int g = (q / 7) * 8 + wid, tA = g & 15, tB = tA ^ 8, w2 = g >> 4;
        const __amdgpu_buffer_rsrc_t ro = mk_rsrc(p.out[rank], OUT_BYTES);
        const char* ofA = ctl + CTL_OFLG + (src * 16 + tA) * 128;
        const char* ofB = ctl + CTL_OFLG + (src * 16 + tB) * 128;
        const unsigned slA = OFF_OEX + (unsigned)((rank * 16 + tA) * 8 + w2) * OSLOT, slB = OFF_OEX + (unsigned)((rank * 16 + tB) * 8 + w2 + 4) * OSLOT;
        const unsigned arA = lds0 + wid * 2 * LAREA, arB = arA + LAREA;
        const unsigned obase = (unsigned)(lane & 15) * GN + (unsigned)(src * NS + (tA & 7) * 256 + w2 * 64 + (lane >> 4) * 16);
        const unsigned rowA = (tA >> 3) * 256, rowB = (tB >> 3) * 256 + 64;
        unsigned adA[32], seA[32], adB[32], seB[32], pA, pB, lenA, lenB;
        u32x4 bmA, bmB;
#define PLAN(X, i0, i1) do { _Pragma("unroll") for (int i = i0; i < i1; ++i) { const unsigned nib = (bm##X[i >> 3] >> (4 * (i & 7))) & 15u; \
                ad##X[i] = p##X & ~3u; se##X[i] = lctl[64 + nib * 4 + (p##X & 3u)]; p##X += __builtin_popcount(nib); } \
            _Pragma("unroll") for (int i = i0; i < i1; ++i) asm volatile("" : "+v"(ad##X[i]), "+v"(se##X[i])); } while (0)
        bool ok = true;
        unsigned fl = 0;
        auto wait2 = [&](unsigned need, unsigned who) {
            const u64 t0 = __builtin_amdgcn_s_memrealtime(); unsigned spins = 0;
            for (;;) {
                fl = ld_sys(lane & 1 ? ofB : ofA);
                if ((unsigned)__ballot(fl >= need) & who) break;
                if (give_up(ctl, call, t0, spins)) { ok = false; fl = ~0u; break; }
                __builtin_amdgcn_s_sleep(8);
            }
        };
        constexpr unsigned whoA = 1, whoB = 2;
        wait2(2u * call - 1u, 3);
        const bool a_first = (unsigned)__builtin_amdgcn_readlane((int)fl, 0) >= 2u * call - 1u;
        bool both = false;
        STAMP(3);
#define WSTAMP(k) do { } while (0)
#define LOAD_EVEN(X, k) __builtin_amdgcn_raw_buffer_load_b128(rs, (k) * 1024u + lane * 16u < len##X ? lane * 16u : OOB, sl##X + 1024u + (k) * 1024u, AUX_SYS)
#define LOAD_ODD(X, k) __builtin_amdgcn_raw_buffer_load_b96(rs, (k) * 1024u + lane * 16u < len##X ? lane * 12u : OOB, sl##X + 1024u + (k) * 1024u, AUX_SYS)
#define PUT_EVEN(X, k, v) do { d7 = ((v.x >> 7) & 0x01010101u) | ((v.y >> 6) & 0x02020202u) | ((v.z >> 5) & 0x04040404u) | ((v.w >> 4) & 0x08080808u); \
            *(LAS u32x4*)(size_t)(ar##X + (k) * 1024u + lane * 16u) = v & 0x7f7f7f7fu; } while (0)
#define PUT_ODD(X, k, w) do { d7 |= ((w.x >> 3) & 0x10101010u) | ((w.y >> 2) & 0x20202020u) | ((w.z >> 1) & 0x40404040u); \
            *(LAS u32x4*)(size_t)(ar##X + (k) * 1024u + lane * 16u) = (u32x4){w.x & 0x7f7f7f7fu, w.y & 0x7f7f7f7fu, w.z & 0x7f7f7f7fu, d7}; } while (0)
#define REST(X, other) do { bool have = false; u32x4 v; \
            for (unsigned k = 2; k * 1024u < len##X; k += 2) { \
                if (!have) v = LOAD_EVEN(X, k); \
                PUT_EVEN(X, k, v); \
                if ((k + 1) * 1024u < len##X) { \
                    const u32x3 w = LOAD_ODD(X, k + 1); \
                    have = len##X <= (k + 2) * 1024u + MERGE_B; \
                    if (have) v = LOAD_EVEN(X, k + 2); \
                    const unsigned fo = __builtin_amdgcn_raw_buffer_load_b32(rc, both ? OOB : aoff, CTL_OFLG + (src * 16 + (other)) * 128, AUX_SYS); \
                    PUT_ODD(X, k + 1, w); \
                    both = both || (unsigned)__builtin_amdgcn_readfirstlane(fo) >= 2u * call; \
                } } } while (0)
#define EXPAND(X) do { unsigned hi[32], lo = *(const LAS unsigned*)(size_t)ad##X[0]; \
            _Pragma("unroll") for (int i = 0; i < 32; ++i) hi[i] = *(const LAS unsigned*)(size_t)(ad##X[i] + 4u); \
            if (!ok) { _Pragma("unroll") for (int i = 0; i < 32; ++i) se##X[i] = ~0u; }        \
            _Pragma("unroll") for (int blk = 0; blk < 8; ++blk) { u32x4 o; \
                _Pragma("unroll") for (int e = 0; e < 4; ++e) { const int i = blk * 4 + e; \
                    if (i) lo = ad##X[i] != ad##X[i - 1] ? hi[i - 1] : lo; \
                    o[e] = __builtin_amdgcn_perm(hi[i], lo, se##X[i]); } \
                st128<AUX_SYS>(o, ro, obase, (row##X + (blk >> 2) * 128 + (blk & 3) * 16) * GN); } } while (0)
#define RUN(X, Y) do { unsigned d7; \
            bm##X = __builtin_amdgcn_raw_buffer_load_b128(rs, lane * 16, sl##X, AUX_SYS); \
            const bool early = (unsigned)__ballot(fl >= 2u * call - 1u) & who##Y; \
            bm##Y = __builtin_amdgcn_raw_buffer_load_b128(rs, early ? lane * 16u : OOB, sl##Y, AUX_SYS); \
            len##X = stream_base(bm##X, ar##X, p##X); \
            PLAN(X, 0, 32); \
            if (!early) { wait2(2u * call - 1u, who##Y); bm##Y = __builtin_amdgcn_raw_buffer_load_b128(rs, lane * 16, sl##Y, AUX_SYS); } \
            len##Y = stream_base(bm##Y, ar##Y, p##Y); \
            wait2(2u * call, who##X); \
            both = (unsigned)__ballot(fl >= 2u * call) & who##Y; \
            STAMP(5); WSTAMP(0); \
            { const u32x4 v = LOAD_EVEN(X, 0u); PLAN(Y, 0, 16); PUT_EVEN(X, 0u, v); } \
            { const u32x3 w = LOAD_ODD(X, 1u); PLAN(Y, 16, 32); PUT_ODD(X, 1u, w); } \
            REST(X, t##Y); \
            WSTAMP(1); \
            if (!both) { wait2(2u * call, who##Y); both = true; } \
            const u32x4 v0 = LOAD_EVEN(Y, 0u); const u32x3 w1 = LOAD_ODD(Y, 1u);        \
            EXPAND(X); \
            WSTAMP(2); \
            PUT_EVEN(Y, 0u, v0); PUT_ODD(Y, 1u, w1); \
            REST(Y, t##X); \
            WSTAMP(3); \
            EXPAND(Y); } while (0)
        if (a_first) RUN(A, B); else RUN(B, A);
        WSTAMP(4);
        WAIT_V(0);
        WSTAMP(5);
        STAMP(4);
        return;
    }

    const int gi = bid - (N_EXP + N_PULL), xg = gi & 7, ix = gi >> 3;
    const int ti = (xg >> 1) * 4 + (ix & 3), tj = (xg & 1) * 4 + (ix >> 2);
    const int wr = wid >> 2, wc = wid & 3;
    int scv;
    {
        int j0 = 0;
#pragma unroll
        for (int j = 1; j < 7; ++j) if (rot_of(j) <= ti) j0 = j;
        const int t2 = lane - 8, c = t2 / 7, k = t2 - c * 7;
        int j = j0 - k; if (j < 0) j += 7;
        scv = lane < 8 ? rank * 8 + lane : ((rank + 1 + j) & 7) * 8 + c;
    }
    const unsigned target = call * 32u;
    u64 avail = 0, taken = 0;
    unsigned pa = 0, pb = 0;
    const char* const pxa = ctl + CTL_XCNT + (ti * 64 + scv) * 4;
    const char* const pwa = ctl + CTL_WCNT + (tj * 64 + scv) * 4;
#define POLL_WAIT_V6(n) do { if ((n) >= EARLY_FROM && (n) < 64 && wid == 0 && avail == 0) { \
        asm volatile("global_load_dword %0, %2, off sc1\n\tglobal_load_dword %1, %3, off sc1\n\ts_waitcnt vmcnt(8)" : "=&v"(pa), "=&v"(pb) : "v"(pxa), "v"(pwa) : "memory"); \
    } else WAIT_V(6); } while (0)
    auto next_step = [&](int n) -> unsigned {
        if (avail == 0) {
            if (wid == 0) {
                if (n >= EARLY_FROM) {
                    asm volatile("s_waitcnt vmcnt(2)" : "+v"(pa), "+v"(pb) :: "memory");
                    avail = __ballot(pa >= target && pb >= target) & ~taken;
                }
                const u64 t0 = __builtin_amdgcn_s_memrealtime(); unsigned spins = 0;
                while (avail == 0) {
                    const unsigned a = ld_agt(pxa), b = ld_agt(pwa);
                    avail = __ballot(a >= target && b >= target) & ~taken;
                    if (avail != 0) break;
                    if (give_up(ctl, call, t0, spins)) { avail = ~taken; break; }
                    if (n < 56) __builtin_amdgcn_s_sleep(16); else __builtin_amdgcn_s_sleep(1);
                }
                const u32x4 mv = {(unsigned)avail, (unsigned)(avail >> 32), (unsigned)n + 1u, 0u};
                asm volatile("ds_write_b128 %0, %1" :: "v"(lctl_addr + 16), "v"(mv) : "memory");
            } else {
                for (;;) {
                    u32x4 mv;
                    asm volatile("ds_read_b128 %0, %1\n\ts_waitcnt lgkmcnt(0)" : "=v"(mv) : "v"(lctl_addr + 16) : "memory");
                    if ((unsigned)__builtin_amdgcn_readfirstlane(mv.z) == (unsigned)n + 1u) {
                        avail = ((u64)(unsigned)__builtin_amdgcn_readfirstlane(mv.y) << 32) | (unsigned)__builtin_amdgcn_readfirstlane(mv.x);
                        break;
                    }
                    __builtin_amdgcn_s_sleep(1);
                }
            }
        }
        const int bit = __builtin_ctzll(avail);
        avail &= avail - 1; taken |= 1ull << bit;
        return (unsigned)__builtin_amdgcn_readlane(scv, bit);
    };
    const char* const xbase = ws + OFF_X + (size_t)ti * PIECE;
    const char* const wbase = ws + OFF_WIN + (size_t)tj * PIECE;
#define A_OF(sc) (xbase + (size_t)((sc) * (16u * PIECE)))
#define B_OF(sc) (wbase + (size_t)((sc) * (8u * PIECE)))
    const unsigned voff = (unsigned)tid * 16u, ldsw = (unsigned)wid * 1024u;
    const int aoff = wr * 8192 + lane * 16, boff = wc * 4096 + lane * 16;
#define SA(b, h) (((b) * 2 + (h)) * HALFB)
#define SB(b, h) ((4 + (b) * 2 + (h)) * HALFB)
#define STAGE(bufoff, gbase) do { _Pragma("unroll") for (int _i = 0; _i < 2; ++_i) \
        __builtin_amdgcn_global_load_lds((const unsigned*)((gbase) + voff + _i * 8192), (LAS unsigned*)(lds + (bufoff) + ldsw + _i * 8192), 16, 0, AUX_AGT); } while (0)
#define LDA(dst, b, h) do { _Pragma("unroll") for (int m = 0; m < 4; ++m) _Pragma("unroll") for (int k = 0; k < 2; ++k) dst[m][k] = *(const LAS bf16x8*)(lds + SA(b, h) + aoff + m * 2048 + k * 1024); } while (0)
#define LDB(dst, b, h) do { _Pragma("unroll") for (int n = 0; n < 2; ++n) _Pragma("unroll") for (int k = 0; k < 2; ++k) dst[n][k] = *(const LAS bf16x8*)(lds + SB(b, h) + boff + n * 2048 + k * 1024); } while (0)
#define MMA(ai, bj, At, Bt) do { __builtin_amdgcn_s_setprio(1); _Pragma("unroll") for (int m = 0; m < 4; ++m) _Pragma("unroll") for (int n = 0; n < 2; ++n) _Pragma("unroll") for (int k = 0; k < 2; ++k) \
        acc[ai][bj][m][n] = __builtin_amdgcn_mfma_f32_16x16x32_bf16(Bt[n][k], At[m][k], acc[ai][bj][m][n], 0, 0, 0); __builtin_amdgcn_s_setprio(0); } while (0)
    f32x4 acc[2][2][4][2];
#pragma unroll
    for (int a = 0; a < 2; ++a)
#pragma unroll
        for (int b = 0; b < 2; ++b)
#pragma unroll
            for (int m = 0; m < 4; ++m)
#pragma unroll
                for (int n = 0; n < 2; ++n) acc[a][b][m][n] = (f32x4){0.f, 0.f, 0.f, 0.f};
    bf16x8 At[4][2], B0[2][2], B1[2][2];
    constexpr int NT = 64;
    const char* a1;
    {
        const unsigned s0 = next_step(0);
        const char *a0 = A_OF(s0), *b0 = B_OF(s0);
        STAGE(SB(0, 0), b0); STAGE(SA(0, 0), a0); STAGE(SB(0, 1), b0 + HALFB); STAGE(SA(0, 1), a0 + HALFB);
        if (wr == 1) BAR;
        WAIT_V(4); BAR;
        const unsigned s1 = next_step(1);
        a1 = A_OF(s1);
        const char* b1 = B_OF(s1);
        STAGE(SB(1, 0), b1); STAGE(SA(1, 0), a1); STAGE(SB(1, 1), b1 + HALFB);
        WAIT_V(6); BAR;
    }
    STAMP(1);
    for (int t = 0; t < NT - 2; t += 2) {
        LDB(B0, 0, 0); SCHED; LDA(At, 0, 0); STAGE(SA(1, 1), a1 + HALFB);
        WAIT_L(8); BAR; WAIT_L(0); MMA(0, 0, At, B0); BAR; SCHED;
        const unsigned s2 = next_step(t + 2);
        const char *a2 = A_OF(s2), *b2 = B_OF(s2);
        LDB(B1, 0, 1); STAGE(SB(0, 0), b2);
        BAR; WAIT_L(0); MMA(0, 1, At, B1); BAR;
        LDA(At, 0, 1); STAGE(SA(0, 0), a2);
        BAR; WAIT_L(0); MMA(1, 0, At, B0); BAR; SCHED;
        STAGE(SB(0, 1), b2 + HALFB);
        POLL_WAIT_V6(t + 3); BAR; MMA(1, 1, At, B1); BAR;
        LDB(B0, 1, 0); SCHED; LDA(At, 1, 0); STAGE(SA(0, 1), a2 + HALFB);
        WAIT_L(8); BAR; WAIT_L(0); MMA(0, 0, At, B0); BAR; SCHED;
        const unsigned s3 = next_step(t + 3);
        a1 = A_OF(s3);
        const char* b3 = B_OF(s3);
        LDB(B1, 1, 1); STAGE(SB(1, 0), b3);
        BAR; WAIT_L(0); MMA(0, 1, At, B1); BAR;
        LDA(At, 1, 1); STAGE(SA(1, 0), a1);
        BAR; WAIT_L(0); MMA(1, 0, At, B0); BAR; SCHED;
        STAGE(SB(1, 1), b3 + HALFB);
        POLL_WAIT_V6(t + 4); BAR; MMA(1, 1, At, B1); BAR;
    }
    STAMP(2);
    {
        STAGE(SA(1, 1), a1 + HALFB);
        if (wr == 0) BARF;
        asm volatile("" ::: "memory");
        bf16x8 A2[4][2];
#define MMF(ai, bj, At, Bt) do { _Pragma("unroll") for (int m = 0; m < 4; ++m) _Pragma("unroll") for (int n = 0; n < 2; ++n) _Pragma("unroll") for (int k = 0; k < 2; ++k) \
        acc[ai][bj][m][n] = __builtin_amdgcn_mfma_f32_16x16x32_bf16(Bt[n][k], At[m][k], acc[ai][bj][m][n], 0, 0, 0); } while (0)
        LDB(B0, 0, 0); LDA(At, 0, 0); LDB(B1, 0, 1); SCHED;
        LDA(A2, 0, 1);
        MMF(0, 0, At, B0); MMF(0, 1, At, B1); SCHED;
        WAIT_V(0); BARF;
        LDA(At, 1, 0);
        MMF(1, 0, A2, B0); SCHED;
        LDB(B0, 1, 0);
        MMF(1, 1, A2, B1); SCHED;
        LDB(B1, 1, 1); LDA(A2, 1, 1);
        MMF(0, 0, At, B0); MMF(0, 1, At, B1);
        MMF(1, 0, A2, B0); MMF(1, 1, A2, B1);
    }
    STAMP(3);

    int mx = 0;
#pragma unroll
    for (int i = 0; i < 32; ++i) {
        const i32x4 v = __builtin_bit_cast(i32x4, acc[i >> 4][(i >> 1) & 1][(i >> 2) & 3][i & 1]);
        mx = max(max(mx, v[0]), v[1]); mx = max(max(mx, v[2]), v[3]);
    }
    {
        const unsigned wm = wave_max_u32((unsigned)mx);
        if (lane == 63) __hip_atomic_fetch_max(lctl + 8, wm, __ATOMIC_RELAXED, __HIP_MEMORY_SCOPE_WORKGROUP);
    }
    __syncthreads();
    if (tid < NR) {
        char* pw = (char*)(size_t)*(LAS u64*)(lctl + 16 + 2 * tid);
        __hip_atomic_store((gu64*)(size_t)(pw + OFF_CTL + CTL_AMAX + (rank * 128 + gi) * 8), ((u64)call << 32) | lctl[8], __ATOMIC_RELAXED, __HIP_MEMORY_SCOPE_SYSTEM);
    }
    STAMP(6);
    const int owner = ti >> 1;
    const bool local = owner == rank;
    char* optr = p.out[owner];
    char* oflg = p.ws[owner] + OFF_CTL + CTL_OFLG + (rank * 16 + (ti & 1) * 8 + tj) * 128;
    asm volatile("" : "+s"(optr), "+s"(oflg));
    const __amdgpu_buffer_rsrc_t ro = mk_rsrc(optr, OUT_BYTES), rx = mk_rsrc(ws + OFF_OEX, 16u << 20);
    const unsigned slot = (unsigned)((owner * 16 + (ti & 1) * 8 + tj) * 8 + wid) * OSLOT, area = lds0 + wid * 16384;
#define ACC(i) acc[(i) >> 4][((i) >> 1) & 1][((i) >> 2) & 3][(i) & 1]
    unsigned addr[33], len = 0;
    u32x2 sel[32];
    if (!local) {
        u32x4 bm;
#pragma unroll
        for (int k = 0; k < 4; ++k) {
            unsigned w = 0;
#pragma unroll
            for (int i = 31; i >= 0; --i) w = __builtin_amdgcn_alignbit(w, __float_as_uint(ACC(k * 8 + (i >> 2))[i & 3]), 31);
            bm[k] = ~w;
        }
        st128<AUX_SYS>(bm, rx, lane * 16, slot);
        unsigned a;
        const unsigned spare = area + 8192u + lane * 4u;
        len = stream_base(bm, area, a);
#pragma unroll
        for (int i = 0; i < 32; ++i) {
            const unsigned nib = (bm[i >> 3] >> (4 * (i & 7))) & 15u;
            sel[i] = *(const LAS u32x2*)(size_t)(((nib << 5) | (lctl_addr + 1024u)) | ((a << 3) & 0x18u));
            addr[i] = nib ? a & ~3u : spare; a += __builtin_popcount(nib);
            if (i == 15) {
                WAIT_V(0);
                BARF;
                if (tid == 0) st_sys(oflg, 2u * call - 1u);
            }
        }
        addr[32] = (a & 3u) ? a & ~3u : spare;
#pragma unroll
        for (int i = 0; i < 32; ++i) asm volatile("" : "+v"(addr[i]), "+v"(sel[i]));
    }
    STAMP(7);
    unsigned gmx, tmo;
    {
        const u64 t0 = __builtin_amdgcn_s_memrealtime(); unsigned spins = 0;
        for (;;) {
            const u32x4 g = __builtin_amdgcn_raw_buffer_load_b128(rc, tid * 16, CTL_AMAX, AUX_SYS);
            tmo = ld_agt(ctl + CTL_TMO);
            gmx = max(g.x, g.z);
            if (__all(g.y >= call && g.w >= call)) break;
            if (give_up(ctl, call, t0, spins)) { tmo = call; break; }
            __builtin_amdgcn_s_sleep(1);
        }
    }
    if (tmo == call) gmx = 0xffffffffu;
    gmx = wave_max_u32(gmx);
    if (lane == 63) __hip_atomic_fetch_max(lctl + 9, gmx, __ATOMIC_RELAXED, __HIP_MEMORY_SCOPE_WORKGROUP);
    __syncthreads();
    STAMP(4);
    const float scale = __uint_as_float(lctl[9]) / 448.0f, inv = 1.0f / scale;
    const f32x2 inv2 = {inv, inv};
    auto quant4 = [&](int i, bool relu) -> unsigned {
        f32x4 v = ACC(i);
        if (relu) {
            i32x4 iv = __builtin_bit_cast(i32x4, v);
#pragma unroll
            for (int e = 0; e < 4; ++e) iv[e] = max(iv[e], 0);
            v = __builtin_bit_cast(f32x4, iv);
        }
        const f32x2 lo = (f32x2){v[0], v[1]} * inv2, hi = (f32x2){v[2], v[3]} * inv2;
        const int r = __builtin_amdgcn_cvt_pk_fp8_f32(lo.x, lo.y, __float_as_int(lo.x), false);
        return (unsigned)__builtin_amdgcn_cvt_pk_fp8_f32(hi.x, hi.y, r, true);
    };
    if (local) {
        const unsigned obase = (unsigned)((ti & 1) * 256 + wr * 64 + (lane & 15)) * GN + (unsigned)(rank * NS + tj * 256 + wc * 64 + (lane >> 4) * 16);
#pragma unroll
        for (int blk = 0; blk < 8; ++blk)
            st128<AUX_SYS>((u32x4){quant4(blk * 4, true), quant4(blk * 4 + 1, true), quant4(blk * 4 + 2, true), quant4(blk * 4 + 3, true)}, ro, obase,
                                                   (unsigned)((blk >> 2) * 128 + (blk & 3) * 16) * GN);
    } else {
        unsigned carry = 0;
#pragma unroll
        for (int i = 0; i < 32; ++i) {
            const unsigned r = quant4(i, false);
            *(volatile LAS unsigned*)(size_t)addr[i] = __builtin_amdgcn_perm(carry, r, sel[i].x);
            carry = __builtin_amdgcn_perm(carry, r, sel[i].y);
        }
        *(volatile LAS unsigned*)(size_t)addr[32] = carry;
        for (unsigned k = 0; k * 1024u < len; k += 2) {
            const u32x4 lo = *(volatile LAS u32x4*)(size_t)(area + k * 1024u + lane * 16u), hi = *(volatile LAS u32x4*)(size_t)(area + (k + 1) * 1024u + lane * 16u);
            unsigned d[7] = {lo.x, lo.y, lo.z, lo.w, hi.x, hi.y, hi.z};
#pragma unroll
            for (int b = 0; b < 7; ++b) d[b] = (d[b] & 0x7f7f7f7fu) | ((hi.w << (7 - b)) & 0x80808080u);
            st128<AUX_SYS>((u32x4){d[0], d[1], d[2], d[3]}, rx, k * 1024u + lane * 16u < len ? lane * 16u : OOB, slot + 1024u + k * 1024u);
            st96<AUX_SYS>((u32x3){d[4], d[5], d[6]}, rx, (k + 1) * 1024u + lane * 16u < len ? lane * 12u : OOB, slot + 2048u + k * 1024u);
        }
        STAMP(8);
        WAIT_V(0);
        BARF;
        if (tid == 0) st_sys(oflg, 2u * call);
    }
    STAMP(5);
}

}

extern "C" void kernel_launch(int rank, int nranks, ncclComm_t comm,
                              hipStream_t stream,
                              void* const* d_in, const int* in_sizes, int n_in,
                              void* d_out, int out_size,
                              const int* in_dtypes, int out_dtype,
                              void* const* d_out_all,
                              void* const* d_ws_all, size_t ws_bytes) {
    (void)comm; (void)in_dtypes; (void)out_dtype; (void)d_out;
    if (nranks != NR || (unsigned)rank >= (unsigned)NR || n_in < 2 || ws_bytes < WS_NEED || in_sizes[0] != GM * KS ||
        in_sizes[1] != KS * GN || (unsigned)out_size != OUT_BYTES) {
        fprintf(stderr, "kernel_launch: unexpected problem shape\n");
        return;
    }
    HIP_CHECK(hipFuncSetAttribute((const void*)fused_gemm_quant, hipFuncAttributeMaxDynamicSharedMemorySize, LDS_BYTES));
    Params p;
    p.x = (const char*)d_in[0];
    p.w = (const char*)d_in[1];
    for (int r = 0; r < NR; ++r) { p.ws[r] = (char*)d_ws_all[r]; p.out[r] = (char*)d_out_all[r]; }
    p.rank = rank;
    hipLaunchKernelGGL(fused_gemm_quant, dim3(N_WG), dim3(512), LDS_BYTES, stream, p);
    HIP_CHECK(hipGetLastError());
}
```

```cpp
#include <hip/hip_runtime.h>
#include <rccl/rccl.h>
#include <cstdio>
#include <cstdint>

#define HIP_CHECK(expr) do { hipError_t _e = (expr); if (_e != hipSuccess) { \
    fprintf(stderr, "HIP error %s at %s:%d\n", hipGetErrorString(_e), __FILE__, __LINE__); \
    return; } } while (0)

namespace {

#define LAS __attribute__((address_space(3)))
typedef short bf16x8 __attribute__((ext_vector_type(8)));
typedef short s16x4 __attribute__((ext_vector_type(4)));
typedef float f32x4 __attribute__((ext_vector_type(4)));
typedef float f32x2 __attribute__((ext_vector_type(2)));
typedef int i32x4 __attribute__((ext_vector_type(4)));
typedef unsigned u32x4 __attribute__((ext_vector_type(4)));
typedef unsigned u32x3 __attribute__((ext_vector_type(3)));
typedef unsigned u32x2 __attribute__((ext_vector_type(2)));
typedef unsigned long long u64;
typedef __attribute__((address_space(1))) unsigned gu32;
typedef __attribute__((address_space(1))) u64 gu64;

constexpr int NR = 8;
constexpr int GM = 4096, GN = 16384;
constexpr int KS = 512, NS = 2048, MS = 512;
constexpr unsigned PIECE = 32768;
constexpr unsigned HALFB = 16384;
constexpr unsigned OSLOT = 16384;
constexpr unsigned OFF_X = 0;
constexpr unsigned OFF_WIN = 32u << 20;
constexpr unsigned OFF_WEX = 48u << 20;
constexpr unsigned OFF_OEX = 64u << 20;
constexpr unsigned OFF_CTL = 80u << 20;
constexpr unsigned CTL_SEQ = 0;
constexpr unsigned CTL_OFLG = 16384;
constexpr unsigned CTL_TMO = 32768;
constexpr unsigned CTL_EXPF = 36864;
constexpr unsigned CTL_XCNT = 45056;
constexpr unsigned CTL_WCNT = 49152;
constexpr unsigned CTL_AMAX = 53248;
constexpr unsigned CTL_TICK = 63488;
constexpr unsigned CTL_DBG = 65536;
constexpr size_t WS_NEED = (size_t)OFF_CTL + (1u << 20);
constexpr unsigned OUT_BYTES = (unsigned)MS * GN;
constexpr unsigned OOB = 0x80000000u;

constexpr int N_PULL = 56, N_GEMM = 128, N_EXP = 72, N_WG = N_EXP + N_PULL + N_GEMM;
constexpr int LAREA = 9216;
constexpr int LDS_CTL = 16 * LAREA, LDS_BYTES = LDS_CTL + 3072;
static_assert(LDS_CTL >= 131072, "the GEMM staging buffers live below the control words");
constexpr u64 DEADLINE = 50000000ull;
constexpr int N_PUSH = 3;
constexpr int EARLY_FROM = 57;
constexpr unsigned MERGE_B = 512;

struct Params {
    const char* x;
    const char* w;
    char* self;
    int rank;
    char* ws[NR];
    char* out[NR];
};

__device__ __forceinline__ __amdgpu_buffer_rsrc_t mk_rsrc(const void* p, unsigned bytes) {
    return __builtin_amdgcn_make_buffer_rsrc(const_cast<void*>(p), 0, bytes, 0x00020000);
}
constexpr int AUX_SYS = 17;
constexpr int AUX_AGT = 16;

template <int AUX> __device__ __forceinline__ void st128(u32x4 v, __amdgpu_buffer_rsrc_t r, unsigned voff, unsigned soff) { __builtin_amdgcn_raw_buffer_store_b128(v, r, voff + soff, 0, AUX); }
template <int AUX> __device__ __forceinline__ void st96(u32x3 v, __amdgpu_buffer_rsrc_t r, unsigned voff, unsigned soff) { __builtin_amdgcn_raw_buffer_store_b96(v, r, voff + soff, 0, AUX); }
__device__ __forceinline__ unsigned ld_sys(const void* p) { return __hip_atomic_load((const gu32*)(size_t)p, __ATOMIC_RELAXED, __HIP_MEMORY_SCOPE_SYSTEM); }
__device__ __forceinline__ unsigned ld_agt(const void* p) { return __hip_atomic_load((const gu32*)(size_t)p, __ATOMIC_RELAXED, __HIP_MEMORY_SCOPE_AGENT); }
__device__ __forceinline__ void st_sys(void* p, unsigned v) { __hip_atomic_store((gu32*)(size_t)p, v, __ATOMIC_RELAXED, __HIP_MEMORY_SCOPE_SYSTEM); }
#define WAIT_V(n) asm volatile("s_waitcnt vmcnt(" #n ")" ::: "memory")
#define WAIT_L(n) asm volatile("s_waitcnt lgkmcnt(" #n ")" ::: "memory")
#define BAR __builtin_amdgcn_s_barrier()
#define BARF do { asm volatile("" ::: "memory"); __builtin_amdgcn_s_barrier(); asm volatile("" ::: "memory"); } while (0)
#define SCHED __builtin_amdgcn_sched_barrier(0)

__device__ __forceinline__ bool give_up(char* ctl, unsigned call, u64 t0, unsigned& spins) {
    if ((++spins & 31u) != 0u) return false;
    const unsigned tm = __builtin_amdgcn_readfirstlane(ld_agt(ctl + CTL_TMO));
    if (tm == call) return true;
    if (__builtin_amdgcn_s_memrealtime() - t0 > DEADLINE) {
        __hip_atomic_store((gu32*)(size_t)(ctl + CTL_TMO), call, __ATOMIC_RELAXED, __HIP_MEMORY_SCOPE_AGENT);
        return true;
    }
    return false;
}
#define DPP(v, ctrl, rows) ((unsigned)__builtin_amdgcn_update_dpp(0, (int)(v), ctrl, rows, 0xf, false))
__device__ __forceinline__ unsigned wave_max_u32(unsigned v) {
    v = max(v, DPP(v, 0x111, 0xf)); v = max(v, DPP(v, 0x112, 0xf)); v = max(v, DPP(v, 0x114, 0xf)); v = max(v, DPP(v, 0x118, 0xf));
    v = max(v, DPP(v, 0x142, 0xa)); v = max(v, DPP(v, 0x143, 0xc));
    return v;
}
__device__ __forceinline__ unsigned wave_scan_u32(unsigned v) {
    v += DPP(v, 0x111, 0xf); v += DPP(v, 0x112, 0xf); v += DPP(v, 0x114, 0xf); v += DPP(v, 0x118, 0xf);
    v += DPP(v, 0x142, 0xa); v += DPP(v, 0x143, 0xc);
    return v;
}
__device__ __forceinline__ unsigned stream_base(const u32x4 bm, unsigned area, unsigned& a) {
    const unsigned len = __builtin_popcount(bm.x) + __builtin_popcount(bm.y) + __builtin_popcount(bm.z) + __builtin_popcount(bm.w);
    const unsigned reg = (len + 3u) & ~3u, inc = wave_scan_u32(reg);
    a = area + inc - reg;
    return (unsigned)__builtin_amdgcn_readlane((int)inc, 63);
}

#define STAMP(k) do { } while (0)

__device__ __forceinline__ int rot_of(int j) { return (j * 16 + 3) / 7; }

__global__ void __launch_bounds__(512, 2) fused_gemm_quant(Params p) {
    extern __shared__ __attribute__((aligned(16))) unsigned char smem[];
    LAS unsigned char* lds = (LAS unsigned char*)smem;
    const int tid = threadIdx.x, lane = tid & 63;
    const int wid = __builtin_amdgcn_readfirstlane(tid >> 6);
    const int rank = p.rank, bid = blockIdx.x;
    char* const ws = p.self;
    char* const ctl = ws + OFF_CTL;
    STAMP(0);
    const __amdgpu_buffer_rsrc_t rc = mk_rsrc(ctl, 65536);
    const unsigned call_v = (unsigned)__builtin_amdgcn_raw_ptr_buffer_atomic_add_i32(1, rc, lane == 0 ? 0u : OOB, CTL_SEQ + bid * 64 + wid * 8, 0);
    const unsigned ef0 = __builtin_amdgcn_raw_buffer_load_b32(rc, bid >= N_EXP && bid < N_EXP + N_PULL ? lane * 4u : OOB, CTL_EXPF + ((rank + 1 + (bid - N_EXP) % 7) & 7) * 1024, AUX_SYS);
    LAS unsigned* lctl = (LAS unsigned*)(lds + LDS_CTL);
    const unsigned lds0 = (unsigned)(size_t)lds, lctl_addr = lds0 + LDS_CTL;

    if (__builtin_expect(bid < N_EXP, 1)) {
        const __amdgpu_buffer_rsrc_t rx = mk_rsrc(p.x, (unsigned)GM * KS * 2), rw = mk_rsrc(p.w, (unsigned)KS * GN * 2);
        const __amdgpu_buffer_rsrc_t rd = mk_rsrc(ws, OFF_CTL);
        constexpr int PITCH = 528;
        if (bid >= 7 * (N_PUSH + 4)) __builtin_amdgcn_s_sleep(24);
        for (int q = bid; q < 640; q += N_EXP) {
            const int c = q / 80, idx = q % 80;
            const bool isx = idx >= 56 && idx < 72, push = q < 7 * N_PUSH;
            int dst = rank, pj = 0;
            if (isx) {
                pj = idx - 56;
                u32x4 v[4];
                const unsigned src = (unsigned)((pj * 256 + wid * 32 + (lane & 15)) * (KS * 2) + c * 128 + (lane >> 4) * 16);
#pragma unroll
                for (int u = 0; u < 4; ++u) v[u] = __builtin_amdgcn_raw_buffer_load_b128(rx, src + (u >> 1) * 16 * (KS * 2) + (u & 1) * 64, 0, 0);
                const unsigned d0 = OFF_X + (unsigned)((rank * 8 + c) * 16 + pj) * PIECE + wid * 4096 + lane * 16;
#pragma unroll
                for (int u = 0; u < 4; ++u) st128<AUX_SYS>(v[u], rd, d0 + u * 1024, 0);
            } else {
                if (idx < 56) { dst = (rank + 1 + idx % 7) & 7; pj = idx / 7; } else { pj = idx - 72; }
                u32x4 v[4];
                const unsigned src = (unsigned)(c * 64 + (tid >> 5)) * (GN * 2) + (unsigned)(dst * NS + pj * 256) * 2 + (tid & 31) * 16;
#pragma unroll
                for (int u = 0; u < 4; ++u) v[u] = __builtin_amdgcn_raw_buffer_load_b128(rw, src + u * 16 * (GN * 2), 0, 0);
#pragma unroll
                for (int u = 0; u < 4; ++u) *(LAS u32x4*)(lds + ((tid >> 5) + 16 * u) * PITCH + (tid & 31) * 16) = v[u];
                __syncthreads();
                if (q == bid) STAMP(3);
                const int bj = wid >> 2, wc = wid & 3;
                const int g = lane >> 4, qq = (lane >> 2) & 3, pp = lane & 3;
                const __amdgpu_buffer_rsrc_t rw2 = mk_rsrc(push ? p.ws[dst] : ws, OFF_CTL);
                const unsigned d0 = (push ? OFF_WIN + (unsigned)(rank * 64 + pj) * PIECE : (dst == rank ? OFF_WIN : OFF_WEX) + (unsigned)((dst * 8 + c) * 8 + pj) * PIECE) + wid * 4096 + lane * 16;
#pragma unroll
                for (int u = 0; u < 4; ++u) {
                    const int n = u >> 1, kf = u & 1;
                    const int a = (kf * 32 + g * 8 + qq) * PITCH + (wc * 64 + pp * 16 + (bj * 2 + n) * 4) * 2;
                    const s16x4 lo = __builtin_amdgcn_ds_read_tr16_b64_v4i16((LAS s16x4*)(lds + a));
                    const s16x4 hi = __builtin_amdgcn_ds_read_tr16_b64_v4i16((LAS s16x4*)(lds + a + 4 * PITCH));
                    const u32x2 l2 = __builtin_bit_cast(u32x2, lo), h2 = __builtin_bit_cast(u32x2, hi);
                    const u32x4 o = {l2.x, l2.y, h2.x, h2.y};
                    st128<AUX_SYS>(o, rw2, d0 + u * 1024, 0);
                }
            }
            WAIT_V(0);
            __syncthreads();
            if (wid == 0) {
                const unsigned call = __builtin_amdgcn_readfirstlane(call_v) + 1u;
                if (isx) {
                    if (lane < NR) {
                        if (lane == rank) st_sys(ctl + CTL_XCNT + (pj * 64 + rank * 8 + c) * 4, call * 32u);
                        else st_sys(p.ws[lane] + OFF_CTL + CTL_EXPF + (rank * 256 + c * 24 + 8 + pj) * 4, call);
                    }
                } else if (lane == 0) {
                    if (dst == rank || push) st_sys(p.ws[dst] + OFF_CTL + CTL_WCNT + (pj * 64 + rank * 8 + c) * 4, call * 32u);
                    else st_sys(p.ws[dst] + OFF_CTL + CTL_EXPF + (rank * 256 + c * 24 + pj) * 4, call);
                }
            }
            if (q == bid) STAMP(1);
        }
        STAMP(2);
        return;
    }

    if (tid == 0) { lctl[6] = 0u; lctl[8] = 0u; lctl[9] = 0u; }
    if (tid >= 64 && tid < 64 + NR) *(LAS u64*)(lctl + 16 + 2 * (tid - 64)) = (u64)(size_t)p.ws[tid - 64];
    if (tid >= 128 && tid < 192) {
        const unsigned m = (tid - 128) >> 2, off = tid & 3;
        u64 cs = 0x0c0c0c0c0c0c0c0cull; unsigned es = 0x0c0c0c0cu, k = off;
        for (unsigned e = 0; e < off; ++e) cs = (cs & ~(0xffull << (8 * e))) | ((u64)(4 + e) << (8 * e));
        for (unsigned e = 0; e < 4; ++e)
            if ((m >> e) & 1u) { cs = (cs & ~(0xffull << (8 * k))) | ((u64)e << (8 * k)); es = (es & ~(0xffu << (8 * e))) | (k << (8 * e)); ++k; }
        *(LAS u32x2*)(lctl + 256 + 2 * (tid - 128)) = (u32x2){(unsigned)cs, k >= 4 ? (unsigned)(cs >> 32) : (unsigned)cs};
        lctl[64 + tid - 128] = es;
    }
    if (bid >= N_EXP + N_PULL) __syncthreads();
    const unsigned call = __builtin_amdgcn_readfirstlane(call_v) + 1u;

    if (bid < N_EXP + N_PULL) {
        const int q = bid - N_EXP, j = q % 7;
        const int src = (rank + 1 + j) & 7, rot = rot_of(j);
        const __amdgpu_buffer_rsrc_t rs = mk_rsrc(p.ws[src], OFF_CTL), rd = mk_rsrc(ws, OFF_CTL);
        const char* ef = ctl + CTL_EXPF + src * 1024 + lane * 4;
        u64 m0 = __ballot(ef0 >= call), m1 = 0, m2 = 0;
        const unsigned aoff = lane == 0 ? 0u : OOB;
        constexpr unsigned NU = (192 - N_PUSH) * 16;
        const unsigned tbase = (call - 1u) * NU - 64u, toff = CTL_TICK + j * 128;
        unsigned tk = (q / 7) * 8 + wid;
        unsigned cnt1 = 0, off1 = OOB;
        while (tk < NU) {
            const int pp = tk / 16 + N_PUSH, c = pp / 24, r = pp % 24, sc = src * 8 + c;
            int id; unsigned so, dof, cnt;
            if (r < 8) { id = c * 24 + r; so = OFF_WEX + (unsigned)((rank * 8 + c) * 8 + r) * PIECE; dof = OFF_WIN + (unsigned)(sc * 8 + r) * PIECE; cnt = CTL_WCNT + (r * 64 + sc) * 4; }
            else { const int i = (rot + r - 8) & 15; id = c * 24 + 8 + i; so = OFF_X + (unsigned)(sc * 16 + i) * PIECE; dof = so; cnt = CTL_XCNT + (i * 64 + sc) * 4; }
            so += (tk % 16) * 2048u; dof += (tk % 16) * 2048u;
            const u64 mm = id < 64 ? m0 : (id < 128 ? m1 : m2);
            if (!((mm >> (id & 63)) & 1)) {
                const u64 t0 = __builtin_amdgcn_s_memrealtime(); unsigned spins = 0;
                for (;;) {
                    const unsigned f0 = ld_sys(ef), f1 = ld_sys(ef + 256), f2 = ld_sys(ef + 512);
                    m0 = __ballot(f0 >= call); m1 = __ballot(f1 >= call); m2 = __ballot(f2 >= call);
                    const u64 mn = id < 64 ? m0 : (id < 128 ? m1 : m2);
                    if ((mn >> (id & 63)) & 1) break;
                    if (give_up(ctl, call, t0, spins)) { m0 = m1 = m2 = ~0ull; break; }
                    __builtin_amdgcn_s_sleep(2);
                }
            }
            if (off1 == OOB) STAMP(1);
            u32x4 v = __builtin_amdgcn_raw_buffer_load_b128(rs, lane * 16, so, AUX_SYS);
            WAIT_V(1);
            __builtin_amdgcn_raw_ptr_buffer_atomic_add_i32(2, rc, off1, cnt1, 0);
            WAIT_V(1);
            st128<AUX_AGT>(v, rd, lane * 16, dof);
            v = __builtin_amdgcn_raw_buffer_load_b128(rs, lane * 16, so + 1024, AUX_SYS);
            const int tnext = __builtin_amdgcn_raw_ptr_buffer_atomic_add_i32(1, rc, aoff, toff, 0);
            WAIT_V(0);
            st128<AUX_AGT>(v, rd, lane * 16, dof + 1024);
            cnt1 = cnt; off1 = aoff;
            tk = (unsigned)__builtin_amdgcn_readfirstlane(tnext) - tbase;
        }
        WAIT_V(0);
        __builtin_amdgcn_raw_ptr_buffer_atomic_add_i32(2, rc, off1, cnt1, 0);
        STAMP(2);
        __syncthreads();

        const int g = (q / 7) * 8 + wid, tA = g & 15, tB = tA ^ 8, w2 = g >> 4;
        const __amdgpu_buffer_rsrc_t ro = mk_rsrc(p.out[rank], OUT_BYTES);
        const char* ofA = ctl + CTL_OFLG + (src * 16 + tA) * 128;
        const char* ofB = ctl + CTL_OFLG + (src * 16 + tB) * 128;
        const unsigned slA = OFF_OEX + (unsigned)((rank * 16 + tA) * 8 + w2) * OSLOT, slB = OFF_OEX + (unsigned)((rank * 16 + tB) * 8 + w2 + 4) * OSLOT;
        const unsigned arA = lds0 + wid * 2 * LAREA, arB = arA + LAREA;
        const unsigned obase = (unsigned)(lane & 15) * GN + (unsigned)(src * NS + (tA & 7) * 256 + w2 * 64 + (lane >> 4) * 16);
        const unsigned rowA = (tA >> 3) * 256, rowB = (tB >> 3) * 256 + 64;
        unsigned adA[32], seA[32], adB[32], seB[32], pA, pB, lenA, lenB;
        u32x4 bmA, bmB;
#define PLAN(X, i0, i1) do { _Pragma("unroll") for (int i = i0; i < i1; ++i) { const unsigned nib = (bm##X[i >> 3] >> (4 * (i & 7))) & 15u; \
                ad##X[i] = p##X & ~3u; se##X[i] = lctl[64 + nib * 4 + (p##X & 3u)]; p##X += __builtin_popcount(nib); } \
            _Pragma("unroll") for (int i = i0; i < i1; ++i) asm volatile("" : "+v"(ad##X[i]), "+v"(se##X[i])); } while (0)
        bool ok = true;
        unsigned fl = 0;
        auto wait2 = [&](unsigned need, unsigned who) {
            const u64 t0 = __builtin_amdgcn_s_memrealtime(); unsigned spins = 0;
            for (;;) {
                fl = ld_sys(lane & 1 ? ofB : ofA);
                if ((unsigned)__ballot(fl >= need) & who) break;
                if (give_up(ctl, call, t0, spins)) { ok = false; fl = ~0u; break; }
                __builtin_amdgcn_s_sleep(8);
            }
        };
        constexpr unsigned whoA = 1, whoB = 2;
        wait2(2u * call - 1u, 3);
        const bool a_first = (unsigned)__builtin_amdgcn_readlane((int)fl, 0) >= 2u * call - 1u;
        bool both = false;
        STAMP(3);
#define WSTAMP(k) do { } while (0)
#define LOAD_EVEN(X, k) __builtin_amdgcn_raw_buffer_load_b128(rs, (k) * 1024u + lane * 16u < len##X ? lane * 16u : OOB, sl##X + 1024u + (k) * 1024u, AUX_SYS)
#define LOAD_ODD(X, k) __builtin_amdgcn_raw_buffer_load_b96(rs, (k) * 1024u + lane * 16u < len##X ? lane * 12u : OOB, sl##X + 1024u + (k) * 1024u, AUX_SYS)
#define PUT_EVEN(X, k, v) do { d7 = ((v.x >> 7) & 0x01010101u) | ((v.y >> 6) & 0x02020202u) | ((v.z >> 5) & 0x04040404u) | ((v.w >> 4) & 0x08080808u); \
            *(LAS u32x4*)(size_t)(ar##X + (k) * 1024u + lane * 16u) = v & 0x7f7f7f7fu; } while (0)
#define PUT_ODD(X, k, w) do { d7 |= ((w.x >> 3) & 0x10101010u) | ((w.y >> 2) & 0x20202020u) | ((w.z >> 1) & 0x40404040u); \
            *(LAS u32x4*)(size_t)(ar##X + (k) * 1024u + lane * 16u) = (u32x4){w.x & 0x7f7f7f7fu, w.y & 0x7f7f7f7fu, w.z & 0x7f7f7f7fu, d7}; } while (0)
#define REST(X, other) do { bool have = false; u32x4 v; \
            for (unsigned k = 2; k * 1024u < len##X; k += 2) { \
                if (!have) v = LOAD_EVEN(X, k); \
                PUT_EVEN(X, k, v); \
                if ((k + 1) * 1024u < len##X) { \
                    const u32x3 w = LOAD_ODD(X, k + 1); \
                    have = len##X <= (k + 2) * 1024u + MERGE_B; \
                    if (have) v = LOAD_EVEN(X, k + 2); \
                    const unsigned fo = __builtin_amdgcn_raw_buffer_load_b32(rc, both ? OOB : aoff, CTL_OFLG + (src * 16 + (other)) * 128, AUX_SYS); \
                    PUT_ODD(X, k + 1, w); \
                    both = both || (unsigned)__builtin_amdgcn_readfirstlane(fo) >= 2u * call; \
                } } } while (0)
#define EXPAND(X) do { unsigned hi[32], lo = *(const LAS unsigned*)(size_t)ad##X[0]; \
            _Pragma("unroll") for (int i = 0; i < 32; ++i) hi[i] = *(const LAS unsigned*)(size_t)(ad##X[i] + 4u); \
            if (!ok) { _Pragma("unroll") for (int i = 0; i < 32; ++i) se##X[i] = ~0u; }        \
            _Pragma("unroll") for (int blk = 0; blk < 8; ++blk) { u32x4 o; \
                _Pragma("unroll") for (int e = 0; e < 4; ++e) { const int i = blk * 4 + e; \
                    if (i) lo = ad##X[i] != ad##X[i - 1] ? hi[i - 1] : lo; \
                    o[e] = __builtin_amdgcn_perm(hi[i], lo, se##X[i]); } \
                st128<AUX_SYS>(o, ro, obase, (row##X + (blk >> 2) * 128 + (blk & 3) * 16) * GN); } } while (0)
#define RUN(X, Y) do { unsigned d7; \
            bm##X = __builtin_amdgcn_raw_buffer_load_b128(rs, lane * 16, sl##X, AUX_SYS); \
            const bool early = (unsigned)__ballot(fl >= 2u * call - 1u) & who##Y; \
            bm##Y = __builtin_amdgcn_raw_buffer_load_b128(rs, early ? lane * 16u : OOB, sl##Y, AUX_SYS); \
            len##X = stream_base(bm##X, ar##X, p##X); \
            PLAN(X, 0, 32); \
            if (!early) { wait2(2u * call - 1u, who##Y); bm##Y = __builtin_amdgcn_raw_buffer_load_b128(rs, lane * 16, sl##Y, AUX_SYS); } \
            len##Y = stream_base(bm##Y, ar##Y, p##Y); \
            wait2(2u * call, who##X); \
            both = (unsigned)__ballot(fl >= 2u * call) & who##Y; \
            STAMP(5); WSTAMP(0); \
            { const u32x4 v = LOAD_EVEN(X, 0u); PLAN(Y, 0, 16); PUT_EVEN(X, 0u, v); } \
            { const u32x3 w = LOAD_ODD(X, 1u); PLAN(Y, 16, 32); PUT_ODD(X, 1u, w); } \
            REST(X, t##Y); \
            WSTAMP(1); \
            if (!both) { wait2(2u * call, who##Y); both = true; } \
            const u32x4 v0 = LOAD_EVEN(Y, 0u); const u32x3 w1 = LOAD_ODD(Y, 1u);        \
            EXPAND(X); \
            WSTAMP(2); \
            PUT_EVEN(Y, 0u, v0); PUT_ODD(Y, 1u, w1); \
            REST(Y, t##X); \
            WSTAMP(3); \
            EXPAND(Y); } while (0)
        if (a_first) RUN(A, B); else RUN(B, A);
        WSTAMP(4);
        WAIT_V(0);
        WSTAMP(5);
        STAMP(4);
        return;
    }

    const int gi = bid - (N_EXP + N_PULL), xg = gi & 7, ix = gi >> 3;
    const int ti = (xg >> 1) * 4 + (ix & 3), tj = (xg & 1) * 4 + (ix >> 2);
    const int wr = wid >> 2, wc = wid & 3;
    int scv;
    {
        int j0 = 0;
#pragma unroll
        for (int j = 1; j < 7; ++j) if (rot_of(j) <= ti) j0 = j;
        const int t2 = lane - 8, c = t2 / 7, k = t2 - c * 7;
        int j = j0 - k; if (j < 0) j += 7;
        scv = lane < 8 ? rank * 8 + lane : ((rank + 1 + j) & 7) * 8 + c;
    }
    const unsigned target = call * 32u;
    u64 avail = 0, taken = 0;
    unsigned pa = 0, pb = 0;
    const char* const pxa = ctl + CTL_XCNT + (ti * 64 + scv) * 4;
    const char* const pwa = ctl + CTL_WCNT + (tj * 64 + scv) * 4;
#define POLL_WAIT_V6(n) do { if ((n) >= EARLY_FROM && (n) < 64 && wid == 0 && avail == 0) { \
        asm volatile("global_load_dword %0, %2, off sc1\n\tglobal_load_dword %1, %3, off sc1\n\ts_waitcnt vmcnt(8)" : "=&v"(pa), "=&v"(pb) : "v"(pxa), "v"(pwa) : "memory"); \
    } else WAIT_V(6); } while (0)
    auto next_step = [&](int n) -> unsigned {
        if (avail == 0) {
            if (wid == 0) {
                if (n >= EARLY_FROM) {
                    asm volatile("s_waitcnt vmcnt(2)" : "+v"(pa), "+v"(pb) :: "memory");
                    avail = __ballot(pa >= target && pb >= target) & ~taken;
                }
                const u64 t0 = __builtin_amdgcn_s_memrealtime(); unsigned spins = 0;
                while (avail == 0) {
                    const unsigned a = ld_agt(pxa), b = ld_agt(pwa);
                    avail = __ballot(a >= target && b >= target) & ~taken;
                    if (avail != 0) break;
                    if (give_up(ctl, call, t0, spins)) { avail = ~taken; break; }
                    if (n < 50) __builtin_amdgcn_s_sleep(100); else if (n < 56) __builtin_amdgcn_s_sleep(16); else __builtin_amdgcn_s_sleep(1);
                }
                const u32x4 mv = {(unsigned)avail, (unsigned)(avail >> 32), (unsigned)n + 1u, 0u};
                asm volatile("ds_write_b128 %0, %1" :: "v"(lctl_addr + 16), "v"(mv) : "memory");
            } else {
                for (;;) {
                    u32x4 mv;
                    asm volatile("ds_read_b128 %0, %1\n\ts_waitcnt lgkmcnt(0)" : "=v"(mv) : "v"(lctl_addr + 16) : "memory");
                    if ((unsigned)__builtin_amdgcn_readfirstlane(mv.z) == (unsigned)n + 1u) {
                        avail = ((u64)(unsigned)__builtin_amdgcn_readfirstlane(mv.y) << 32) | (unsigned)__builtin_amdgcn_readfirstlane(mv.x);
                        break;
                    }
                    __builtin_amdgcn_s_sleep(1);
                }
            }
        }
        const int bit = __builtin_ctzll(avail);
        avail &= avail - 1; taken |= 1ull << bit;
        return (unsigned)__builtin_amdgcn_readlane(scv, bit);
    };
    const char* const xbase = ws + OFF_X + (size_t)ti * PIECE;
    const char* const wbase = ws + OFF_WIN + (size_t)tj * PIECE;
#define A_OF(sc) (xbase + (size_t)((sc) * (16u * PIECE)))
#define B_OF(sc) (wbase + (size_t)((sc) * (8u * PIECE)))
    const unsigned voff = (unsigned)tid * 16u, ldsw = (unsigned)wid * 1024u;
    const int aoff = wr * 8192 + lane * 16, boff = wc * 4096 + lane * 16;
#define SA(b, h) (((b) * 2 + (h)) * HALFB)
#define SB(b, h) ((4 + (b) * 2 + (h)) * HALFB)
#define STAGE(bufoff, gbase) do { _Pragma("unroll") for (int _i = 0; _i < 2; ++_i) \
        __builtin_amdgcn_global_load_lds((const unsigned*)((gbase) + voff + _i * 8192), (LAS unsigned*)(lds + (bufoff) + ldsw + _i * 8192), 16, 0, AUX_AGT); } while (0)
#define LDA(dst, b, h) do { _Pragma("unroll") for (int m = 0; m < 4; ++m) _Pragma("unroll") for (int k = 0; k < 2; ++k) dst[m][k] = *(const LAS bf16x8*)(lds + SA(b, h) + aoff + m * 2048 + k * 1024); } while (0)
#define LDB(dst, b, h) do { _Pragma("unroll") for (int n = 0; n < 2; ++n) _Pragma("unroll") for (int k = 0; k < 2; ++k) dst[n][k] = *(const LAS bf16x8*)(lds + SB(b, h) + boff + n * 2048 + k * 1024); } while (0)
#define MMA(ai, bj, At, Bt) do { __builtin_amdgcn_s_setprio(1); _Pragma("unroll") for (int m = 0; m < 4; ++m) _Pragma("unroll") for (int n = 0; n < 2; ++n) _Pragma("unroll") for (int k = 0; k < 2; ++k) \
        acc[ai][bj][m][n] = __builtin_amdgcn_mfma_f32_16x16x32_bf16(Bt[n][k], At[m][k], acc[ai][bj][m][n], 0, 0, 0); __builtin_amdgcn_s_setprio(0); } while (0)
    f32x4 acc[2][2][4][2];
#pragma unroll
    for (int a = 0; a < 2; ++a)
#pragma unroll
        for (int b = 0; b < 2; ++b)
#pragma unroll
            for (int m = 0; m < 4; ++m)
#pragma unroll
                for (int n = 0; n < 2; ++n) acc[a][b][m][n] = (f32x4){0.f, 0.f, 0.f, 0.f};
    bf16x8 At[4][2], B0[2][2], B1[2][2];
    constexpr int NT = 64;
    const char* a1;
    {
        const unsigned s0 = next_step(0);
        const char *a0 = A_OF(s0), *b0 = B_OF(s0);
        STAGE(SB(0, 0), b0); STAGE(SA(0, 0), a0); STAGE(SB(0, 1), b0 + HALFB); STAGE(SA(0, 1), a0 + HALFB);
        if (wr == 1) BAR;
        WAIT_V(4); BAR;
        const unsigned s1 = next_step(1);
        a1 = A_OF(s1);
        const char* b1 = B_OF(s1);
        STAGE(SB(1, 0), b1); STAGE(SA(1, 0), a1); STAGE(SB(1, 1), b1 + HALFB);
        WAIT_V(6); BAR;
    }
    STAMP(1);
    for (int t = 0; t < NT - 2; t += 2) {
        LDB(B0, 0, 0); SCHED; LDA(At, 0, 0); STAGE(SA(1, 1), a1 + HALFB);
        WAIT_L(8); BAR; WAIT_L(0); MMA(0, 0, At, B0); BAR; SCHED;
        const unsigned s2 = next_step(t + 2);
        const char *a2 = A_OF(s2), *b2 = B_OF(s2);
        LDB(B1, 0, 1); STAGE(SB(0, 0), b2);
        BAR; WAIT_L(0); MMA(0, 1, At, B1); BAR;
        LDA(At, 0, 1); STAGE(SA(0, 0), a2);
        BAR; WAIT_L(0); MMA(1, 0, At, B0); BAR; SCHED;
        STAGE(SB(0, 1), b2 + HALFB);
        POLL_WAIT_V6(t + 3); BAR; MMA(1, 1, At, B1); BAR;
        LDB(B0, 1, 0); SCHED; LDA(At, 1, 0); STAGE(SA(0, 1), a2 + HALFB);
        WAIT_L(8); BAR; WAIT_L(0); MMA(0, 0, At, B0); BAR; SCHED;
        const unsigned s3 = next_step(t + 3);
        a1 = A_OF(s3);
        const char* b3 = B_OF(s3);
        LDB(B1, 1, 1); STAGE(SB(1, 0), b3);
        BAR; WAIT_L(0); MMA(0, 1, At, B1); BAR;
        LDA(At, 1, 1); STAGE(SA(1, 0), a1);
        BAR; WAIT_L(0); MMA(1, 0, At, B0); BAR; SCHED;
        STAGE(SB(1, 1), b3 + HALFB);
        POLL_WAIT_V6(t + 4); BAR; MMA(1, 1, At, B1); BAR;
    }
    STAMP(2);
    {
        STAGE(SA(1, 1), a1 + HALFB);
        if (wr == 0) BARF;
        asm volatile("" ::: "memory");
        bf16x8 A2[4][2];
#define MMF(ai, bj, At, Bt) do { _Pragma("unroll") for (int m = 0; m < 4; ++m) _Pragma("unroll") for (int n = 0; n < 2; ++n) _Pragma("unroll") for (int k = 0; k < 2; ++k) \
        acc[ai][bj][m][n] = __builtin_amdgcn_mfma_f32_16x16x32_bf16(Bt[n][k], At[m][k], acc[ai][bj][m][n], 0, 0, 0); } while (0)
        LDB(B0, 0, 0); LDA(At, 0, 0); LDB(B1, 0, 1); SCHED;
        LDA(A2, 0, 1);
        MMF(0, 0, At, B0); MMF(0, 1, At, B1); SCHED;
        WAIT_V(0); BARF;
        LDA(At, 1, 0);
        MMF(1, 0, A2, B0); SCHED;
        LDB(B0, 1, 0);
        MMF(1, 1, A2, B1); SCHED;
        LDB(B1, 1, 1); LDA(A2, 1, 1);
        MMF(0, 0, At, B0); MMF(0, 1, At, B1);
        MMF(1, 0, A2, B0); MMF(1, 1, A2, B1);
    }
    STAMP(3);

    int mx = 0;
#pragma unroll
    for (int i = 0; i < 32; ++i) {
        const i32x4 v = __builtin_bit_cast(i32x4, acc[i >> 4][(i >> 1) & 1][(i >> 2) & 3][i & 1]);
        mx = max(max(mx, v[0]), v[1]); mx = max(max(mx, v[2]), v[3]);
    }
    {
        const unsigned wm = wave_max_u32((unsigned)mx);
        if (lane == 63) __hip_atomic_fetch_max(lctl + 8, wm, __ATOMIC_RELAXED, __HIP_MEMORY_SCOPE_WORKGROUP);
    }
    __syncthreads();
    if (tid < NR) {
        char* pw = (char*)(size_t)*(LAS u64*)(lctl + 16 + 2 * tid);
        __hip_atomic_store((gu64*)(size_t)(pw + OFF_CTL + CTL_AMAX + (rank * 128 + gi) * 8), ((u64)call << 32) | lctl[8], __ATOMIC_RELAXED, __HIP_MEMORY_SCOPE_SYSTEM);
    }
    STAMP(6);
    const int owner = ti >> 1;
    const bool local = owner == rank;
    char* optr = p.out[owner];
    char* oflg = p.ws[owner] + OFF_CTL + CTL_OFLG + (rank * 16 + (ti & 1) * 8 + tj) * 128;
    asm volatile("" : "+s"(optr), "+s"(oflg));
    const __amdgpu_buffer_rsrc_t ro = mk_rsrc(optr, OUT_BYTES), rx = mk_rsrc(ws + OFF_OEX, 16u << 20);
    const unsigned slot = (unsigned)((owner * 16 + (ti & 1) * 8 + tj) * 8 + wid) * OSLOT, area = lds0 + wid * 16384;
#define ACC(i) acc[(i) >> 4][((i) >> 1) & 1][((i) >> 2) & 3][(i) & 1]
    unsigned addr[33], len = 0;
    u32x2 sel[32];
    if (!local) {
        u32x4 bm;
#pragma unroll
        for (int k = 0; k < 4; ++k) {
            unsigned w = 0;
#pragma unroll
            for (int i = 31; i >= 0; --i) w = __builtin_amdgcn_alignbit(w, __float_as_uint(ACC(k * 8 + (i >> 2))[i & 3]), 31);
            bm[k] = ~w;
        }
        st128<AUX_SYS>(bm, rx, lane * 16, slot);
        unsigned a;
        const unsigned spare = area + 8192u + lane * 4u;
        len = stream_base(bm, area, a);
#pragma unroll
        for (int i = 0; i < 32; ++i) {
            const unsigned nib = (bm[i >> 3] >> (4 * (i & 7))) & 15u;
            sel[i] = *(const LAS u32x2*)(size_t)(((nib << 5) | (lctl_addr + 1024u)) | ((a << 3) & 0x18u));
            addr[i] = nib ? a & ~3u : spare; a += __builtin_popcount(nib);
            if (i == 15) {
                WAIT_V(0);
                BARF;
                if (tid == 0) st_sys(oflg, 2u * call - 1u);
            }
        }
        addr[32] = (a & 3u) ? a & ~3u : spare;
#pragma unroll
        for (int i = 0; i < 32; ++i) asm volatile("" : "+v"(addr[i]), "+v"(sel[i]));
    }
    STAMP(7);
    unsigned gmx, tmo;
    {
        const u64 t0 = __builtin_amdgcn_s_memrealtime(); unsigned spins = 0;
        for (;;) {
            const u32x4 g = __builtin_amdgcn_raw_buffer_load_b128(rc, tid * 16, CTL_AMAX, AUX_SYS);
            tmo = ld_agt(ctl + CTL_TMO);
            gmx = max(g.x, g.z);
            if (__all(g.y >= call && g.w >= call)) break;
            if (give_up(ctl, call, t0, spins)) { tmo = call; break; }
            __builtin_amdgcn_s_sleep(1);
        }
    }
    if (tmo == call) gmx = 0xffffffffu;
    gmx = wave_max_u32(gmx);
    if (lane == 63) __hip_atomic_fetch_max(lctl + 9, gmx, __ATOMIC_RELAXED, __HIP_MEMORY_SCOPE_WORKGROUP);
    __syncthreads();
    STAMP(4);
    const float inv = 448.0f / __uint_as_float(lctl[9]);
    const f32x2 inv2 = {inv, inv};
    auto quant4 = [&](int i, bool relu) -> unsigned {
        f32x4 v = ACC(i);
        if (relu) {
            i32x4 iv = __builtin_bit_cast(i32x4, v);
#pragma unroll
            for (int e = 0; e < 4; ++e) iv[e] = max(iv[e], 0);
            v = __builtin_bit_cast(f32x4, iv);
        }
        const f32x2 lo = (f32x2){v[0], v[1]} * inv2, hi = (f32x2){v[2], v[3]} * inv2;
        const int r = __builtin_amdgcn_cvt_pk_fp8_f32(lo.x, lo.y, __float_as_int(lo.x), false);
        return (unsigned)__builtin_amdgcn_cvt_pk_fp8_f32(hi.x, hi.y, r, true);
    };
    if (local) {
        const unsigned obase = (unsigned)((ti & 1) * 256 + wr * 64 + (lane & 15)) * GN + (unsigned)(rank * NS + tj * 256 + wc * 64 + (lane >> 4) * 16);
#pragma unroll
        for (int blk = 0; blk < 8; ++blk)
            st128<AUX_SYS>((u32x4){quant4(blk * 4, true), quant4(blk * 4 + 1, true), quant4(blk * 4 + 2, true), quant4(blk * 4 + 3, true)}, ro, obase,
                                                   (unsigned)((blk >> 2) * 128 + (blk & 3) * 16) * GN);
    } else {
        unsigned carry = 0;
#pragma unroll
        for (int i = 0; i < 32; ++i) {
            const unsigned r = quant4(i, false);
            *(volatile LAS unsigned*)(size_t)addr[i] = __builtin_amdgcn_perm(carry, r, sel[i].x);
            carry = __builtin_amdgcn_perm(carry, r, sel[i].y);
        }
        *(volatile LAS unsigned*)(size_t)addr[32] = carry;
        for (unsigned k = 0; k * 1024u < len; k += 2) {
            const u32x4 lo = *(volatile LAS u32x4*)(size_t)(area + k * 1024u + lane * 16u), hi = *(volatile LAS u32x4*)(size_t)(area + (k + 1) * 1024u + lane * 16u);
            unsigned d[7] = {lo.x, lo.y, lo.z, lo.w, hi.x, hi.y, hi.z};
#pragma unroll
            for (int b = 0; b < 7; ++b) d[b] = (d[b] & 0x7f7f7f7fu) | ((hi.w << (7 - b)) & 0x80808080u);
            st128<AUX_SYS>((u32x4){d[0], d[1], d[2], d[3]}, rx, k * 1024u + lane * 16u < len ? lane * 16u : OOB, slot + 1024u + k * 1024u);
            st96<AUX_SYS>((u32x3){d[4], d[5], d[6]}, rx, (k + 1) * 1024u + lane * 16u < len ? lane * 12u : OOB, slot + 2048u + k * 1024u);
        }
        STAMP(8);
        WAIT_V(0);
        BARF;
        if (tid == 0) st_sys(oflg, 2u * call);
    }
    STAMP(5);
}

}

extern "C" void kernel_launch(int rank, int nranks, ncclComm_t comm,
                              hipStream_t stream,
                              void* const* d_in, const int* in_sizes, int n_in,
                              void* d_out, int out_size,
                              const int* in_dtypes, int out_dtype,
                              void* const* d_out_all,
                              void* const* d_ws_all, size_t ws_bytes) {
    (void)comm; (void)in_dtypes; (void)out_dtype; (void)d_out;
    if (nranks != NR || (unsigned)rank >= (unsigned)NR || n_in < 2 || ws_bytes < WS_NEED || in_sizes[0] != GM * KS ||
        in_sizes[1] != KS * GN || (unsigned)out_size != OUT_BYTES) {
        fprintf(stderr, "kernel_launch: unexpected problem shape\n");
        return;
    }
    HIP_CHECK(hipFuncSetAttribute((const void*)fused_gemm_quant, hipFuncAttributeMaxDynamicSharedMemorySize, LDS_BYTES));
    Params p;
    p.x = (const char*)d_in[0];
    p.w = (const char*)d_in[1];
    for (int r = 0; r < NR; ++r) { p.ws[r] = (char*)d_ws_all[r]; p.out[r] = (char*)d_out_all[r]; }
    p.rank = rank;
    p.self = (char*)d_ws_all[rank];
    hipLaunchKernelGGL(fused_gemm_quant, dim3(N_WG), dim3(512), LDS_BYTES, stream, p);
    HIP_CHECK(hipGetLastError());
}
```

```cpp
#include <hip/hip_runtime.h>
#include <rccl/rccl.h>
#include <cstdio>
#include <cstdint>

#define HIP_CHECK(expr) do { hipError_t _e = (expr); if (_e != hipSuccess) { \
    fprintf(stderr, "HIP error %s at %s:%d\n", hipGetErrorString(_e), __FILE__, __LINE__); \
    return; } } while (0)

namespace {

#define LAS __attribute__((address_space(3)))
typedef short bf16x8 __attribute__((ext_vector_type(8)));
typedef short s16x4 __attribute__((ext_vector_type(4)));
typedef float f32x4 __attribute__((ext_vector_type(4)));
typedef float f32x2 __attribute__((ext_vector_type(2)));
typedef int i32x4 __attribute__((ext_vector_type(4)));
typedef unsigned u32x4 __attribute__((ext_vector_type(4)));
typedef unsigned u32x3 __attribute__((ext_vector_type(3)));
typedef unsigned u32x2 __attribute__((ext_vector_type(2)));
typedef unsigned long long u64;
typedef __attribute__((address_space(1))) unsigned gu32;
typedef __attribute__((address_space(1))) u64 gu64;

constexpr int NR = 8;
constexpr int GM = 4096, GN = 16384;
constexpr int KS = 512, NS = 2048, MS = 512;
constexpr unsigned PIECE = 32768;
constexpr unsigned HALFB = 16384;
constexpr unsigned OSLOT = 16384;
constexpr unsigned OFF_X = 0;
constexpr unsigned OFF_WIN = 32u << 20;
constexpr unsigned OFF_WEX = 48u << 20;
constexpr unsigned OFF_OEX = 64u << 20;
constexpr unsigned OFF_CTL = 80u << 20;
constexpr unsigned CTL_SEQ = 0;
constexpr unsigned CTL_OFLG = 16384;
constexpr unsigned CTL_TMO = 32768;
constexpr unsigned CTL_EXPF = 36864;
constexpr unsigned CTL_XCNT = 45056;
constexpr unsigned CTL_WCNT = 49152;
constexpr unsigned CTL_AMAX = 53248;
constexpr unsigned CTL_TICK = 63488;
constexpr unsigned CTL_DBG = 65536;
constexpr size_t WS_NEED = (size_t)OFF_CTL + (1u << 20);
constexpr unsigned OUT_BYTES = (unsigned)MS * GN;
constexpr unsigned OOB = 0x80000000u;

constexpr int N_PULL = 56, N_GEMM = 128, N_EXP = 72, N_WG = N_EXP + N_PULL + N_GEMM;
constexpr int LAREA = 9216;
constexpr int LDS_CTL = 16 * LAREA, LDS_BYTES = LDS_CTL + 3072;
static_assert(LDS_CTL >= 131072, "the GEMM staging buffers live below the control words");
constexpr u64 DEADLINE = 50000000ull;
constexpr int N_PUSH = 3;
constexpr int EARLY_FROM = 57;
constexpr unsigned MERGE_B = 512;

struct Params {
    const char* x;
    const char* w;
    char* self;
    int rank;
    char* ws[NR];
    char* out[NR];
};

__device__ __forceinline__ __amdgpu_buffer_rsrc_t mk_rsrc(const void* p, unsigned bytes) {
    return __builtin_amdgcn_make_buffer_rsrc(const_cast<void*>(p), 0, bytes, 0x00020000);
}
constexpr int AUX_SYS = 17;
constexpr int AUX_AGT = 16;

template <int AUX> __device__ __forceinline__ void st128(u32x4 v, __amdgpu_buffer_rsrc_t r, unsigned voff, unsigned soff) { __builtin_amdgcn_raw_buffer_store_b128(v, r, voff + soff, 0, AUX); }
template <int AUX> __device__ __forceinline__ void st96(u32x3 v, __amdgpu_buffer_rsrc_t r, unsigned voff, unsigned soff) { __builtin_amdgcn_raw_buffer_store_b96(v, r, voff + soff, 0, AUX); }
__device__ __forceinline__ unsigned ld_sys(const void* p) { return __hip_atomic_load((const gu32*)(size_t)p, __ATOMIC_RELAXED, __HIP_MEMORY_SCOPE_SYSTEM); }
__device__ __forceinline__ unsigned ld_agt(const void* p) { return __hip_atomic_load((const gu32*)(size_t)p, __ATOMIC_RELAXED, __HIP_MEMORY_SCOPE_AGENT); }
__device__ __forceinline__ void st_sys(void* p, unsigned v) { __hip_atomic_store((gu32*)(size_t)p, v, __ATOMIC_RELAXED, __HIP_MEMORY_SCOPE_SYSTEM); }
#define WAIT_V(n) asm volatile("s_waitcnt vmcnt(" #n ")" ::: "memory")
#define WAIT_L(n) asm volatile("s_waitcnt lgkmcnt(" #n ")" ::: "memory")
#define BAR __builtin_amdgcn_s_barrier()
#define BARF do { asm volatile("" ::: "memory"); __builtin_amdgcn_s_barrier(); asm volatile("" ::: "memory"); } while (0)
#define SCHED __builtin_amdgcn_sched_barrier(0)

__device__ __forceinline__ bool give_up(char* ctl, unsigned call, u64 t0, unsigned& spins) {
    if ((++spins & 31u) != 0u) return false;
    const unsigned tm = __builtin_amdgcn_readfirstlane(ld_agt(ctl + CTL_TMO));
    if (tm == call) return true;
    if (__builtin_amdgcn_s_memrealtime() - t0 > DEADLINE) {
        __hip_atomic_store((gu32*)(size_t)(ctl + CTL_TMO), call, __ATOMIC_RELAXED, __HIP_MEMORY_SCOPE_AGENT);
        return true;
    }
    return false;
}
#define DPP(v, ctrl, rows) ((unsigned)__builtin_amdgcn_update_dpp(0, (int)(v), ctrl, rows, 0xf, false))
__device__ __forceinline__ unsigned wave_max_u32(unsigned v) {
    v = max(v, DPP(v, 0x111, 0xf)); v = max(v, DPP(v, 0x112, 0xf)); v = max(v, DPP(v, 0x114, 0xf)); v = max(v, DPP(v, 0x118, 0xf));
    v = max(v, DPP(v, 0x142, 0xa)); v = max(v, DPP(v, 0x143, 0xc));
    return v;
}
__device__ __forceinline__ unsigned wave_scan_u32(unsigned v) {
    v += DPP(v, 0x111, 0xf); v += DPP(v, 0x112, 0xf); v += DPP(v, 0x114, 0xf); v += DPP(v, 0x118, 0xf);
    v += DPP(v, 0x142, 0xa); v += DPP(v, 0x143, 0xc);
    return v;
}
__device__ __forceinline__ unsigned stream_base(const u32x4 bm, unsigned area, unsigned& a) {
    const unsigned len = __builtin_popcount(bm.x) + __builtin_popcount(bm.y) + __builtin_popcount(bm.z) + __builtin_popcount(bm.w);
    const unsigned reg = (len + 3u) & ~3u, inc = wave_scan_u32(reg);
    a = area + inc - reg;
    return (unsigned)__builtin_amdgcn_readlane((int)inc, 63);
}

#define STAMP(k) do { } while (0)

__device__ __forceinline__ int rot_of(int j) { return (j * 16 + 3) / 7; }

__global__ void __launch_bounds__(512, 2) fused_gemm_quant(Params p) {
    extern __shared__ __attribute__((aligned(16))) unsigned char smem[];
    LAS unsigned char* lds = (LAS unsigned char*)smem;
    const int tid = threadIdx.x, lane = tid & 63;
    const int wid = __builtin_amdgcn_readfirstlane(tid >> 6);
    const int rank = p.rank, bid = blockIdx.x;
    char* const ws = p.self;
    char* const ctl = ws + OFF_CTL;
    STAMP(0);
    const __amdgpu_buffer_rsrc_t rc = mk_rsrc(ctl, 65536);
    const unsigned call_v = (unsigned)__builtin_amdgcn_raw_ptr_buffer_atomic_add_i32(1, rc, lane == 0 ? 0u : OOB, CTL_SEQ + bid * 64 + wid * 8, 0);
    const unsigned ef0 = __builtin_amdgcn_raw_buffer_load_b32(rc, bid >= N_EXP && bid < N_EXP + N_PULL ? lane * 4u : OOB, CTL_EXPF + ((rank + 1 + (bid - N_EXP) % 7) & 7) * 1024, AUX_SYS);
    LAS unsigned* lctl = (LAS unsigned*)(lds + LDS_CTL);
    const unsigned lds0 = (unsigned)(size_t)lds, lctl_addr = lds0 + LDS_CTL;

    if (__builtin_expect(bid < N_EXP, 1)) {
        const __amdgpu_buffer_rsrc_t rx = mk_rsrc(p.x, (unsigned)GM * KS * 2), rw = mk_rsrc(p.w, (unsigned)KS * GN * 2);
        const __amdgpu_buffer_rsrc_t rd = mk_rsrc(ws, OFF_CTL);
        constexpr int PITCH = 528;
        if (bid >= 7 * (N_PUSH + 4)) __builtin_amdgcn_s_sleep(24);
        for (int q = bid; q < 640; q += N_EXP) {
            const int c = q / 80, idx = q % 80;
            const bool isx = idx >= 56 && idx < 72, push = q < 7 * N_PUSH;
            int dst = rank, pj = 0;
            if (isx) {
                pj = idx - 56;
                u32x4 v[4];
                const unsigned src = (unsigned)((pj * 256 + wid * 32 + (lane & 15)) * (KS * 2) + c * 128 + (lane >> 4) * 16);
#pragma unroll
                for (int u = 0; u < 4; ++u) v[u] = __builtin_amdgcn_raw_buffer_load_b128(rx, src + (u >> 1) * 16 * (KS * 2) + (u & 1) * 64, 0, 0);
                const unsigned d0 = OFF_X + (unsigned)((rank * 8 + c) * 16 + pj) * PIECE + wid * 4096 + lane * 16;
#pragma unroll
                for (int u = 0; u < 4; ++u) st128<AUX_SYS>(v[u], rd, d0 + u * 1024, 0);
            } else {
                if (idx < 56) { dst = (rank + 1 + idx % 7) & 7; pj = idx / 7; } else { pj = idx - 72; }
                u32x4 v[4];
                const unsigned src = (unsigned)(c * 64 + (tid >> 5)) * (GN * 2) + (unsigned)(dst * NS + pj * 256) * 2 + (tid & 31) * 16;
#pragma unroll
                for (int u = 0; u < 4; ++u) v[u] = __builtin_amdgcn_raw_buffer_load_b128(rw, src + u * 16 * (GN * 2), 0, 0);
#pragma unroll
                for (int u = 0; u < 4; ++u) *(LAS u32x4*)(lds + ((tid >> 5) + 16 * u) * PITCH + (tid & 31) * 16) = v[u];
                __syncthreads();
                if (q == bid) STAMP(3);
                const int bj = wid >> 2, wc = wid & 3;
                const int g = lane >> 4, qq = (lane >> 2) & 3, pp = lane & 3;
                const __amdgpu_buffer_rsrc_t rw2 = mk_rsrc(push ? p.ws[dst] : ws, OFF_CTL);
                const unsigned d0 = (push ? OFF_WIN + (unsigned)(rank * 64 + pj) * PIECE : (dst == rank ? OFF_WIN : OFF_WEX) + (unsigned)((dst * 8 + c) * 8 + pj) * PIECE) + wid * 4096 + lane * 16;
#pragma unroll
                for (int u = 0; u < 4; ++u) {
                    const int n = u >> 1, kf = u & 1;
                    const int a = (kf * 32 + g * 8 + qq) * PITCH + (wc * 64 + pp * 16 + (bj * 2 + n) * 4) * 2;
                    const s16x4 lo = __builtin_amdgcn_ds_read_tr16_b64_v4i16((LAS s16x4*)(lds + a));
                    const s16x4 hi = __builtin_amdgcn_ds_read_tr16_b64_v4i16((LAS s16x4*)(lds + a + 4 * PITCH));
                    const u32x2 l2 = __builtin_bit_cast(u32x2, lo), h2 = __builtin_bit_cast(u32x2, hi);
                    const u32x4 o = {l2.x, l2.y, h2.x, h2.y};
                    st128<AUX_SYS>(o, rw2, d0 + u * 1024, 0);
                }
            }
            WAIT_V(0);
            __syncthreads();
            if (wid == 0) {
                const unsigned call = __builtin_amdgcn_readfirstlane(call_v) + 1u;
                if (isx) {
                    if (lane < NR) {
                        if (lane == rank) st_sys(ctl + CTL_XCNT + (pj * 64 + rank * 8 + c) * 4, call * 32u);
                        else st_sys(p.ws[lane] + OFF_CTL + CTL_EXPF + (rank * 256 + c * 24 + 8 + pj) * 4, call);
                    }
                } else if (lane == 0) {
                    if (dst == rank || push) st_sys(p.ws[dst] + OFF_CTL + CTL_WCNT + (pj * 64 + rank * 8 + c) * 4, call * 32u);
                    else st_sys(p.ws[dst] + OFF_CTL + CTL_EXPF + (rank * 256 + c * 24 + pj) * 4, call);
                }
            }
            if (q == bid) STAMP(1);
        }
        STAMP(2);
        return;
    }

    if (tid == 0) { lctl[6] = 0u; lctl[8] = 0u; lctl[9] = 0u; }
    if (tid >= 64 && tid < 64 + NR) *(LAS u64*)(lctl + 16 + 2 * (tid - 64)) = (u64)(size_t)p.ws[tid - 64];
    if (tid >= 128 && tid < 192) {
        const unsigned m = (tid - 128) >> 2, off = tid & 3;
        u64 cs = 0x0c0c0c0c0c0c0c0cull; unsigned es = 0x0c0c0c0cu, k = off;
        for (unsigned e = 0; e < off; ++e) cs = (cs & ~(0xffull << (8 * e))) | ((u64)(4 + e) << (8 * e));
        for (unsigned e = 0; e < 4; ++e)
            if ((m >> e) & 1u) { cs = (cs & ~(0xffull << (8 * k))) | ((u64)e << (8 * k)); es = (es & ~(0xffu << (8 * e))) | (k << (8 * e)); ++k; }
        *(LAS u32x2*)(lctl + 256 + 2 * (tid - 128)) = (u32x2){(unsigned)cs, k >= 4 ? (unsigned)(cs >> 32) : (unsigned)cs};
        lctl[64 + tid - 128] = es;
    }
    if (bid >= N_EXP + N_PULL) __syncthreads();
    const unsigned call = __builtin_amdgcn_readfirstlane(call_v) + 1u;

    if (bid < N_EXP + N_PULL) {
        const int q = bid - N_EXP, j = q % 7;
        const int src = (rank + 1 + j) & 7, rot = rot_of(j);
        const __amdgpu_buffer_rsrc_t rs = mk_rsrc(p.ws[src], OFF_CTL), rd = mk_rsrc(ws, OFF_CTL);
        const char* ef = ctl + CTL_EXPF + src * 1024 + lane * 4;
        u64 m0 = __ballot(ef0 >= call), m1 = 0, m2 = 0;
        const unsigned aoff = lane == 0 ? 0u : OOB;
        constexpr unsigned NU = (192 - N_PUSH) * 16;
        const unsigned tbase = (call - 1u) * NU - 64u, toff = CTL_TICK + j * 128;
        unsigned tk = (q / 7) * 8 + wid;
        unsigned cnt1 = 0, off1 = OOB;
        while (tk < NU) {
            const int pp = tk / 16 + N_PUSH, c = pp / 24, r = pp % 24, sc = src * 8 + c;
            int id; unsigned so, dof, cnt;
            if (r < 8) { id = c * 24 + r; so = OFF_WEX + (unsigned)((rank * 8 + c) * 8 + r) * PIECE; dof = OFF_WIN + (unsigned)(sc * 8 + r) * PIECE; cnt = CTL_WCNT + (r * 64 + sc) * 4; }
            else { const int i = (rot + r - 8) & 15; id = c * 24 + 8 + i; so = OFF_X + (unsigned)(sc * 16 + i) * PIECE; dof = so; cnt = CTL_XCNT + (i * 64 + sc) * 4; }
            so += (tk % 16) * 2048u; dof += (tk % 16) * 2048u;
            const u64 mm = id < 64 ? m0 : (id < 128 ? m1 : m2);
            if (!((mm >> (id & 63)) & 1)) {
                const u64 t0 = __builtin_amdgcn_s_memrealtime(); unsigned spins = 0;
                for (;;) {
                    const unsigned f0 = ld_sys(ef), f1 = ld_sys(ef + 256), f2 = ld_sys(ef + 512);
                    m0 = __ballot(f0 >= call); m1 = __ballot(f1 >= call); m2 = __ballot(f2 >= call);
                    const u64 mn = id < 64 ? m0 : (id < 128 ? m1 : m2);
                    if ((mn >> (id & 63)) & 1) break;
                    if (give_up(ctl, call, t0, spins)) { m0 = m1 = m2 = ~0ull; break; }
                    __builtin_amdgcn_s_sleep(2);
                }
            }
            if (off1 == OOB) STAMP(1);
            u32x4 v = __builtin_amdgcn_raw_buffer_load_b128(rs, lane * 16, so, AUX_SYS);
            WAIT_V(1);
            __builtin_amdgcn_raw_ptr_buffer_atomic_add_i32(2, rc, off1, cnt1, 0);
            WAIT_V(1);
            st128<AUX_AGT>(v, rd, lane * 16, dof);
            v = __builtin_amdgcn_raw_buffer_load_b128(rs, lane * 16, so + 1024, AUX_SYS);
            const int tnext = __builtin_amdgcn_raw_ptr_buffer_atomic_add_i32(1, rc, aoff, toff, 0);
            WAIT_V(0);
            st128<AUX_AGT>(v, rd, lane * 16, dof + 1024);
            cnt1 = cnt; off1 = aoff;
            tk = (unsigned)__builtin_amdgcn_readfirstlane(tnext) - tbase;
        }
        WAIT_V(0);
        __builtin_amdgcn_raw_ptr_buffer_atomic_add_i32(2, rc, off1, cnt1, 0);
        STAMP(2);
        __syncthreads();

        const int g = (q / 7) * 8 + wid, tA = g & 15, tB = tA ^ 8, w2 = g >> 4;
        const __amdgpu_buffer_rsrc_t ro = mk_rsrc(p.out[rank], OUT_BYTES);
        const char* ofA = ctl + CTL_OFLG + (src * 16 + tA) * 128;
        const char* ofB = ctl + CTL_OFLG + (src * 16 + tB) * 128;
        const unsigned slA = OFF_OEX + (unsigned)((rank * 16 + tA) * 8 + w2) * OSLOT, slB = OFF_OEX + (unsigned)((rank * 16 + tB) * 8 + w2 + 4) * OSLOT;
        const unsigned arA = lds0 + wid * 2 * LAREA, arB = arA + LAREA;
        const unsigned obase = (unsigned)(lane & 15) * GN + (unsigned)(src * NS + (tA & 7) * 256 + w2 * 64 + (lane >> 4) * 16);
        const unsigned rowA = (tA >> 3) * 256, rowB = (tB >> 3) * 256 + 64;
        unsigned adA[32], seA[32], adB[32], seB[32], pA, pB, lenA, lenB;
        u32x4 bmA, bmB;
#define PLAN(X, i0, i1) do { _Pragma("unroll") for (int i = i0; i < i1; ++i) { const unsigned nib = (bm##X[i >> 3] >> (4 * (i & 7))) & 15u; \
                ad##X[i] = p##X & ~3u; se##X[i] = lctl[64 + nib * 4 + (p##X & 3u)]; p##X += __builtin_popcount(nib); } \
            _Pragma("unroll") for (int i = i0; i < i1; ++i) asm volatile("" : "+v"(ad##X[i]), "+v"(se##X[i])); } while (0)
        bool ok = true;
        unsigned fl = 0;
        auto wait2 = [&](unsigned need, unsigned who) {
            const u64 t0 = __builtin_amdgcn_s_memrealtime(); unsigned spins = 0;
            for (;;) {
                fl = ld_sys(lane & 1 ? ofB : ofA);
                if ((unsigned)__ballot(fl >= need) & who) break;
                if (give_up(ctl, call, t0, spins)) { ok = false; fl = ~0u; break; }
                __builtin_amdgcn_s_sleep(8);
            }
        };
        constexpr unsigned whoA = 1, whoB = 2;
        wait2(2u * call - 1u, 3);
        const bool a_first = (unsigned)__builtin_amdgcn_readlane((int)fl, 0) >= 2u * call - 1u;
        bool both = false;
        STAMP(3);
#define WSTAMP(k) do { } while (0)
#define LOAD_EVEN(X, k) __builtin_amdgcn_raw_buffer_load_b128(rs, (k) * 1024u + lane * 16u < len##X ? lane * 16u : OOB, sl##X + 1024u + (k) * 1024u, AUX_SYS)
#define LOAD_ODD(X, k) __builtin_amdgcn_raw_buffer_load_b96(rs, (k) * 1024u + lane * 16u < len##X ? lane * 12u : OOB, sl##X + 1024u + (k) * 1024u, AUX_SYS)
#define PUT_EVEN(X, k, v) do { d7 = ((v.x >> 7) & 0x01010101u) | ((v.y >> 6) & 0x02020202u) | ((v.z >> 5) & 0x04040404u) | ((v.w >> 4) & 0x08080808u); \
            *(LAS u32x4*)(size_t)(ar##X + (k) * 1024u + lane * 16u) = v & 0x7f7f7f7fu; } while (0)
#define PUT_ODD(X, k, w) do { d7 |= ((w.x >> 3) & 0x10101010u) | ((w.y >> 2) & 0x20202020u) | ((w.z >> 1) & 0x40404040u); \
            *(LAS u32x4*)(size_t)(ar##X + (k) * 1024u + lane * 16u) = (u32x4){w.x & 0x7f7f7f7fu, w.y & 0x7f7f7f7fu, w.z & 0x7f7f7f7fu, d7}; } while (0)
#define REST(X, other) do { bool have = false; u32x4 v; \
            for (unsigned k = 2; k * 1024u < len##X; k += 2) { \
                if (!have) v = LOAD_EVEN(X, k); \
                PUT_EVEN(X, k, v); \
                if ((k + 1) * 1024u < len##X) { \
                    const u32x3 w = LOAD_ODD(X, k + 1); \
                    have = len##X <= (k + 2) * 1024u + MERGE_B; \
                    if (have) v = LOAD_EVEN(X, k + 2); \
                    const unsigned fo = __builtin_amdgcn_raw_buffer_load_b32(rc, both ? OOB : aoff, CTL_OFLG + (src * 16 + (other)) * 128, AUX_SYS); \
                    PUT_ODD(X, k + 1, w); \
                    both = both || (unsigned)__builtin_amdgcn_readfirstlane(fo) >= 2u * call; \
                } } } while (0)
#define EXPAND(X) do { unsigned hi[32], lo = *(const LAS unsigned*)(size_t)ad##X[0]; \
            _Pragma("unroll") for (int i = 0; i < 32; ++i) hi[i] = *(const LAS unsigned*)(size_t)(ad##X[i] + 4u); \
            if (!ok) { _Pragma("unroll") for (int i = 0; i < 32; ++i) se##X[i] = ~0u; }        \
            _Pragma("unroll") for (int blk = 0; blk < 8; ++blk) { u32x4 o; \
                _Pragma("unroll") for (int e = 0; e < 4; ++e) { const int i = blk * 4 + e; \
                    if (i) lo = ad##X[i] != ad##X[i - 1] ? hi[i - 1] : lo; \
                    o[e] = __builtin_amdgcn_perm(hi[i], lo, se##X[i]); } \
                st128<AUX_SYS>(o, ro, obase, (row##X + (blk >> 2) * 128 + (blk & 3) * 16) * GN); } } while (0)
#define RUN(X, Y) do { unsigned d7; \
            bm##X = __builtin_amdgcn_raw_buffer_load_b128(rs, lane * 16, sl##X, AUX_SYS); \
            const bool early = (unsigned)__ballot(fl >= 2u * call - 1u) & who##Y; \
            bm##Y = __builtin_amdgcn_raw_buffer_load_b128(rs, early ? lane * 16u : OOB, sl##Y, AUX_SYS); \
            len##X = stream_base(bm##X, ar##X, p##X); \
            PLAN(X, 0, 32); \
            if (!early) { wait2(2u * call - 1u, who##Y); bm##Y = __builtin_amdgcn_raw_buffer_load_b128(rs, lane * 16, sl##Y, AUX_SYS); } \
            len##Y = stream_base(bm##Y, ar##Y, p##Y); \
            wait2(2u * call, who##X); \
            both = (unsigned)__ballot(fl >= 2u * call) & who##Y; \
            STAMP(5); WSTAMP(0); \
            { const u32x4 v = LOAD_EVEN(X, 0u); PLAN(Y, 0, 16); PUT_EVEN(X, 0u, v); } \
            { const u32x3 w = LOAD_ODD(X, 1u); PLAN(Y, 16, 32); PUT_ODD(X, 1u, w); } \
            REST(X, t##Y); \
            WSTAMP(1); \
            if (!both) { wait2(2u * call, who##Y); both = true; } \
            const u32x4 v0 = LOAD_EVEN(Y, 0u); const u32x3 w1 = LOAD_ODD(Y, 1u);        \
            EXPAND(X); \
            WSTAMP(2); \
            PUT_EVEN(Y, 0u, v0); PUT_ODD(Y, 1u, w1); \
            REST(Y, t##X); \
            WSTAMP(3); \
            EXPAND(Y); } while (0)
        if (a_first) RUN(A, B); else RUN(B, A);
        WSTAMP(4);
        WAIT_V(0);
        WSTAMP(5);
        STAMP(4);
        return;
    }

    const int gi = bid - (N_EXP + N_PULL), xg = gi & 7, ix = gi >> 3;
    const int ti = (xg >> 1) * 4 + (ix & 3), tj = (xg & 1) * 4 + (ix >> 2);
    const int wr = wid >> 2, wc = wid & 3;
    int scv;
    {
        int j0 = 0;
#pragma unroll
        for (int j = 1; j < 7; ++j) if (rot_of(j) <= ti) j0 = j;
        const int t2 = lane - 8, c = t2 / 7, k = t2 - c * 7;
        int j = j0 - k; if (j < 0) j += 7;
        scv = lane < 8 ? rank * 8 + lane : ((rank + 1 + j) & 7) * 8 + c;
    }
    const unsigned target = call * 32u;
    u64 avail = 0, taken = 0;
    unsigned pa = 0;
    u64 wok = 0;
    bool early = false;
    const char* const pxa = ctl + CTL_XCNT + (ti * 64 + scv) * 4;
    const unsigned pwo = CTL_WCNT + tj * 256;
#define POLL_WAIT_V6(n) do { early = (n) >= EARLY_FROM && (n) < 64 && wid == 0 && avail == 0 && (wok | taken) == ~0ull; if (early) { \
        asm volatile("global_load_dword %0, %1, off sc1\n\ts_waitcnt vmcnt(7)" : "=&v"(pa) : "v"(pxa) : "memory"); \
    } else WAIT_V(6); } while (0)
    auto next_step = [&](int n) -> unsigned {
        if (avail == 0) {
            if (wid == 0) {
                if (early) {
                    asm volatile("s_waitcnt vmcnt(2)" : "+v"(pa) :: "memory");
                    avail = __ballot(pa >= target) & ~taken;
                }
                const u64 t0 = __builtin_amdgcn_s_memrealtime(); unsigned spins = 0;
                while (avail == 0) {
                    const unsigned a = ld_agt(pxa), b = __builtin_amdgcn_raw_buffer_load_b32(rc, ((wok | taken) >> lane) & 1 ? OOB : scv * 4u, pwo, AUX_AGT);
                    wok |= __ballot(b >= target);
                    avail = __ballot(a >= target) & wok & ~taken;
                    if (avail != 0) break;
                    if (give_up(ctl, call, t0, spins)) { avail = ~taken; break; }
                    if (n < 50) __builtin_amdgcn_s_sleep(100); else if (n < 56) __builtin_amdgcn_s_sleep(16); else __builtin_amdgcn_s_sleep(1);
                }
                const u32x4 mv = {(unsigned)avail, (unsigned)(avail >> 32), (unsigned)n + 1u, 0u};
                asm volatile("ds_write_b128 %0, %1" :: "v"(lctl_addr + 16), "v"(mv) : "memory");
            } else {
                for (;;) {
                    u32x4 mv;
                    asm volatile("ds_read_b128 %0, %1\n\ts_waitcnt lgkmcnt(0)" : "=v"(mv) : "v"(lctl_addr + 16) : "memory");
                    if ((unsigned)__builtin_amdgcn_readfirstlane(mv.z) == (unsigned)n + 1u) {
                        avail = ((u64)(unsigned)__builtin_amdgcn_readfirstlane(mv.y) << 32) | (unsigned)__builtin_amdgcn_readfirstlane(mv.x);
                        break;
                    }
                    __builtin_amdgcn_s_sleep(1);
                }
            }
        }
        const int bit = __builtin_ctzll(avail);
        avail &= avail - 1; taken |= 1ull << bit;
        return (unsigned)__builtin_amdgcn_readlane(scv, bit);
    };
    const char* const xbase = ws + OFF_X + (size_t)ti * PIECE;
    const char* const wbase = ws + OFF_WIN + (size_t)tj * PIECE;
#define A_OF(sc) (xbase + (size_t)((sc) * (16u * PIECE)))
#define B_OF(sc) (wbase + (size_t)((sc) * (8u * PIECE)))
    const unsigned voff = (unsigned)tid * 16u, ldsw = (unsigned)wid * 1024u;
    const int aoff = wr * 8192 + lane * 16, boff = wc * 4096 + lane * 16;
#define SA(b, h) (((b) * 2 + (h)) * HALFB)
#define SB(b, h) ((4 + (b) * 2 + (h)) * HALFB)
#define STAGE(bufoff, gbase) do { _Pragma("unroll") for (int _i = 0; _i < 2; ++_i) \
        __builtin_amdgcn_global_load_lds((const unsigned*)((gbase) + voff + _i * 8192), (LAS unsigned*)(lds + (bufoff) + ldsw + _i * 8192), 16, 0, AUX_AGT); } while (0)
#define LDA(dst, b, h) do { _Pragma("unroll") for (int m = 0; m < 4; ++m) _Pragma("unroll") for (int k = 0; k < 2; ++k) dst[m][k] = *(const LAS bf16x8*)(lds + SA(b, h) + aoff + m * 2048 + k * 1024); } while (0)
#define LDB(dst, b, h) do { _Pragma("unroll") for (int n = 0; n < 2; ++n) _Pragma("unroll") for (int k = 0; k < 2; ++k) dst[n][k] = *(const LAS bf16x8*)(lds + SB(b, h) + boff + n * 2048 + k * 1024); } while (0)
#define MMA(ai, bj, At, Bt) do { __builtin_amdgcn_s_setprio(1); _Pragma("unroll") for (int m = 0; m < 4; ++m) _Pragma("unroll") for (int n = 0; n < 2; ++n) _Pragma("unroll") for (int k = 0; k < 2; ++k) \
        acc[ai][bj][m][n] = __builtin_amdgcn_mfma_f32_16x16x32_bf16(Bt[n][k], At[m][k], acc[ai][bj][m][n], 0, 0, 0); __builtin_amdgcn_s_setprio(0); } while (0)
    f32x4 acc[2][2][4][2];
#pragma unroll
    for (int a = 0; a < 2; ++a)
#pragma unroll
        for (int b = 0; b < 2; ++b)
#pragma unroll
            for (int m = 0; m < 4; ++m)
#pragma unroll
                for (int n = 0; n < 2; ++n) acc[a][b][m][n] = (f32x4){0.f, 0.f, 0.f, 0.f};
    bf16x8 At[4][2], B0[2][2], B1[2][2];
    constexpr int NT = 64;
    const char* a1;
    {
        const unsigned s0 = next_step(0);
        const char *a0 = A_OF(s0), *b0 = B_OF(s0);
        STAGE(SB(0, 0), b0); STAGE(SA(0, 0), a0); STAGE(SB(0, 1), b0 + HALFB); STAGE(SA(0, 1), a0 + HALFB);
        if (wr == 1) BAR;
        WAIT_V(4); BAR;
        const unsigned s1 = next_step(1);
        a1 = A_OF(s1);
        const char* b1 = B_OF(s1);
        STAGE(SB(1, 0), b1); STAGE(SA(1, 0), a1); STAGE(SB(1, 1), b1 + HALFB);
        WAIT_V(6); BAR;
    }
    STAMP(1);
    for (int t = 0; t < NT - 2; t += 2) {
        LDB(B0, 0, 0); SCHED; LDA(At, 0, 0); STAGE(SA(1, 1), a1 + HALFB);
        WAIT_L(8); BAR; WAIT_L(0); MMA(0, 0, At, B0); BAR; SCHED;
        const unsigned s2 = next_step(t + 2);
        const char *a2 = A_OF(s2), *b2 = B_OF(s2);
        LDB(B1, 0, 1); STAGE(SB(0, 0), b2);
        BAR; WAIT_L(0); MMA(0, 1, At, B1); BAR;
        LDA(At, 0, 1); STAGE(SA(0, 0), a2);
        BAR; WAIT_L(0); MMA(1, 0, At, B0); BAR; SCHED;
        STAGE(SB(0, 1), b2 + HALFB);
        POLL_WAIT_V6(t + 3); BAR; MMA(1, 1, At, B1); BAR;
        LDB(B0, 1, 0); SCHED; LDA(At, 1, 0); STAGE(SA(0, 1), a2 + HALFB);
        WAIT_L(8); BAR; WAIT_L(0); MMA(0, 0, At, B0); BAR; SCHED;
        const unsigned s3 = next_step(t + 3);
        a1 = A_OF(s3);
        const char* b3 = B_OF(s3);
        LDB(B1, 1, 1); STAGE(SB(1, 0), b3);
        BAR; WAIT_L(0); MMA(0, 1, At, B1); BAR;
        LDA(At, 1, 1); STAGE(SA(1, 0), a1);
        BAR; WAIT_L(0); MMA(1, 0, At, B0); BAR; SCHED;
        STAGE(SB(1, 1), b3 + HALFB);
        POLL_WAIT_V6(t + 4); BAR; MMA(1, 1, At, B1); BAR;
    }
    STAMP(2);
    {
        STAGE(SA(1, 1), a1 + HALFB);
        if (wr == 0) BARF;
        asm volatile("" ::: "memory");
        bf16x8 A2[4][2];
#define MMF(ai, bj, At, Bt) do { _Pragma("unroll") for (int m = 0; m < 4; ++m) _Pragma("unroll") for (int n = 0; n < 2; ++n) _Pragma("unroll") for (int k = 0; k < 2; ++k) \
        acc[ai][bj][m][n] = __builtin_amdgcn_mfma_f32_16x16x32_bf16(Bt[n][k], At[m][k], acc[ai][bj][m][n], 0, 0, 0); } while (0)
        LDB(B0, 0, 0); LDA(At, 0, 0); LDB(B1, 0, 1); SCHED;
        LDA(A2, 0, 1);
        MMF(0, 0, At, B0); MMF(0, 1, At, B1); SCHED;
        WAIT_V(0); BARF;
        LDA(At, 1, 0);
        MMF(1, 0, A2, B0); SCHED;
        LDB(B0, 1, 0);
        MMF(1, 1, A2, B1); SCHED;
        LDB(B1, 1, 1); LDA(A2, 1, 1);
        MMF(0, 0, At, B0); MMF(0, 1, At, B1);
        MMF(1, 0, A2, B0); MMF(1, 1, A2, B1);
    }
    STAMP(3);

    int mx = 0;
#pragma unroll
    for (int i = 0; i < 32; ++i) {
        const i32x4 v = __builtin_bit_cast(i32x4, acc[i >> 4][(i >> 1) & 1][(i >> 2) & 3][i & 1]);
        mx = max(max(mx, v[0]), v[1]); mx = max(max(mx, v[2]), v[3]);
    }
    {
        const unsigned wm = wave_max_u32((unsigned)mx);
        if (lane == 63) __hip_atomic_fetch_max(lctl + 8, wm, __ATOMIC_RELAXED, __HIP_MEMORY_SCOPE_WORKGROUP);
    }
    __syncthreads();
    if (tid < NR) {
        char* pw = (char*)(size_t)*(LAS u64*)(lctl + 16 + 2 * tid);
        __hip_atomic_store((gu64*)(size_t)(pw + OFF_CTL + CTL_AMAX + (rank * 128 + gi) * 8), ((u64)call << 32) | lctl[8], __ATOMIC_RELAXED, __HIP_MEMORY_SCOPE_SYSTEM);
    }
    STAMP(6);
    const int owner = ti >> 1;
    const bool local = owner == rank;
    char* optr = p.out[owner];
    char* oflg = p.ws[owner] + OFF_CTL + CTL_OFLG + (rank * 16 + (ti & 1) * 8 + tj) * 128;
    asm volatile("" : "+s"(optr), "+s"(oflg));
    const __amdgpu_buffer_rsrc_t ro = mk_rsrc(optr, OUT_BYTES), rx = mk_rsrc(ws + OFF_OEX, 16u << 20);
    const unsigned slot = (unsigned)((owner * 16 + (ti & 1) * 8 + tj) * 8 + wid) * OSLOT, area = lds0 + wid * 16384;
#define ACC(i) acc[(i) >> 4][((i) >> 1) & 1][((i) >> 2) & 3][(i) & 1]
    unsigned addr[33], len = 0;
    u32x2 sel[32];
    if (!local) {
        u32x4 bm;
#pragma unroll
        for (int k = 0; k < 4; ++k) {
            unsigned w = 0;
#pragma unroll
            for (int i = 31; i >= 0; --i) w = __builtin_amdgcn_alignbit(w, __float_as_uint(ACC(k * 8 + (i >> 2))[i & 3]), 31);
            bm[k] = ~w;
        }
        st128<AUX_SYS>(bm, rx, lane * 16, slot);
        unsigned a;
        const unsigned spare = area + 8192u + lane * 4u;
        len = stream_base(bm, area, a);
#pragma unroll
        for (int i = 0; i < 32; ++i) {
            const unsigned nib = (bm[i >> 3] >> (4 * (i & 7))) & 15u;
            sel[i] = *(const LAS u32x2*)(size_t)(((nib << 5) | (lctl_addr + 1024u)) | ((a << 3) & 0x18u));
            addr[i] = nib ? a & ~3u : spare; a += __builtin_popcount(nib);
            if (i == 15) {
                WAIT_V(0);
                BARF;
                if (tid == 0) st_sys(oflg, 2u * call - 1u);
            }
        }
        addr[32] = (a & 3u) ? a & ~3u : spare;
#pragma unroll
        for (int i = 0; i < 32; ++i) asm volatile("" : "+v"(addr[i]), "+v"(sel[i]));
    }
    STAMP(7);
    unsigned gmx, tmo;
    {
        const u64 t0 = __builtin_amdgcn_s_memrealtime(); unsigned spins = 0;
        for (;;) {
            const u32x4 g = __builtin_amdgcn_raw_buffer_load_b128(rc, tid * 16, CTL_AMAX, AUX_SYS);
            tmo = ld_agt(ctl + CTL_TMO);
            gmx = max(g.x, g.z);
            if (__all(g.y >= call && g.w >= call)) break;
            if (give_up(ctl, call, t0, spins)) { tmo = call; break; }
            __builtin_amdgcn_s_sleep(1);
        }
    }
    if (tmo == call) gmx = 0xffffffffu;
    gmx = wave_max_u32(gmx);
    if (lane == 63) __hip_atomic_fetch_max(lctl + 9, gmx, __ATOMIC_RELAXED, __HIP_MEMORY_SCOPE_WORKGROUP);
    __syncthreads();
    STAMP(4);
    const float inv = 448.0f / __uint_as_float(lctl[9]);
    const f32x2 inv2 = {inv, inv};
    auto quant4 = [&](int i, bool relu) -> unsigned {
        f32x4 v = ACC(i);
        if (relu) {
            i32x4 iv = __builtin_bit_cast(i32x4, v);
#pragma unroll
            for (int e = 0; e < 4; ++e) iv[e] = max(iv[e], 0);
            v = __builtin_bit_cast(f32x4, iv);
        }
        const f32x2 lo = (f32x2){v[0], v[1]} * inv2, hi = (f32x2){v[2], v[3]} * inv2;
        const int r = __builtin_amdgcn_cvt_pk_fp8_f32(lo.x, lo.y, __float_as_int(lo.x), false);
        return (unsigned)__builtin_amdgcn_cvt_pk_fp8_f32(hi.x, hi.y, r, true);
    };
    if (local) {
        const unsigned obase = (unsigned)((ti & 1) * 256 + wr * 64 + (lane & 15)) * GN + (unsigned)(rank * NS + tj * 256 + wc * 64 + (lane >> 4) * 16);
#pragma unroll
        for (int blk = 0; blk < 8; ++blk)
            st128<AUX_SYS>((u32x4){quant4(blk * 4, true), quant4(blk * 4 + 1, true), quant4(blk * 4 + 2, true), quant4(blk * 4 + 3, true)}, ro, obase,
                                                   (unsigned)((blk >> 2) * 128 + (blk & 3) * 16) * GN);
    } else {
        unsigned carry = 0;
#pragma unroll
        for (int i = 0; i < 32; ++i) {
            const unsigned r = quant4(i, false);
            *(volatile LAS unsigned*)(size_t)addr[i] = __builtin_amdgcn_perm(carry, r, sel[i].x);
            carry = __builtin_amdgcn_perm(carry, r, sel[i].y);
        }
        *(volatile LAS unsigned*)(size_t)addr[32] = carry;
        for (unsigned k = 0; k * 1024u < len; k += 2) {
            const u32x4 lo = *(volatile LAS u32x4*)(size_t)(area + k * 1024u + lane * 16u), hi = *(volatile LAS u32x4*)(size_t)(area + (k + 1) * 1024u + lane * 16u);
            unsigned d[7] = {lo.x, lo.y, lo.z, lo.w, hi.x, hi.y, hi.z};
#pragma unroll
            for (int b = 0; b < 7; ++b) d[b] = (d[b] & 0x7f7f7f7fu) | ((hi.w << (7 - b)) & 0x80808080u);
            st128<AUX_SYS>((u32x4){d[0], d[1], d[2], d[3]}, rx, k * 1024u + lane * 16u < len ? lane * 16u : OOB, slot + 1024u + k * 1024u);
            st96<AUX_SYS>((u32x3){d[4], d[5], d[6]}, rx, (k + 1) * 1024u + lane * 16u < len ? lane * 12u : OOB, slot + 2048u + k * 1024u);
        }
        STAMP(8);
        WAIT_V(0);
        BARF;
        if (tid == 0) st_sys(oflg, 2u * call);
    }
    STAMP(5);
}

}

extern "C" void kernel_launch(int rank, int nranks, ncclComm_t comm,
                              hipStream_t stream,
                              void* const* d_in, const int* in_sizes, int n_in,
                              void* d_out, int out_size,
                              const int* in_dtypes, int out_dtype,
                              void* const* d_out_all,
                              void* const* d_ws_all, size_t ws_bytes) {
    (void)comm; (void)in_dtypes; (void)out_dtype; (void)d_out;
    if (nranks != NR || (unsigned)rank >= (unsigned)NR || n_in < 2 || ws_bytes < WS_NEED || in_sizes[0] != GM * KS ||
        in_sizes[1] != KS * GN || (unsigned)out_size != OUT_BYTES) {
        fprintf(stderr, "kernel_launch: unexpected problem shape\n");
        return;
    }
    HIP_CHECK(hipFuncSetAttribute((const void*)fused_gemm_quant, hipFuncAttributeMaxDynamicSharedMemorySize, LDS_BYTES));
    Params p;
    p.x = (const char*)d_in[0];
    p.w = (const char*)d_in[1];
    for (int r = 0; r < NR; ++r) { p.ws[r] = (char*)d_ws_all[r]; p.out[r] = (char*)d_out_all[r]; }
    p.rank = rank;
    p.self = (char*)d_ws_all[rank];
    hipLaunchKernelGGL(fused_gemm_quant, dim3(N_WG), dim3(512), LDS_BYTES, stream, p);
    HIP_CHECK(hipGetLastError());
}
```
